```python
import numpy as np

kernel_name = "gpu_ag_gemm_m4096_k4096_n4096_f32_relu"


ABSMAX_RTOL = 0.001

COMPUTE_DTYPE = "float32"
EPILOGUE = "relu"
IN_QUANT = None
OUT_QUANT = None
IN_DTYPES = {"x": "float32", "w_mat": "float32"}
WORLD_SIZE = 8

def setup_inputs():

    x = np.random.randn(4096, 4096).astype(np.float32)
    w_mat = (np.random.randn(4096, 4096) / np.sqrt(4096)).astype(np.float32)
    return {"x": x, "w_mat": w_mat}

def _epilogue(y):
    return np.maximum(y, 0.0).astype(np.float32)

def reference(x, w_mat):
    y = (x @ w_mat).astype(np.float32)
    return _epilogue(y)
```

```cpp
#include <hip/hip_runtime.h>
#include <rccl/rccl.h>
#include <cstdio>

#define HIP_CHECK(expr) do { hipError_t _e = (expr); if (_e != hipSuccess) { \
    fprintf(stderr, "HIP error %s at %s:%d\n", hipGetErrorString(_e), __FILE__, __LINE__); \
    return; } } while (0)

namespace {

typedef float f4 __attribute__((ext_vector_type(4)));
typedef unsigned u4 __attribute__((ext_vector_type(4)));
typedef unsigned u2 __attribute__((ext_vector_type(2)));
typedef _Float16 h8 __attribute__((ext_vector_type(8)));
typedef __amdgpu_buffer_rsrc_t rsrc_t;

#define CFG_ND 23
#define CFG_S1 40
#define CFG_S3 24
#define CFG_GATEBACK 1
#define CFG_OUTWIN 0
#define CFG_STAGE_DELAY 0
#define CFG_NT 0
#define CFG_POLL_SLEEP 2
#define CFG_ROTMASK 7
#define CFG_ROTMUL 1

constexpr int NR = 8;
constexpr int ND = CFG_ND;
constexpr int NXB = 32 - ND;
constexpr int S1 = CFG_S1;
constexpr int S3 = CFG_S3;
constexpr int NSTEP = ND * 32;
constexpr int NWST = 256;
constexpr int E2 = NWST + NXB * 32;
constexpr int GATE_STEP = E2 - CFG_GATEBACK;
static_assert(ND >= 20 && ND <= 24, "slot table assumes <= 3 D blocks and <= 2 X blocks per group");
static_assert(S1 <= 128 && S3 <= S1 && GATE_STEP >= S1, "sender configuration");

constexpr unsigned long long DEADLINE = 40000000ull;

constexpr unsigned OFF_PL = 0;
constexpr unsigned OFF_DONE = 40960;
constexpr unsigned OFF_TMO = 42240;
constexpr unsigned OFF_CNT = 45056;
constexpr unsigned OFF_CTR = 65536;
constexpr unsigned OFF_DIAG = 131072;
constexpr unsigned OFF_X = 1u << 20;
constexpr unsigned OFF_W = 65u << 20;
constexpr unsigned WS_USED = 81u << 20;
constexpr int CI_TICK = 0;
constexpr int CI_SX = 2;
constexpr int CI_SW = 34;
constexpr int CI_LX = 50;
constexpr int CI_LW = 82;
constexpr int CI_END = 146;
constexpr int NSLOT = 21;
constexpr int F_W = 21;
constexpr int F_GATE = 25;
constexpr int F_ABORT = 31;

constexpr int NLDSK = 14;
constexpr int NBACK = 15;
constexpr int L_B = 0;
constexpr int L_PART = 4 * NLDSK * 2048;
constexpr int L_BACK = L_PART + 16384;
constexpr int L_CTL = L_BACK + NBACK * 2048;
constexpr int C_QOFF = 0;
constexpr int C_QSLOT = 24;
constexpr int C_PCNT = 48;
constexpr int C_RED = 50;
constexpr int C_FL = 51;
constexpr int C_MASK = 52;
constexpr int C_WORDS = 64;
constexpr int LDS_BYTES = L_CTL + C_WORDS * 4;
static_assert(LDS_BYTES <= 163840, "LDS");

struct Params {
  const float* x;
  const float* w;
  char* ws[NR];
  float* out[NR];
  int rank;
};

#define WAITV0() do { asm volatile("" ::: "memory"); __builtin_amdgcn_s_waitcnt(0x0F70); asm volatile("" ::: "memory"); } while (0)
#define CBAR() asm volatile("" ::: "memory")

__device__ __forceinline__ rsrc_t mk_rsrc(const void* p, unsigned bytes) {
  return __builtin_amdgcn_make_buffer_rsrc(const_cast<void*>(p), 0, bytes, 0x00020000);
}
__device__ __forceinline__ f4 ld_in(rsrc_t r, int voff, int soff) {
  return __builtin_bit_cast(f4, __builtin_amdgcn_raw_buffer_load_b128(r, voff, soff, CFG_NT));
}
__device__ __forceinline__ u4 ld_sys(rsrc_t r, int voff, int soff) {
  return __builtin_amdgcn_raw_buffer_load_b128(r, voff, soff, 17);
}
__device__ __forceinline__ u4 ld_sys_nt(rsrc_t r, int voff, int soff) {
  return __builtin_amdgcn_raw_buffer_load_b128(r, voff, soff, 17 | CFG_NT);
}
__device__ __forceinline__ unsigned ld_sys32(rsrc_t r, int voff, int soff) {
  return __builtin_amdgcn_raw_buffer_load_b32(r, voff, soff, 17);
}
__device__ __forceinline__ void st_sys(u4 v, rsrc_t r, int voff, int soff) {
  __builtin_amdgcn_raw_buffer_store_b128(v, r, voff, soff, 17);
}
__device__ __forceinline__ void st_sys32(unsigned v, rsrc_t r, int voff, int soff) {
  __builtin_amdgcn_raw_buffer_store_b32(v, r, voff, soff, 17);
}
__device__ __forceinline__ u4 cvt8(f4 a, f4 b) {
  h8 h = h8{(_Float16)a.x, (_Float16)a.y, (_Float16)a.z, (_Float16)a.w,
            (_Float16)b.x, (_Float16)b.y, (_Float16)b.z, (_Float16)b.w};
  return __builtin_bit_cast(u4, h);
}
template <int I>
__device__ __forceinline__ u4 cvtcol(const f4* r) {
  h8 h = h8{(_Float16)r[0][I], (_Float16)r[1][I], (_Float16)r[2][I], (_Float16)r[3][I],
            (_Float16)r[4][I], (_Float16)r[5][I], (_Float16)r[6][I], (_Float16)r[7][I]};
  return __builtin_bit_cast(u4, h);
}
__device__ __forceinline__ u4 cvtcol_i(const f4* r, int i) {
  return i == 0 ? cvtcol<0>(r) : i == 1 ? cvtcol<1>(r) : i == 2 ? cvtcol<2>(r) : cvtcol<3>(r);
}
__device__ __forceinline__ void load_xpiece(f4* r, rsrc_t rx, int vx, int pi) {
  const int so = (pi >> 5) * (16 * 16384) + (pi & 31) * 512;
#pragma unroll
  for (int t = 0; t < 8; ++t) r[t] = ld_in(rx, vx + t * 64, so);
}
__device__ __forceinline__ void load_wgran(f4* r, rsrc_t rw, int vw, int cg, int kg) {
  const int so = kg * (64 * 2048) + cg * 128;
#pragma unroll
  for (int j = 0; j < 8; ++j) r[j] = ld_in(rw, vw + (j & 1) * 2048, so + (16 * (j >> 2) + (j & 2)) * 2048);
}
__device__ __forceinline__ unsigned lds_ld(const unsigned* p) {
  return __hip_atomic_load(p, __ATOMIC_RELAXED, __HIP_MEMORY_SCOPE_WORKGROUP);
}
__device__ __forceinline__ void lds_st(unsigned* p, unsigned v) {
  __hip_atomic_store(p, v, __ATOMIC_RELAXED, __HIP_MEMORY_SCOPE_WORKGROUP);
}

__device__ __forceinline__ int src_off(int d) { return d == 0 ? 5 : d < 4 ? 3 * (d - 1) : 2 * (d - 4) + 1; }
__device__ __forceinline__ int src_sbase(int d) { return d == 0 ? 0 : d < 4 ? 4 + 3 * (d - 1) : 13 + 2 * (d - 4); }
__device__ __forceinline__ int src_lo(int d) { return d < 4 ? 0 : ND; }
__device__ __forceinline__ int flag_off(int d, int b, int cu) {
  return ((cu * 8 + ((b + src_off(d)) & 7)) * 32 + src_sbase(d) + ((b - src_lo(d)) >> 3)) * 4;
}
__device__ __forceinline__ bool slot_block(int slot, int gx, bool helper, int q, int& d, int& b) {
  d = slot < 4 ? 0 : slot < 13 ? 1 + (slot - 4) / 3 : (4 + (slot - 13) / 2) & 7;
  const int lo = src_lo(d);
  const int hi = d == 0 ? (helper ? ND : 32) : d < 4 ? ND : ((helper && d == 4 + q) ? ND : 32);
  b = lo + ((gx - src_off(d) - lo) & 7) + 8 * (slot - src_sbase(d));
  return slot < NSLOT && b < hi;
}

constexpr bool k_in_lds(int t) { return ((t & 15) & 1) && (t & 15) != 15; }
constexpr int lds_index(int t) { return (t >> 4) * 7 + ((t & 15) >> 1); }
constexpr int reg_index(int t) { return (t >> 4) * 9 + ((t & 15) == 15 ? 8 : ((t & 15) >> 1)); }

#define STAMP(idx) do {} while (0)

__global__ void __launch_bounds__(512, 2) ag_gemm_relu(Params p) {
  __shared__ __attribute__((aligned(16))) char smem[LDS_BYTES];
  unsigned* ctl = (unsigned*)(smem + L_CTL);
  const int lane = threadIdx.x & 63;
  const int wid = __builtin_amdgcn_readfirstlane(threadIdx.x >> 6);
  const int gx = blockIdx.x & 7;
  const int cu = blockIdx.x >> 3;
  const bool helper = cu >= 16;
  const int q = (cu - 16) >> 2;
  const rsrc_t rws = mk_rsrc(p.ws[0], WS_USED);
  const rsrc_t rx = mk_rsrc(p.x, 512u * 4096u * 4u);
  const rsrc_t rw = mk_rsrc(p.w, 4096u * 512u * 4u);
  const int vx = (lane & 15) * 16384 + (lane >> 4) * 16;
  const int vw = (32 * (lane >> 5) + 4 * ((lane >> 3) & 3)) * 2048 + (lane & 7) * 16;
  const int vl = lane * 16;

  const unsigned cprev = ld_sys32(rws, 0, OFF_CTR + blockIdx.x * 64);

  const rsrc_t rcnt = mk_rsrc(p.ws[0] + OFF_CNT, CI_END * 64);

  if (wid == 6) {
    const int j = blockIdx.x;
    const bool active = j < S1;
    f4 rd[8], ra[8], rb[24];
    if (active) {
      load_xpiece(rd, rx, vx, j);
      load_wgran(ra, rw, vw, j & 3, j >> 2);
#pragma unroll
      for (int t = 0; t < 3; ++t) load_wgran(rb + 8 * t, rw, vw, 4 * (t + 1) + (j & 3), j >> 2);
    }
    const unsigned call = __builtin_amdgcn_readfirstlane(cprev) + 1u;
    asm volatile("" :: "s"(call) : "memory");
    __builtin_amdgcn_s_barrier();
    if (!active) {
      if (blockIdx.x == 255) {
        const unsigned long long t0 = __builtin_amdgcn_s_memrealtime();
        for (;;) {
          const unsigned a0 = ld_sys32(rws, lane * 4, OFF_DONE);
          const unsigned a1 = ld_sys32(rws, lane * 4, OFF_DONE + 256);
          const unsigned tm = ld_sys32(rws, 0, OFF_TMO);
          if (__ballot((int)(a0 - call) < 0 || (int)(a1 - call) < 0) == 0ull) break;
          if (__builtin_amdgcn_readfirstlane(tm) == call || __builtin_amdgcn_s_memrealtime() - t0 > DEADLINE) {
            st_sys32(call, rws, lane == 0 ? 0 : (int)WS_USED, OFF_TMO);
            const rsrc_t ro = mk_rsrc(p.out[0], 4096u * 512u * 4u);
            const unsigned long long bad = __ballot((int)(a0 - call) < 0);
            const int f = bad ? __builtin_ctzll(bad) : 64 + __builtin_ctzll(__ballot((int)(a1 - call) < 0));
            const int row = ((p.rank ^ (4 + (f >> 5))) & 7) * 512 + (((f & 7) - src_off(0)) & 7) * 16;
            st_sys32(0x7fc00000u, ro, lane == 0 ? 0 : 0x40000000, row * 2048 + ((f >> 5) * 4 + ((f >> 3) & 3)) * 128);
            st_sys32(0x7fc00000u, ro, lane == 0 ? 0 : 0x40000000, p.rank * 512 * 2048);
            break;
          }
          __builtin_amdgcn_s_sleep(2);
        }
        STAMP(7);
      }
      return;
    }
    __builtin_amdgcn_s_setprio(2);
    STAMP(0);
    const unsigned par = call & 1u;
    rsrc_t rp[NR];
#pragma unroll
    for (int d = 1; d < NR; ++d) rp[d] = mk_rsrc(p.ws[d], WS_USED);
    if (j == 0) st_sys32(0u, rcnt, lane == 0 ? 0 : CI_END * 64, (CI_TICK + (par ^ 1u)) * 64);
    const int xbase = OFF_X + par * (32u << 20);
    const int wbase = OFF_W + par * (8u << 20);
    const int lcu = lane & 31;
    const int voob = lane < 32 ? 0 : (int)WS_USED;
    int i = j, prev = -1;
    WAITV0();
    for (;;) {
      const bool quit = i >= NSTEP || (i >= E2 && j >= S3);
      int cv = CI_END;
      if (lane == 0 && !quit) cv = CI_TICK + par;
      if (lane == 1 && prev >= 0) cv = CI_SX + (prev >> 5);
      if (lane == 2 && prev >= 0 && prev < E2) cv = prev < NWST ? CI_SW + (prev >> 6) * 4 + (prev & 3) : CI_SX + ND + ((prev - NWST) >> 5);
      const int tv = __builtin_amdgcn_raw_ptr_buffer_atomic_add_i32(1, rcnt, cv * 64, 0, 0);
      int t0v, t1v, t2v;
      if (i == GATE_STEP) {
        st_sys32(call, rws, (128 + lane) * 128, OFF_PL + F_GATE * 4);
        st_sys32(call, rws, (192 + lane) * 128, OFF_PL + F_GATE * 4);
      }
      if (i < NWST) {
        const int wo = wbase + (16 + (i & 3)) * 262144 + (i >> 2) * 4096;
#pragma unroll
        for (int t = 0; t < 4; ++t) {
          const u4 h = cvt8(rd[2 * t], rd[2 * t + 1]);
#pragma unroll
          for (int d = 1; d < 4; ++d) st_sys(h, rp[d], vl + t * 1024, xbase + d * (4 << 20) + i * 4096);
          st_sys(cvtcol_i(ra, t), rp[4], vl + t * 1024, wo);
#pragma unroll
          for (int d = 5; d < 8; ++d) st_sys(cvtcol_i(rb + 8 * (d - 5), t), rp[d], vl + t * 1024, wo + (d - 4) * (4 * 262144));
        }
        t0v = __builtin_amdgcn_readlane(tv, 0); t1v = __builtin_amdgcn_readlane(tv, 1); t2v = __builtin_amdgcn_readlane(tv, 2);
      } else if (i < E2) {
        const int xi = NSTEP + i - NWST;
#pragma unroll
        for (int t = 0; t < 4; ++t) {
          const u4 h = cvt8(rd[2 * t], rd[2 * t + 1]);
          const u4 g = cvt8(ra[2 * t], ra[2 * t + 1]);
#pragma unroll
          for (int d = 1; d < 4; ++d) st_sys(h, rp[d], vl + t * 1024, xbase + d * (4 << 20) + i * 4096);
#pragma unroll
          for (int d = 4; d < 8; ++d) st_sys(g, rp[d], vl + t * 1024, xbase + d * (4 << 20) + xi * 4096);
        }
        t0v = __builtin_amdgcn_readlane(tv, 0); t1v = __builtin_amdgcn_readlane(tv, 1); t2v = __builtin_amdgcn_readlane(tv, 2);
      } else if (i < NSTEP) {
#pragma unroll
        for (int t = 0; t < 4; ++t) {
          const u4 h = cvt8(rd[2 * t], rd[2 * t + 1]);
#pragma unroll
          for (int d = 1; d < 4; ++d) st_sys(h, rp[d], vl + t * 1024, xbase + d * (4 << 20) + i * 4096);
        }
        t0v = __builtin_amdgcn_readlane(tv, 0); t1v = __builtin_amdgcn_readlane(tv, 1); t2v = __builtin_amdgcn_readlane(tv, 2);
      } else {
        t0v = __builtin_amdgcn_readlane(tv, 0); t1v = __builtin_amdgcn_readlane(tv, 1); t2v = __builtin_amdgcn_readlane(tv, 2);
      }
      if (prev >= 0) {
        if ((t1v & 31) == 31) {
#pragma unroll
          for (int d = 1; d < 4; ++d) st_sys32(call, rp[d], voob + lcu * 1024, OFF_PL + flag_off(d, prev >> 5, 0));
        }
        if (prev < NWST) {
          if ((t2v & 15) == 15) {
#pragma unroll
            for (int d = 4; d < 8; ++d)
              st_sys32(call, rp[d], lane < 8 ? lane * 128 : (int)WS_USED, OFF_PL + (16 + 4 * (d - 4) + (prev & 3)) * 1024 + (F_W + (prev >> 6)) * 4);
          }
        } else if (prev < E2) {
          if ((t2v & 31) == 31) {
#pragma unroll
            for (int d = 4; d < 8; ++d) st_sys32(call, rp[d], voob + lcu * 1024, OFF_PL + flag_off(d, ND + ((prev - NWST) >> 5), 0));
          }
        }
      }
      if (i >= NSTEP) break;
      int nxt = t0v + S1;
      if (quit || (unsigned)nxt >= (unsigned)NSTEP) nxt = NSTEP;
      if (nxt < NSTEP) {
        load_xpiece(rd, rx, vx, nxt);
        if (nxt < NWST) {
          load_wgran(ra, rw, vw, nxt & 3, nxt >> 2);
#pragma unroll
          for (int t = 0; t < 3; ++t) load_wgran(rb + 8 * t, rw, vw, 4 * (t + 1) + (nxt & 3), nxt >> 2);
        } else if (nxt < E2) {
          load_xpiece(ra, rx, vx, NSTEP + nxt - NWST);
        }
      }
      WAITV0();
      prev = i;
      i = nxt;
    }
    STAMP(1);
    return;
  }

  int sd, sb;
  const bool sused = slot_block(lane, gx, helper, q, sd, sb);
  const unsigned needmask = (unsigned)__ballot(sused);

  const int nneed = __builtin_popcount(needmask);

  if (wid >= 4) {
    if (wid == 4) lds_st(ctl + lane, 0u);
    const unsigned call = __builtin_amdgcn_readfirstlane(cprev) + 1u;
    asm volatile("s_waitcnt lgkmcnt(0)" :: "s"(call) : "memory");
    __builtin_amdgcn_s_barrier();
    CBAR();
    __builtin_amdgcn_s_setprio(2);

    if (wid == 4) {
      const unsigned want = needmask | (15u << F_W) | (helper ? 1u << F_GATE : 0u);
      const int pv = lane < 32 ? (int)(OFF_PL + blockIdx.x * 128 + lane * 4) : lane == 32 ? (int)OFF_TMO : (int)WS_USED;
      const unsigned boff = OFF_X + (call & 1u) * (32u << 20) + sd * (4 << 20) + sb * 131072;
      unsigned seen = 0;
      const unsigned long long t0 = __builtin_amdgcn_s_memrealtime();
      for (;;) {
        const unsigned fv = ld_sys32(rws, pv, 0);
        const unsigned m = (unsigned)__ballot((int)(fv - call) >= 0) & want;
        if (m != seen) {
          const unsigned fresh = m & ~seen & needmask;
          if (lane < 32 && ((fresh >> lane) & 1u)) {
            const int pos = __builtin_popcount(seen & needmask) + __builtin_popcount(fresh >> lane) - 1;
            lds_st(ctl + C_QSLOT + pos, (unsigned)lane);
            lds_st(ctl + C_QOFF + pos, boff);
          }
          if (lane == 0) lds_st(ctl + C_MASK, m);
          seen = m;
          if (m == want) break;
        }
        if (__builtin_amdgcn_readlane(fv, 32) == call || __builtin_amdgcn_s_memrealtime() - t0 > DEADLINE) {
          st_sys32(call, rws, lane == 0 ? 0 : (int)WS_USED, OFF_TMO);
          if (lane == 0) lds_st(ctl + C_MASK, seen | (1u << F_ABORT));
          break;
        }
        __builtin_amdgcn_s_sleep(CFG_POLL_SLEEP);
      }
      STAMP(4);
    } else if (wid == 5) {
      unsigned pending = needmask;
      const int rowoff = (((p.rank ^ sd) & 7) * 512 + sb * 16) * 2048;
      float* outp = p.out[0];
      if (helper) outp = q == 0 ? p.out[4] : q == 1 ? p.out[5] : q == 2 ? p.out[6] : p.out[7];
      const rsrc_t ro = mk_rsrc(outp, 4096u * 512u * 4u);
      const int vo = (lane >> 3) * 2048 + (lane & 7) * 16 + (cu & 15) * 128;
      int bi = 0;
      for (int n = 0; n < nneed; ++n) {
        unsigned c;
        bool abort;
        for (;;) {
          c = lds_ld(ctl + lane);
          const unsigned mk = __builtin_amdgcn_readlane(c, C_MASK);
          abort = (mk >> F_ABORT) & 1u;
          if (abort || ((int)__builtin_amdgcn_readlane(c, C_RED) > n && (!helper || ((mk >> F_GATE) & 1u)))) break;
          __builtin_amdgcn_s_sleep(1);
        }
        if (abort) break;
        CBAR();
        const int sl = __builtin_amdgcn_readlane(c, C_QSLOT + n);
        const int ro_s = __builtin_amdgcn_readlane(rowoff, sl);
        pending &= ~(1u << sl);
        const u4 o0 = *(const u4*)(smem + L_BACK + bi * 2048 + vl);
        const u4 o1 = *(const u4*)(smem + L_BACK + bi * 2048 + 1024 + vl);
        if (helper) {
          if (CFG_OUTWIN == 0) asm volatile("s_waitcnt vmcnt(0)" ::: "memory");
          else asm volatile("s_waitcnt vmcnt(2)" ::: "memory");
        }
        st_sys(o0, ro, vo, ro_s);
        st_sys(o1, ro, vo, ro_s + 16384);
        CBAR();
        if (lane == 0) lds_st(ctl + C_FL, (unsigned)(n + 1));
        bi = bi + 1 == NBACK ? 0 : bi + 1;
      }
      while (pending != 0u) {
        const int sl = __builtin_ctz(pending);
        pending &= pending - 1u;
        const int ro_s = __builtin_amdgcn_readlane(rowoff, sl);
        const u4 o = u4{0x7fc00000u, 0x7fc00000u, 0x7fc00000u, 0x7fc00000u};
        st_sys(o, ro, vo, ro_s);
        st_sys(o, ro, vo, ro_s + 16384);
      }
      if (helper) {
        WAITV0();
        char* wsq = q == 0 ? p.ws[4] : q == 1 ? p.ws[5] : q == 2 ? p.ws[6] : p.ws[7];
        if (lane == 0)
          __hip_atomic_store((unsigned*)(wsq + OFF_DONE) + q * 32 + (cu & 3) * 8 + gx, call, __ATOMIC_RELAXED, __HIP_MEMORY_SCOPE_SYSTEM);
      }
      STAMP(5);
    } else {
      st_sys32(call, rws, lane == 0 ? 0 : (int)WS_USED, OFF_CTR + blockIdx.x * 64);
      STAMP(6);
      const int wo = (lane >> 4) * 512 + (lane & 7) * 16 + ((lane >> 3) & 1) * 4;
      int bi = 0;
      for (int n = 0; n < nneed; ++n) {
        bool abort;
        for (;;) {
          const unsigned c = lds_ld(ctl + lane);
          abort = (__builtin_amdgcn_readlane(c, C_MASK) >> F_ABORT) & 1u;
          const int pc = (n & 1) ? __builtin_amdgcn_readlane(c, C_PCNT + 1) : __builtin_amdgcn_readlane(c, C_PCNT);
          if (abort || (pc >= 4 * ((n >> 1) + 1) && n - (int)__builtin_amdgcn_readlane(c, C_FL) < NBACK)) break;
          __builtin_amdgcn_s_sleep(1);
        }
        if (abort) break;
        CBAR();
        const char* pp = smem + L_PART + (n & 1) * 8192 + vl;
        float* bo = (float*)(smem + L_BACK + bi * 2048 + wo);
#pragma unroll
        for (int m = 0; m < 2; ++m) {
          const f4 v = (*(const f4*)(pp + m * 1024) + *(const f4*)(pp + 2048 + m * 1024)) +
                       (*(const f4*)(pp + 4096 + m * 1024) + *(const f4*)(pp + 6144 + m * 1024));
#pragma unroll
          for (int jj = 0; jj < 4; ++jj) bo[jj * 32 + m * 2] = v[jj] < 0.f ? 0.f : v[jj];
        }
        CBAR();
        if (lane == 0) lds_st(ctl + C_RED, (unsigned)(n + 1));
        bi = bi + 1 == NBACK ? 0 : bi + 1;
      }
    }
    return;
  }

  {
    const int gi = blockIdx.x * 4 + wid;
    const unsigned call = __builtin_amdgcn_readfirstlane(cprev) + 1u;
    asm volatile("" :: "s"(call) : "memory");
    __builtin_amdgcn_s_barrier();
    CBAR();
    if (CFG_STAGE_DELAY > 0) __builtin_amdgcn_s_sleep(CFG_STAGE_DELAY);
    f4 sx[8], sw[8];
    load_xpiece(sx, rx, vx, gi);
    load_wgran(sw, rw, vw, gi >> 6, gi & 63);
    const unsigned par = call & 1u;
    const int xbase = OFF_X + par * (32u << 20);
    const int wbase = OFF_W + par * (8u << 20);
#pragma unroll
    for (int t = 0; t < 4; ++t) st_sys(cvtcol_i(sw, t), rws, vl + t * 1024, wbase + gi * 4096);
#pragma unroll
    for (int t = 0; t < 4; ++t) st_sys(cvt8(sx[2 * t], sx[2 * t + 1]), rws, vl + t * 1024, xbase + gi * 4096);
    WAITV0();
    {
      const int tv = __builtin_amdgcn_raw_ptr_buffer_atomic_add_i32(1, rcnt, (lane == 0 ? CI_LX + (gi >> 5) : lane == 1 ? CI_LW + (gi >> 4) : CI_END) * 64, 0, 0);
      if ((__builtin_amdgcn_readlane(tv, 0) & 31) == 31)
        st_sys32(call, rws, lane < 32 ? lane * 1024 : (int)WS_USED, OFF_PL + flag_off(0, gi >> 5, 0));
      if ((__builtin_amdgcn_readlane(tv, 1) & 15) == 15)
        st_sys32(call, rws, lane < 8 ? lane * 128 : (int)WS_USED, OFF_PL + (gi >> 6) * 1024 + (F_W + ((gi >> 4) & 3)) * 4);
    }

    for (;;) {
      const unsigned mk = __builtin_amdgcn_readfirstlane(lds_ld(ctl + C_MASK));
      if ((mk >> F_ABORT) & 1u) return;
      if ((mk >> (F_W + wid)) & 1u) break;
      __builtin_amdgcn_s_sleep(4);
    }
    CBAR();
    const int rot = cu & CFG_ROTMASK;
    int qo[8];
#pragma unroll
    for (int qd = 0; qd < 8; ++qd) qo[qd] = ((qd + rot * CFG_ROTMUL) & 7) * 4096;
    const int vb = ((lane >> 3) & 1) * 1024 + (lane >> 4) * 128 + (lane & 7) * 16;
    u4 breg[18][2];
    char* ldsb = smem + L_B + wid * (NLDSK * 2048) + vl;
#pragma unroll
    for (int hf = 0; hf < 2; ++hf) {
      const int so = wbase + cu * 262144 + wid * 65536;
      u4 tmp[7][2];
#pragma unroll
      for (int t = 0; t < 16; ++t) {
#pragma unroll
        for (int m = 0; m < 2; ++m) {
          const u4 v = ld_sys(rws, vb + m * 2048 + (t & 1) * 512, so + 2 * qo[hf * 4 + (t >> 2)] + ((t >> 1) & 1) * 4096);
          if (k_in_lds(t)) tmp[t >> 1][m] = v; else breg[reg_index(hf * 16 + t)][m] = v;
        }
      }
#pragma unroll
      for (int t = 0; t < 7; ++t) {
#pragma unroll
        for (int m = 0; m < 2; ++m) *(u4*)(ldsb + (hf * 7 + t) * 2048 + m * 1024) = tmp[t][m];
      }
    }
    if (wid == 0) STAMP(8);

    u4 ring[16];
    u4 bt[2][2];
#pragma unroll
    for (int n = 0; n < 2; ++n) { bt[n][0] = *(const u4*)(ldsb + n * 2048); bt[n][1] = *(const u4*)(ldsb + n * 2048 + 1024); }
    const int va = vl + wid * 32768;
    int cur = (int)WS_USED;
    for (int n = 0; n < nneed; ++n) {
      if (cur == (int)WS_USED) {
        for (;;) {
          const unsigned c = lds_ld(ctl + (lane == 0 ? C_QOFF + n : C_MASK));
          cur = __builtin_amdgcn_readlane(c, 0);
          if (cur != 0) break;
          if ((__builtin_amdgcn_readlane(c, 1) >> F_ABORT) & 1u) return;
          __builtin_amdgcn_s_sleep(1);
        }
        CBAR();
#pragma unroll
        for (int t = 0; t < 16; ++t) {
          ring[t] = ld_sys(rws, va + (t & 3) * 1024, cur + qo[t >> 2]);
          __builtin_amdgcn_sched_barrier(0);
        }
      }
      f4 acc0 = f4{0.f, 0.f, 0.f, 0.f}, acc1 = f4{0.f, 0.f, 0.f, 0.f};
      int nb = 0;
      unsigned r0 = 0;
#pragma unroll
      for (int t = 0; t < 32; ++t) {
        if (t == 16) { nb = __builtin_amdgcn_readfirstlane(r0); nb = nb != 0 ? nb : (int)WS_USED; CBAR(); }
        const u4 b0 = k_in_lds(t) ? bt[lds_index(t) & 1][0] : breg[reg_index(t)][0];
        const u4 b1 = k_in_lds(t) ? bt[lds_index(t) & 1][1] : breg[reg_index(t)][1];
        const h8 a = __builtin_bit_cast(h8, ring[t & 15]);
        acc0 = __builtin_amdgcn_mfma_f32_16x16x32_f16(a, __builtin_bit_cast(h8, b0), acc0, 0, 0, 0);
        acc1 = __builtin_amdgcn_mfma_f32_16x16x32_f16(a, __builtin_bit_cast(h8, b1), acc1, 0, 0, 0);
        __builtin_amdgcn_sched_barrier(0);
        ring[t & 15] = ld_sys(rws, va + (t & 3) * 1024, (t < 16 ? cur : nb) + qo[((t + 16) & 31) >> 2]);
        if (k_in_lds(t)) {
          const int nn = (lds_index(t) + 2) % NLDSK;
          bt[nn & 1][0] = *(const u4*)(ldsb + nn * 2048);
          bt[nn & 1][1] = *(const u4*)(ldsb + nn * 2048 + 1024);
        }
        if (t == 14) r0 = lds_ld(ctl + C_QOFF + n + 1);
        if (t == 30) r0 = lds_ld(ctl + C_RED);
        __builtin_amdgcn_sched_barrier(0);
      }
      int red = __builtin_amdgcn_readfirstlane(r0);
      while (red < n - 1) {
        __builtin_amdgcn_s_sleep(1);
        const unsigned c = lds_ld(ctl + (lane == 0 ? C_RED : C_MASK));
        if ((__builtin_amdgcn_readlane(c, 1) >> F_ABORT) & 1u) return;
        red = __builtin_amdgcn_readlane(c, 0);
      }
      CBAR();
      char* pp = smem + L_PART + (n & 1) * 8192 + wid * 2048 + vl;
      *(f4*)pp = acc0;
      *(f4*)(pp + 1024) = acc1;
      CBAR();
      if (lane == 0) __hip_atomic_fetch_add(ctl + C_PCNT + (n & 1), 1u, __ATOMIC_RELAXED, __HIP_MEMORY_SCOPE_WORKGROUP);
      CBAR();
      if (wid == 0) STAMP(16 + n);
      cur = nb;
    }
  }
}

}

extern "C" void kernel_launch(int rank, int nranks, ncclComm_t comm,
                              hipStream_t stream,
                              void* const* d_in, const int* in_sizes, int n_in,
                              void* d_out, int out_size,
                              const int* in_dtypes, int out_dtype,
                              void* const* d_out_all,
                              void* const* d_ws_all, size_t ws_bytes) {
  if (nranks != NR || ws_bytes < (size_t)WS_USED || n_in < 2 || (unsigned)rank >= (unsigned)NR) {
    fprintf(stderr, "unsupported configuration\n");
    return;
  }
  Params p;
  p.x = (const float*)d_in[0];
  p.w = (const float*)d_in[1];
  for (int d = 0; d < NR; ++d) {
    p.ws[d] = (char*)d_ws_all[rank ^ d];
    p.out[d] = (float*)d_out_all[rank ^ d];
  }
  p.rank = rank;
  hipLaunchKernelGGL(ag_gemm_relu, dim3(256), dim3(512), 0, stream, p);
  HIP_CHECK(hipGetLastError());
}
```

```cpp
#include <hip/hip_runtime.h>
#include <rccl/rccl.h>
#include <cstdio>

#define HIP_CHECK(expr) do { hipError_t _e = (expr); if (_e != hipSuccess) { \
    fprintf(stderr, "HIP error %s at %s:%d\n", hipGetErrorString(_e), __FILE__, __LINE__); \
    return; } } while (0)

namespace {

typedef float f4 __attribute__((ext_vector_type(4)));
typedef unsigned u4 __attribute__((ext_vector_type(4)));
typedef unsigned u2 __attribute__((ext_vector_type(2)));
typedef _Float16 h8 __attribute__((ext_vector_type(8)));
typedef __amdgpu_buffer_rsrc_t rsrc_t;

#define CFG_ND 23
#define CFG_S1 44
#define CFG_S2 36
#define CFG_S3 20
#define CFG_Q1 0
#define CFG_Q2 0
#define CFG_QM 144
#define CFG_QL 96
#define CFG_EXCL 31
#define CFG_TIER 28
#define CFG_TIER0 2
#define CFG_S0 127
#define CFG_EARLY 0
#define CFG_PAUSE 0
#define CFG_GATEBACK 1
#define CFG_OUTWIN 0
#define CFG_STAGE_DELAY 30
#define CFG_NT 0
#define CFG_POLL_SLEEP 2
#define CFG_ROTMASK 7
#define CFG_ROTMUL 1

constexpr int NR = 8;
constexpr int ND = CFG_ND;
constexpr int NXB = 32 - ND;
constexpr int S0 = CFG_S0;
constexpr int S1 = CFG_S1;
constexpr int S2 = CFG_S2;
constexpr int S3 = CFG_S3;
constexpr int NSTEP = ND * 32;
constexpr int NWST = 256;
constexpr int E2 = NWST + NXB * 32;
constexpr int GATE_STEP = E2 - CFG_GATEBACK;
static_assert(ND >= 20 && ND <= 24, "slot table assumes <= 3 D blocks and <= 2 X blocks per group");
static_assert(S0 <= 127 && S1 <= S0 && S2 <= 40 && S2 <= S1 && S3 <= S2 && S3 <= 24, "sender configuration");
__device__ __forceinline__ void sender_role(int b, int& gen, int& j, bool& active, int& qstep, int& succ) {
  gen = b >= 128 ? 0 : b < 40 ? 1 : b < 80 ? 2 : b < 104 ? 3 : 4;
  j = b - (gen == 0 ? 128 : gen == 1 ? 0 : gen == 2 ? 40 : gen == 3 ? 80 : 104);
  active = j < (gen == 0 ? S0 : gen < 3 ? S2 : S3);
  qstep = gen == 0 ? (j < S1 ? NWST + CFG_Q1 : 0) : gen == 1 ? NWST + CFG_QM : gen == 2 ? E2 + CFG_Q2 : gen == 3 ? E2 + CFG_QL : NSTEP;
  succ = gen == 0 ? (j < S2 ? j : -1) : gen == 1 ? 40 + j : gen == 2 ? (j < S3 ? 80 + j : -1) : gen == 3 ? 104 + j : -1;
}

constexpr unsigned long long DEADLINE = 40000000ull;

constexpr unsigned OFF_PL = 0;
constexpr unsigned OFF_DONE = 40960;
constexpr unsigned OFF_TMO = 42240;
constexpr unsigned OFF_CNT = 45056;
constexpr unsigned OFF_CTR = 65536;
constexpr unsigned OFF_DIAG = 131072;
constexpr unsigned OFF_X = 1u << 20;
constexpr unsigned OFF_W = 65u << 20;
constexpr unsigned WS_USED = 81u << 20;
constexpr int CI_TICK = 0;
constexpr int CI_SX = 2;
constexpr int CI_SW = 34;
constexpr int CI_LX = 50;
constexpr int CI_LW = 82;
constexpr int CI_END = 146;
constexpr int NSLOT = 21;
constexpr int F_W = 21;
constexpr int F_GATE = 25;
constexpr int F_ABORT = 31;

constexpr int NLDSK = 14;
constexpr int NBACK = 15;
constexpr int L_B = 0;
constexpr int L_PART = 4 * NLDSK * 2048;
constexpr int L_BACK = L_PART + 16384;
constexpr int L_CTL = L_BACK + NBACK * 2048;
constexpr int C_QOFF = 0;
constexpr int C_QSLOT = 24;
constexpr int C_PCNT = 48;
constexpr int C_RED = 50;
constexpr int C_FL = 51;
constexpr int C_MASK = 52;
constexpr int C_SDONE = 53;
constexpr int C_WORDS = 64;
constexpr int LDS_BYTES = L_CTL + C_WORDS * 4;
static_assert(LDS_BYTES <= 163840, "LDS");

struct Params {
  const float* x;
  const float* w;
  char* ws[NR];
  float* out[NR];
  int rank;
};

#define WAITV0() do { asm volatile("" ::: "memory"); __builtin_amdgcn_s_waitcnt(0x0F70); asm volatile("" ::: "memory"); } while (0)
#define CBAR() asm volatile("" ::: "memory")

__device__ __forceinline__ rsrc_t mk_rsrc(const void* p, unsigned bytes) {
  return __builtin_amdgcn_make_buffer_rsrc(const_cast<void*>(p), 0, bytes, 0x00020000);
}
__device__ __forceinline__ f4 ld_in(rsrc_t r, int voff, int soff) {
  return __builtin_bit_cast(f4, __builtin_amdgcn_raw_buffer_load_b128(r, voff, soff, CFG_NT));
}
__device__ __forceinline__ u4 ld_sys(rsrc_t r, int voff, int soff) {
  return __builtin_amdgcn_raw_buffer_load_b128(r, voff, soff, 17);
}
__device__ __forceinline__ u4 ld_sys_nt(rsrc_t r, int voff, int soff) {
  return __builtin_amdgcn_raw_buffer_load_b128(r, voff, soff, 17 | CFG_NT);
}
__device__ __forceinline__ unsigned ld_sys32(rsrc_t r, int voff, int soff) {
  return __builtin_amdgcn_raw_buffer_load_b32(r, voff, soff, 17);
}
__device__ __forceinline__ void st_sys(u4 v, rsrc_t r, int voff, int soff) {
  __builtin_amdgcn_raw_buffer_store_b128(v, r, voff, soff, 17);
}
__device__ __forceinline__ void st_sys32(unsigned v, rsrc_t r, int voff, int soff) {
  __builtin_amdgcn_raw_buffer_store_b32(v, r, voff, soff, 17);
}
__device__ __forceinline__ u4 cvt8(f4 a, f4 b) {
  h8 h = h8{(_Float16)a.x, (_Float16)a.y, (_Float16)a.z, (_Float16)a.w,
            (_Float16)b.x, (_Float16)b.y, (_Float16)b.z, (_Float16)b.w};
  return __builtin_bit_cast(u4, h);
}
template <int I>
__device__ __forceinline__ u4 cvtcol(const f4* r) {
  h8 h = h8{(_Float16)r[0][I], (_Float16)r[1][I], (_Float16)r[2][I], (_Float16)r[3][I],
            (_Float16)r[4][I], (_Float16)r[5][I], (_Float16)r[6][I], (_Float16)r[7][I]};
  return __builtin_bit_cast(u4, h);
}
__device__ __forceinline__ u4 cvtcol_i(const f4* r, int i) {
  return i == 0 ? cvtcol<0>(r) : i == 1 ? cvtcol<1>(r) : i == 2 ? cvtcol<2>(r) : cvtcol<3>(r);
}
__device__ __forceinline__ void load_xpiece(f4* r, rsrc_t rx, int vx, int pi) {
  const int so = (pi >> 5) * (16 * 16384) + (pi & 31) * 512;
#pragma unroll
  for (int t = 0; t < 8; ++t) r[t] = ld_in(rx, vx + t * 64, so);
}
__device__ __forceinline__ void load_wgran(f4* r, rsrc_t rw, int vw, int cg, int kg) {
  const int so = kg * (64 * 2048) + cg * 128;
#pragma unroll
  for (int j = 0; j < 8; ++j) r[j] = ld_in(rw, vw + (j & 1) * 2048, so + (16 * (j >> 2) + (j & 2)) * 2048);
}
__device__ __forceinline__ unsigned lds_ld(const unsigned* p) {
  return __hip_atomic_load(p, __ATOMIC_RELAXED, __HIP_MEMORY_SCOPE_WORKGROUP);
}
__device__ __forceinline__ void lds_st(unsigned* p, unsigned v) {
  __hip_atomic_store(p, v, __ATOMIC_RELAXED, __HIP_MEMORY_SCOPE_WORKGROUP);
}

__device__ __forceinline__ int src_off(int d) { return d == 0 ? 5 : d < 4 ? 3 * (d - 1) : 2 * (d - 4) + 1; }
__device__ __forceinline__ int src_sbase(int d) { return d == 0 ? 0 : d < 4 ? 4 + 3 * (d - 1) : 13 + 2 * (d - 4); }
__device__ __forceinline__ int src_lo(int d) { return d < 4 ? 0 : ND; }
__device__ __forceinline__ int flag_off(int d, int b, int cu) {
  return ((cu * 8 + ((b + src_off(d)) & 7)) * 32 + src_sbase(d) + ((b - src_lo(d)) >> 3)) * 4;
}
__device__ __forceinline__ bool slot_block(int slot, int gx, bool helper, int q, int& d, int& b) {
  d = slot < 4 ? 0 : slot < 13 ? 1 + (slot - 4) / 3 : (4 + (slot - 13) / 2) & 7;
  const int lo = src_lo(d);
  const int hi = d == 0 ? (helper ? ND : 32) : d < 4 ? ND : ((helper && d == 4 + q) ? ND : 32);
  b = lo + ((gx - src_off(d) - lo) & 7) + 8 * (slot - src_sbase(d));
  return slot < NSLOT && b < hi;
}

constexpr bool k_in_lds(int t) { return ((t & 15) & 1) && (t & 15) != 15; }
constexpr int lds_index(int t) { return (t >> 4) * 7 + ((t & 15) >> 1); }
constexpr int reg_index(int t) { return (t >> 4) * 9 + ((t & 15) == 15 ? 8 : ((t & 15) >> 1)); }

#define STAMP(idx) do {} while (0)

__global__ void __launch_bounds__(512, 2) ag_gemm_relu(Params p) {
  __shared__ __attribute__((aligned(16))) char smem[LDS_BYTES];
  unsigned* ctl = (unsigned*)(smem + L_CTL);
  const int lane = threadIdx.x & 63;
  const int wid = __builtin_amdgcn_readfirstlane(threadIdx.x >> 6);
  const int gx = blockIdx.x & 7;
  const int cu = blockIdx.x >> 3;
  const bool helper = cu >= 16;
  const int q = (cu - 16) >> 2;
  const rsrc_t rws = mk_rsrc(p.ws[0], WS_USED);
  const rsrc_t rx = mk_rsrc(p.x, 512u * 4096u * 4u);
  const rsrc_t rw = mk_rsrc(p.w, 4096u * 512u * 4u);
  const int vx = (lane & 15) * 16384 + (lane >> 4) * 16;
  const int vw = (32 * (lane >> 5) + 4 * ((lane >> 3) & 3)) * 2048 + (lane & 7) * 16;
  const int vl = lane * 16;

  const unsigned cprev = ld_sys32(rws, 0, OFF_CTR + blockIdx.x * 64);

  const rsrc_t rcnt = mk_rsrc(p.ws[0] + OFF_CNT, CI_END * 64);
  int gen, j, qstep, succwg;
  bool active;
  sender_role(blockIdx.x, gen, j, active, qstep, succwg);

  if (wid == 6) {
    const int succ = lane == 0 && succwg >= 0 ? succwg * 128 : (int)WS_USED;
    f4 rd[8], ra[8], rb[24];
    if (active && gen == 0) {
      load_xpiece(rd, rx, vx, j);
      load_wgran(ra, rw, vw, j & 3, j >> 2);
#pragma unroll
      for (int t = 0; t < 3; ++t) load_wgran(rb + 8 * t, rw, vw, 4 * (t + 1) + (j & 3), j >> 2);
    }
    const unsigned call = __builtin_amdgcn_readfirstlane(cprev) + 1u;
    asm volatile("" :: "s"(call) : "memory");
    __builtin_amdgcn_s_barrier();
    if (!active) {
      if (blockIdx.x == 255) {
        const unsigned long long t0 = __builtin_amdgcn_s_memrealtime();
        for (;;) {
          const unsigned a0 = ld_sys32(rws, lane * 4, OFF_DONE);
          const unsigned a1 = ld_sys32(rws, lane * 4, OFF_DONE + 256);
          const unsigned tm = ld_sys32(rws, 0, OFF_TMO);
          if (__ballot((int)(a0 - call) < 0 || (int)(a1 - call) < 0) == 0ull) break;
          if (__builtin_amdgcn_readfirstlane(tm) == call || __builtin_amdgcn_s_memrealtime() - t0 > DEADLINE) {
            st_sys32(call, rws, lane == 0 ? 0 : (int)WS_USED, OFF_TMO);
            const rsrc_t ro = mk_rsrc(p.out[0], 4096u * 512u * 4u);
            const unsigned long long bad = __ballot((int)(a0 - call) < 0);
            const int f = bad ? __builtin_ctzll(bad) : 64 + __builtin_ctzll(__ballot((int)(a1 - call) < 0));
            const int row = ((p.rank ^ (4 + (f >> 5))) & 7) * 512 + (((f & 7) - src_off(0)) & 7) * 16;
            st_sys32(0x7fc00000u, ro, lane == 0 ? 0 : 0x40000000, row * 2048 + ((f >> 5) * 4 + ((f >> 3) & 3)) * 128);
            st_sys32(0x7fc00000u, ro, lane == 0 ? 0 : 0x40000000, p.rank * 512 * 2048);
            break;
          }
          __builtin_amdgcn_s_sleep(2);
        }
        STAMP(7);
      }
      return;
    }
    CBAR();
    if (gen != 0) {
      for (;;) {
        const unsigned mk = __builtin_amdgcn_readfirstlane(lds_ld(ctl + C_MASK));
        if ((mk >> F_ABORT) & 1u) return;
        if ((mk >> F_GATE) & 1u) break;
        __builtin_amdgcn_s_sleep(4);
      }
    }
    __builtin_amdgcn_s_setprio(2);
    const unsigned par = call & 1u;
    rsrc_t rp[NR];
#pragma unroll
    for (int d = 1; d < NR; ++d) rp[d] = mk_rsrc(p.ws[d], WS_USED);
    if (blockIdx.x == 128) st_sys32(0u, rcnt, lane == 0 ? 0 : CI_END * 64, (CI_TICK + (par ^ 1u)) * 64);
    const int xbase = OFF_X + par * (32u << 20);
    const int wbase = OFF_W + par * (8u << 20);
    const int lcu = lane & 31;
    const int voob = lane < 32 ? 0 : (int)WS_USED;
    int i = gen == 0 ? j : -1, prev = -1;
    if (gen == 0 && j >= S1)
      for (int z = (j - S1) >> 3; z >= -CFG_TIER0; --z) __builtin_amdgcn_s_sleep(CFG_TIER);
    WAITV0();
    for (;;) {
      const bool quit = i >= qstep;
      int cv = CI_END;
      if (lane == 0 && !quit) cv = CI_TICK + par;
      if (lane == 1 && prev >= 0) cv = CI_SX + (prev >> 5);
      if (lane == 2 && prev >= 0 && prev < E2) cv = prev < NWST ? CI_SW + (prev >> 6) * 4 + (prev & 3) : CI_SX + ND + ((prev - NWST) >> 5);
      const int tv = __builtin_amdgcn_raw_ptr_buffer_atomic_add_i32(1, rcnt, cv * 64, 0, 0);
      int t0v, t1v, t2v;
      if (i == GATE_STEP) {
        st_sys32(call, rws, (128 + lane) * 128, OFF_PL + F_GATE * 4);
        st_sys32(call, rws, (192 + lane) * 128, OFF_PL + F_GATE * 4);
      }
      if (!CFG_EARLY && quit && prev < qstep) st_sys32(call, rws, succ, OFF_PL + F_GATE * 4);
      if (i < 0 || i >= NSTEP) {
        t0v = __builtin_amdgcn_readlane(tv, 0); t1v = __builtin_amdgcn_readlane(tv, 1); t2v = __builtin_amdgcn_readlane(tv, 2);
      } else if (i < NWST) {
        const int wo = wbase + (16 + (i & 3)) * 262144 + (i >> 2) * 4096;
#pragma unroll
        for (int t = 0; t < 4; ++t) {
          const u4 h = cvt8(rd[2 * t], rd[2 * t + 1]);
#pragma unroll
          for (int d = 1; d < 4; ++d) st_sys(h, rp[d], vl + t * 1024, xbase + d * (4 << 20) + i * 4096);
          st_sys(cvtcol_i(ra, t), rp[4], vl + t * 1024, wo);
#pragma unroll
          for (int d = 5; d < 8; ++d) st_sys(cvtcol_i(rb + 8 * (d - 5), t), rp[d], vl + t * 1024, wo + (d - 4) * (4 * 262144));
        }
        t0v = __builtin_amdgcn_readlane(tv, 0); t1v = __builtin_amdgcn_readlane(tv, 1); t2v = __builtin_amdgcn_readlane(tv, 2);
      } else if (i < E2) {
        const int xi = NSTEP + i - NWST;
#pragma unroll
        for (int t = 0; t < 4; ++t) {
          const u4 h = cvt8(rd[2 * t], rd[2 * t + 1]);
          const u4 g = cvt8(ra[2 * t], ra[2 * t + 1]);
#pragma unroll
          for (int d = 1; d < 4; ++d) st_sys(h, rp[d], vl + t * 1024, xbase + d * (4 << 20) + i * 4096);
#pragma unroll
          for (int d = 4; d < 8; ++d) st_sys(g, rp[d], vl + t * 1024, xbase + d * (4 << 20) + xi * 4096);
        }
        t0v = __builtin_amdgcn_readlane(tv, 0); t1v = __builtin_amdgcn_readlane(tv, 1); t2v = __builtin_amdgcn_readlane(tv, 2);
      } else {
#pragma unroll
        for (int t = 0; t < 4; ++t) {
          const u4 h = cvt8(rd[2 * t], rd[2 * t + 1]);
#pragma unroll
          for (int d = 1; d < 4; ++d) st_sys(h, rp[d], vl + t * 1024, xbase + d * (4 << 20) + i * 4096);
        }
        t0v = __builtin_amdgcn_readlane(tv, 0); t1v = __builtin_amdgcn_readlane(tv, 1); t2v = __builtin_amdgcn_readlane(tv, 2);
      }
      if (prev >= 0) {
        if ((t1v & 31) == 31) {
#pragma unroll
          for (int d = 1; d < 4; ++d) st_sys32(call, rp[d], voob + lcu * 1024, OFF_PL + flag_off(d, prev >> 5, 0));
        }
        if (prev < NWST) {
          if ((t2v & 15) == 15) {
#pragma unroll
            for (int d = 4; d < 8; ++d)
              st_sys32(call, rp[d], lane < 8 ? lane * 128 : (int)WS_USED, OFF_PL + (16 + 4 * (d - 4) + (prev & 3)) * 1024 + (F_W + (prev >> 6)) * 4);
          }
        } else if (prev < E2) {
          if ((t2v & 31) == 31) {
#pragma unroll
            for (int d = 4; d < 8; ++d) st_sys32(call, rp[d], voob + lcu * 1024, OFF_PL + flag_off(d, ND + ((prev - NWST) >> 5), 0));
          }
        }
      }
      if (i >= NSTEP) break;
      int nxt = t0v + S0;
      if (quit || (unsigned)nxt >= (unsigned)NSTEP) nxt = NSTEP;
      if (CFG_EARLY && !quit && nxt >= qstep) st_sys32(call, rws, succ, OFF_PL + F_GATE * 4);
      if (nxt < NSTEP) {
        load_xpiece(rd, rx, vx, nxt);
        if (nxt < NWST) {
          load_wgran(ra, rw, vw, nxt & 3, nxt >> 2);
#pragma unroll
          for (int t = 0; t < 3; ++t) load_wgran(rb + 8 * t, rw, vw, 4 * (t + 1) + (nxt & 3), nxt >> 2);
        } else if (nxt < E2) {
          load_xpiece(ra, rx, vx, NSTEP + nxt - NWST);
        }
      }
      WAITV0();
      prev = i;
      i = nxt;
    }
    if (lane == 0) lds_st(ctl + C_SDONE, 1u);
    STAMP(1);
    return;
  }

  int sd, sb;
  const bool sused = slot_block(lane, gx, helper, q, sd, sb);
  const unsigned needmask = (unsigned)__ballot(sused);

  const int nneed = __builtin_popcount(needmask);

  if (wid >= 4) {
    if (wid == 4) lds_st(ctl + lane, 0u);
    const unsigned call = __builtin_amdgcn_readfirstlane(cprev) + 1u;
    asm volatile("s_waitcnt lgkmcnt(0)" :: "s"(call) : "memory");
    __builtin_amdgcn_s_barrier();
    CBAR();
    __builtin_amdgcn_s_setprio(2);

    if (wid == 4) {
      const unsigned want = needmask | (15u << F_W) | (helper || active ? 1u << F_GATE : 0u);
      bool hold = active && ((CFG_EXCL >> gen) & 1);
      const int pv = lane < 32 ? (int)(OFF_PL + blockIdx.x * 128 + lane * 4) : lane == 32 ? (int)OFF_TMO : (int)WS_USED;
      const unsigned boff = OFF_X + (call & 1u) * (32u << 20) + sd * (4 << 20) + sb * 131072;
      unsigned seen = 0;
      const unsigned long long t0 = __builtin_amdgcn_s_memrealtime();
      for (;;) {
        const unsigned fv = ld_sys32(rws, pv, 0);
        unsigned m = (unsigned)__ballot((int)(fv - call) >= 0) & want;
        if (hold && (gen == 0 || ((m >> F_GATE) & 1u))) {
          if (__builtin_amdgcn_readfirstlane(lds_ld(ctl + C_SDONE)) != 0u) hold = false;
          else m = seen | (m & (1u << F_GATE));
        }
        if (m != seen) {
          const unsigned fresh = m & ~seen & needmask;
          if (lane < 32 && ((fresh >> lane) & 1u)) {
            const int pos = __builtin_popcount(seen & needmask) + __builtin_popcount(fresh >> lane) - 1;
            lds_st(ctl + C_QSLOT + pos, (unsigned)lane);
            lds_st(ctl + C_QOFF + pos, boff);
          }
          if (lane == 0) lds_st(ctl + C_MASK, m);
          seen = m;
          if (m == want) break;
        }
        if (__builtin_amdgcn_readlane(fv, 32) == call || __builtin_amdgcn_s_memrealtime() - t0 > DEADLINE) {
          st_sys32(call, rws, lane == 0 ? 0 : (int)WS_USED, OFF_TMO);
          if (lane == 0) lds_st(ctl + C_MASK, seen | (1u << F_ABORT));
          break;
        }
        __builtin_amdgcn_s_sleep(CFG_POLL_SLEEP);
      }
      STAMP(4);
    } else if (wid == 5) {
      unsigned pending = needmask;
      const int rowoff = (((p.rank ^ sd) & 7) * 512 + sb * 16) * 2048;
      float* outp = p.out[0];
      if (helper) outp = q == 0 ? p.out[4] : q == 1 ? p.out[5] : q == 2 ? p.out[6] : p.out[7];
      const rsrc_t ro = mk_rsrc(outp, 4096u * 512u * 4u);
      const int vo = (lane >> 3) * 2048 + (lane & 7) * 16 + (cu & 15) * 128;
      int bi = 0;
      for (int n = 0; n < nneed; ++n) {
        unsigned c;
        bool abort;
        if (helper) {
          if (CFG_OUTWIN == 0) asm volatile("s_waitcnt vmcnt(0)" ::: "memory");
          else if (CFG_OUTWIN == 1) asm volatile("s_waitcnt vmcnt(2)" ::: "memory");
          else asm volatile("s_waitcnt vmcnt(4)" ::: "memory");
        }
        for (;;) {
          c = lds_ld(ctl + lane);
          const unsigned mk = __builtin_amdgcn_readlane(c, C_MASK);
          abort = (mk >> F_ABORT) & 1u;
          const int red = __builtin_amdgcn_readlane(c, C_RED);
          const bool idle = !CFG_PAUSE || red == __builtin_popcount(mk & needmask) || red - n >= NBACK - 1;
          if (abort || (red > n && (!helper || (((mk >> F_GATE) & 1u) && idle)))) break;
          __builtin_amdgcn_s_sleep(1);
        }
        if (abort) break;
        CBAR();
        const int sl = __builtin_amdgcn_readlane(c, C_QSLOT + n);
        const int ro_s = __builtin_amdgcn_readlane(rowoff, sl);
        pending &= ~(1u << sl);
        const u4 o0 = *(const u4*)(smem + L_BACK + bi * 2048 + vl);
        const u4 o1 = *(const u4*)(smem + L_BACK + bi * 2048 + 1024 + vl);
        st_sys(o0, ro, vo, ro_s);
        st_sys(o1, ro, vo, ro_s + 16384);
        CBAR();
        if (lane == 0) lds_st(ctl + C_FL, (unsigned)(n + 1));
        bi = bi + 1 == NBACK ? 0 : bi + 1;
      }
      while (pending != 0u) {
        const int sl = __builtin_ctz(pending);
        pending &= pending - 1u;
        const int ro_s = __builtin_amdgcn_readlane(rowoff, sl);
        const u4 o = u4{0x7fc00000u, 0x7fc00000u, 0x7fc00000u, 0x7fc00000u};
        st_sys(o, ro, vo, ro_s);
        st_sys(o, ro, vo, ro_s + 16384);
      }
      if (helper) {
        WAITV0();
        char* wsq = q == 0 ? p.ws[4] : q == 1 ? p.ws[5] : q == 2 ? p.ws[6] : p.ws[7];
        if (lane == 0)
          __hip_atomic_store((unsigned*)(wsq + OFF_DONE) + q * 32 + (cu & 3) * 8 + gx, call, __ATOMIC_RELAXED, __HIP_MEMORY_SCOPE_SYSTEM);
      }
      STAMP(5);
    } else {
      st_sys32(call, rws, lane == 0 ? 0 : (int)WS_USED, OFF_CTR + blockIdx.x * 64);
      STAMP(6);
      const int wo = (lane >> 4) * 512 + (lane & 7) * 16 + ((lane >> 3) & 1) * 4;
      int bi = 0;
      for (int n = 0; n < nneed; ++n) {
        bool abort;
        for (;;) {
          const unsigned c = lds_ld(ctl + lane);
          abort = (__builtin_amdgcn_readlane(c, C_MASK) >> F_ABORT) & 1u;
          const int pc = (n & 1) ? __builtin_amdgcn_readlane(c, C_PCNT + 1) : __builtin_amdgcn_readlane(c, C_PCNT);
          if (abort || (pc >= 4 * ((n >> 1) + 1) && n - (int)__builtin_amdgcn_readlane(c, C_FL) < NBACK)) break;
          __builtin_amdgcn_s_sleep(1);
        }
        if (abort) break;
        CBAR();
        const char* pp = smem + L_PART + (n & 1) * 8192 + vl;
        float* bo = (float*)(smem + L_BACK + bi * 2048 + wo);
#pragma unroll
        for (int m = 0; m < 2; ++m) {
          const f4 v = (*(const f4*)(pp + m * 1024) + *(const f4*)(pp + 2048 + m * 1024)) +
                       (*(const f4*)(pp + 4096 + m * 1024) + *(const f4*)(pp + 6144 + m * 1024));
#pragma unroll
          for (int jj = 0; jj < 4; ++jj) bo[jj * 32 + m * 2] = v[jj] < 0.f ? 0.f : v[jj];
        }
        CBAR();
        if (lane == 0) lds_st(ctl + C_RED, (unsigned)(n + 1));
        bi = bi + 1 == NBACK ? 0 : bi + 1;
      }
    }
    return;
  }

  {
    const int gi = blockIdx.x * 4 + wid;
    const unsigned call = __builtin_amdgcn_readfirstlane(cprev) + 1u;
    asm volatile("" :: "s"(call) : "memory");
    __builtin_amdgcn_s_barrier();
    CBAR();
    if (CFG_STAGE_DELAY > 0) __builtin_amdgcn_s_sleep(CFG_STAGE_DELAY);
    f4 sx[8], sw[8];
    load_xpiece(sx, rx, vx, gi);
    load_wgran(sw, rw, vw, gi >> 6, gi & 63);
    const unsigned par = call & 1u;
    const int xbase = OFF_X + par * (32u << 20);
    const int wbase = OFF_W + par * (8u << 20);
#pragma unroll
    for (int t = 0; t < 4; ++t) st_sys(cvtcol_i(sw, t), rws, vl + t * 1024, wbase + gi * 4096);
#pragma unroll
    for (int t = 0; t < 4; ++t) st_sys(cvt8(sx[2 * t], sx[2 * t + 1]), rws, vl + t * 1024, xbase + gi * 4096);
    WAITV0();
    {
      const int tv = __builtin_amdgcn_raw_ptr_buffer_atomic_add_i32(1, rcnt, (lane == 0 ? CI_LX + (gi >> 5) : lane == 1 ? CI_LW + (gi >> 4) : CI_END) * 64, 0, 0);
      if ((__builtin_amdgcn_readlane(tv, 0) & 31) == 31)
        st_sys32(call, rws, lane < 32 ? lane * 1024 : (int)WS_USED, OFF_PL + flag_off(0, gi >> 5, 0));
      if ((__builtin_amdgcn_readlane(tv, 1) & 15) == 15)
        st_sys32(call, rws, lane < 8 ? lane * 128 : (int)WS_USED, OFF_PL + (gi >> 6) * 1024 + (F_W + ((gi >> 4) & 3)) * 4);
    }

    for (;;) {
      const unsigned mk = __builtin_amdgcn_readfirstlane(lds_ld(ctl + C_MASK));
      if ((mk >> F_ABORT) & 1u) return;
      if ((mk >> (F_W + wid)) & 1u) break;
      __builtin_amdgcn_s_sleep(4);
    }
    CBAR();
    const int rot = cu & CFG_ROTMASK;
    int qo[8];
#pragma unroll
    for (int qd = 0; qd < 8; ++qd) qo[qd] = ((qd + rot * CFG_ROTMUL) & 7) * 4096;
    const int vb = ((lane >> 3) & 1) * 1024 + (lane >> 4) * 128 + (lane & 7) * 16;
    u4 breg[18][2];
    char* ldsb = smem + L_B + wid * (NLDSK * 2048) + vl;
#pragma unroll
    for (int hf = 0; hf < 2; ++hf) {
      const int so = wbase + cu * 262144 + wid * 65536;
      u4 tmp[7][2];
#pragma unroll
      for (int t = 0; t < 16; ++t) {
#pragma unroll
        for (int m = 0; m < 2; ++m) {
          const u4 v = ld_sys(rws, vb + m * 2048 + (t & 1) * 512, so + 2 * qo[hf * 4 + (t >> 2)] + ((t >> 1) & 1) * 4096);
          if (k_in_lds(t)) tmp[t >> 1][m] = v; else breg[reg_index(hf * 16 + t)][m] = v;
        }
      }
#pragma unroll
      for (int t = 0; t < 7; ++t) {
#pragma unroll
        for (int m = 0; m < 2; ++m) *(u4*)(ldsb + (hf * 7 + t) * 2048 + m * 1024) = tmp[t][m];
      }
    }
    if (wid == 0) STAMP(8);

    u4 ring[16];
    u4 bt[2][2];
#pragma unroll
    for (int n = 0; n < 2; ++n) { bt[n][0] = *(const u4*)(ldsb + n * 2048); bt[n][1] = *(const u4*)(ldsb + n * 2048 + 1024); }
    int va = vl + wid * 32768;
    int cur = (int)WS_USED;
    for (int n = 0; n < nneed; ++n) {
      if (cur == (int)WS_USED) {
        for (;;) {
          const unsigned c = lds_ld(ctl + (lane == 0 ? C_QOFF + n : C_MASK));
          cur = __builtin_amdgcn_readlane(c, 0);
          if (cur != 0) break;
          if ((__builtin_amdgcn_readlane(c, 1) >> F_ABORT) & 1u) return;
          __builtin_amdgcn_s_sleep(1);
        }
        asm volatile("" : "+v"(va) :: "memory");
#pragma unroll
        for (int t = 0; t < 16; ++t) {
          ring[t] = ld_sys(rws, va + (t & 3) * 1024, cur + qo[t >> 2]);
          __builtin_amdgcn_sched_barrier(0);
        }
      }
      f4 acc0 = f4{0.f, 0.f, 0.f, 0.f}, acc1 = f4{0.f, 0.f, 0.f, 0.f};
      int nb = 0;
      unsigned r0 = 0;
      asm volatile("" : "+v"(va));
#pragma unroll
      for (int t = 0; t < 32; ++t) {
        if (t == 16) { nb = __builtin_amdgcn_readfirstlane(r0); nb = nb != 0 ? nb : (int)WS_USED; CBAR(); }
        const u4 b0 = k_in_lds(t) ? bt[lds_index(t) & 1][0] : breg[reg_index(t)][0];
        const u4 b1 = k_in_lds(t) ? bt[lds_index(t) & 1][1] : breg[reg_index(t)][1];
        const h8 a = __builtin_bit_cast(h8, ring[t & 15]);
        acc0 = __builtin_amdgcn_mfma_f32_16x16x32_f16(a, __builtin_bit_cast(h8, b0), acc0, 0, 0, 0);
        acc1 = __builtin_amdgcn_mfma_f32_16x16x32_f16(a, __builtin_bit_cast(h8, b1), acc1, 0, 0, 0);
        __builtin_amdgcn_sched_barrier(0);
        ring[t & 15] = ld_sys(rws, va + (t & 3) * 1024, (t < 16 ? cur : nb) + qo[((t + 16) & 31) >> 2]);
        if (k_in_lds(t)) {
          const int nn = (lds_index(t) + 2) % NLDSK;
          bt[nn & 1][0] = *(const u4*)(ldsb + nn * 2048);
          bt[nn & 1][1] = *(const u4*)(ldsb + nn * 2048 + 1024);
        }
        if (t == 14) r0 = lds_ld(ctl + C_QOFF + n + 1);
        if (t == 30) r0 = lds_ld(ctl + C_RED);
        __builtin_amdgcn_sched_barrier(0);
      }
      int red = __builtin_amdgcn_readfirstlane(r0);
      while (red < n - 1) {
        __builtin_amdgcn_s_sleep(1);
        const unsigned c = lds_ld(ctl + (lane == 0 ? C_RED : C_MASK));
        if ((__builtin_amdgcn_readlane(c, 1) >> F_ABORT) & 1u) return;
        red = __builtin_amdgcn_readlane(c, 0);
      }
      CBAR();
      char* pp = smem + L_PART + (n & 1) * 8192 + wid * 2048 + vl;
      *(f4*)pp = acc0;
      *(f4*)(pp + 1024) = acc1;
      CBAR();
      if (lane == 0) __hip_atomic_fetch_add(ctl + C_PCNT + (n & 1), 1u, __ATOMIC_RELAXED, __HIP_MEMORY_SCOPE_WORKGROUP);
      CBAR();
      if (wid == 0) STAMP(16 + n);
      cur = nb;
    }
  }
}

}

extern "C" void kernel_launch(int rank, int nranks, ncclComm_t comm,
                              hipStream_t stream,
                              void* const* d_in, const int* in_sizes, int n_in,
                              void* d_out, int out_size,
                              const int* in_dtypes, int out_dtype,
                              void* const* d_out_all,
                              void* const* d_ws_all, size_t ws_bytes) {
  if (nranks != NR || ws_bytes < (size_t)WS_USED || n_in < 2 || (unsigned)rank >= (unsigned)NR) {
    fprintf(stderr, "unsupported configuration\n");
    return;
  }
  Params p;
  p.x = (const float*)d_in[0];
  p.w = (const float*)d_in[1];
  for (int d = 0; d < NR; ++d) {
    p.ws[d] = (char*)d_ws_all[rank ^ d];
    p.out[d] = (float*)d_out_all[rank ^ d];
  }
  p.rank = rank;
  hipLaunchKernelGGL(ag_gemm_relu, dim3(256), dim3(512), 0, stream, p);
  HIP_CHECK(hipGetLastError());
}
```

```cpp
#include <hip/hip_runtime.h>
#include <rccl/rccl.h>
#include <cstdio>

#define HIP_CHECK(expr) do { hipError_t _e = (expr); if (_e != hipSuccess) { \
    fprintf(stderr, "HIP error %s at %s:%d\n", hipGetErrorString(_e), __FILE__, __LINE__); \
    return; } } while (0)

namespace {

typedef float f4 __attribute__((ext_vector_type(4)));
typedef unsigned u4 __attribute__((ext_vector_type(4)));
typedef _Float16 h8 __attribute__((ext_vector_type(8)));
typedef unsigned long long u64;
typedef __amdgpu_buffer_rsrc_t rsrc_t;

#define CFG_ND 23
#define CFG_S0 128
#define CFG_S1 44
#define CFG_S2 34
#define CFG_S3 20
#define CFG_G1 240
#define CFG_G2 400
#define CFG_G3 544
#define CFG_G4 640
#define CFG_GATEBACK 12
#define CFG_STAGE_DELAY 45
#define CFG_TIER 26
#define CFG_TIER0 1

constexpr int NR = 8;
constexpr int ND = CFG_ND;
constexpr int NXB = 32 - ND;
constexpr int S0 = CFG_S0;
constexpr int S1 = CFG_S1;
constexpr int S2 = CFG_S2;
constexpr int S3 = CFG_S3;
constexpr int NSTEP = ND * 32;
constexpr int NWST = 256;
constexpr int E2 = NWST + NXB * 32;
constexpr int GATE_STEP = E2 - CFG_GATEBACK;
static_assert(ND >= 20 && ND <= 24, "slot table assumes <= 3 D blocks and <= 2 X blocks per group");
static_assert(S0 <= 128 && S1 <= S0 && S2 <= 40 && S2 <= S1 && S3 <= S2 && S3 <= 24, "sender configuration");
static_assert(CFG_G1 <= NSTEP && CFG_G2 <= NSTEP && CFG_G3 <= NSTEP && CFG_G4 <= NSTEP && GATE_STEP < NSTEP, "hand-over limits");

__device__ __forceinline__ void sender_role(int b, int& gen, int& j, bool& active, int& qstep, int& succ) {
  gen = b >= 128 ? 0 : 1 + (b >= 40) + (b >= 80) + (b >= 104);
  constexpr u64 base = 128ull | (0ull << 10) | (40ull << 20) | (80ull << 30) | (104ull << 40) | (0ull << 50);
  constexpr u64 count = (u64)S0 | ((u64)S2 << 10) | ((u64)S2 << 20) | ((u64)S3 << 30) | ((u64)(CFG_G4 < NSTEP ? S3 : 0) << 40);
  constexpr u64 limit = (u64)CFG_G1 | ((u64)CFG_G2 << 10) | ((u64)CFG_G3 << 20) | ((u64)CFG_G4 << 30) | ((u64)NSTEP << 40);
  j = b - (int)((base >> (10 * gen)) & 1023);
  active = j < (int)((count >> (10 * gen)) & 1023);
  qstep = gen == 0 && j >= S1 ? 0 : (int)((limit >> (10 * gen)) & 1023);
  succ = j < (int)((count >> (10 * gen + 10)) & 1023) ? (int)((base >> (10 * gen + 10)) & 1023) + j : -1;
}

constexpr u64 DEADLINE = 100000000ull;

constexpr unsigned OFF_PL = 0;
constexpr unsigned OFF_TMO = 40960;
constexpr unsigned OFF_CNT = 45056;
constexpr unsigned OFF_CTR = 65536;
constexpr unsigned OFF_SL = 98304;
constexpr unsigned OFF_DIAG = 131072;
constexpr unsigned OFF_X = 1u << 20;
constexpr unsigned OFF_W = 65u << 20;
constexpr unsigned WS_USED = 81u << 20;
constexpr int CI_TICK = 0;
constexpr int CI_SX = 4;
constexpr int CI_SW = 36;
constexpr int CI_LX = 52;
constexpr int CI_LW = 84;
constexpr int CI_END = 148;
constexpr int NSLOT = 21;
constexpr int F_W = 21;
constexpr int F_GATE = 25;
constexpr int F_DONE = 26;
constexpr int F_ABORT = 31;

constexpr int NLDSK = 14;
constexpr int NBACK = 15;
constexpr int L_B = 0;
constexpr int L_PART = 4 * NLDSK * 2048;
constexpr int L_BACK = L_PART + 16384;
constexpr int L_CTL = L_BACK + NBACK * 2048;
constexpr int C_QOFF = 0;
constexpr int C_QSLOT = 24;
constexpr int C_PCNT = 48;
constexpr int C_RED = 50;
constexpr int C_FL = 51;
constexpr int C_MASK = 52;
constexpr int C_SDONE = 53;
constexpr int C_WORDS = 64;
constexpr int LDS_BYTES = L_CTL + C_WORDS * 4;
static_assert(LDS_BYTES <= 163840, "LDS");

struct Params {
  const float* x;
  const float* w;
  char* ws[NR];
  float* out[NR];
  int rank;
};

#define WAITV0() do { asm volatile("" ::: "memory"); __builtin_amdgcn_s_waitcnt(0x0F70); asm volatile("" ::: "memory"); } while (0)
#define CBAR() asm volatile("" ::: "memory")
constexpr int OOB = 0x40000000;

__device__ __forceinline__ rsrc_t mk_rsrc(const void* p, unsigned bytes) {
  return __builtin_amdgcn_make_buffer_rsrc(const_cast<void*>(p), 0, bytes, 0x00020000);
}
__device__ __forceinline__ f4 ld_in(rsrc_t r, int voff, int soff) {
  return __builtin_bit_cast(f4, __builtin_amdgcn_raw_buffer_load_b128(r, voff, soff, 0));
}
__device__ __forceinline__ u4 ld_sys(rsrc_t r, int voff, int soff) {
  return __builtin_amdgcn_raw_buffer_load_b128(r, voff, soff, 17);
}
__device__ __forceinline__ unsigned ld_sys32(rsrc_t r, int voff, int soff) {
  return __builtin_amdgcn_raw_buffer_load_b32(r, voff, soff, 17);
}
__device__ __forceinline__ void st_sys(u4 v, rsrc_t r, int voff, int soff) {
  __builtin_amdgcn_raw_buffer_store_b128(v, r, voff, soff, 17);
}
__device__ __forceinline__ void st_sys32(unsigned v, rsrc_t r, int voff, int soff) {
  __builtin_amdgcn_raw_buffer_store_b32(v, r, voff, soff, 17);
}
__device__ __forceinline__ u4 cvt8(f4 a, f4 b) {
  h8 h = h8{(_Float16)a.x, (_Float16)a.y, (_Float16)a.z, (_Float16)a.w,
            (_Float16)b.x, (_Float16)b.y, (_Float16)b.z, (_Float16)b.w};
  return __builtin_bit_cast(u4, h);
}
__device__ __forceinline__ u4 cvtcol(const f4* r, int i) {
  h8 h = h8{(_Float16)r[0][i], (_Float16)r[1][i], (_Float16)r[2][i], (_Float16)r[3][i],
            (_Float16)r[4][i], (_Float16)r[5][i], (_Float16)r[6][i], (_Float16)r[7][i]};
  return __builtin_bit_cast(u4, h);
}
__device__ __forceinline__ void load_xpiece(f4* r, rsrc_t rx, int vx, int pi) {
  const int so = (pi >> 5) * (16 * 16384) + (pi & 31) * 512;
#pragma unroll
  for (int t = 0; t < 8; ++t) r[t] = ld_in(rx, vx + t * 64, so);
}
__device__ __forceinline__ void load_wgran(f4* r, rsrc_t rw, int vw, int cg, int kg) {
  const int so = kg * (64 * 2048) + cg * 128;
#pragma unroll
  for (int j = 0; j < 8; ++j) r[j] = ld_in(rw, vw + (j & 1) * 2048, so + (16 * (j >> 2) + (j & 2)) * 2048);
}
__device__ __forceinline__ unsigned lds_ld(const unsigned* p) {
  return __hip_atomic_load(p, __ATOMIC_RELAXED, __HIP_MEMORY_SCOPE_WORKGROUP);
}
__device__ __forceinline__ void lds_st(unsigned* p, unsigned v) {
  __hip_atomic_store(p, v, __ATOMIC_RELAXED, __HIP_MEMORY_SCOPE_WORKGROUP);
}

__device__ __forceinline__ int src_off(int d) { return (0x63106301 >> (4 * d)) & 7; }
__device__ __forceinline__ int src_sbase(int d) { return d == 0 ? 0 : d < 4 ? 4 + 3 * (d - 1) : 13 + 2 * (d - 4); }
__device__ __forceinline__ int src_lo(int d) { return d < 4 ? 0 : ND; }
__device__ __forceinline__ int flag_off(int d, int b) {
  return (((b + src_off(d)) & 7) * 32 + src_sbase(d) + ((b - src_lo(d)) >> 3)) * 4;
}
__device__ __forceinline__ bool slot_block(int slot, int gx, bool helper, int q, int& d, int& b) {
  d = slot < 4 ? 0 : slot < 13 ? 1 + (slot - 4) / 3 : (4 + (slot - 13) / 2) & 7;
  const int lo = src_lo(d);
  const int hi = d == 0 ? (helper ? ND : 32) : d < 4 ? ND : ((helper && d == 4 + q) ? 0 : 32);
  b = lo + ((gx - src_off(d) - lo) & 7) + 8 * (slot - src_sbase(d));
  return slot < NSLOT && b < hi;
}

constexpr bool k_in_lds(int t) { return ((t & 15) & 1) && (t & 15) != 15; }
constexpr int lds_index(int t) { return (t >> 4) * 7 + ((t & 15) >> 1); }
constexpr int reg_index(int t) { return (t >> 4) * 9 + ((t & 15) == 15 ? 8 : ((t & 15) >> 1)); }

#define STAMP(idx) do {} while (0)

__global__ void __launch_bounds__(512, 2) ag_gemm_relu(Params p) {
  __shared__ __attribute__((aligned(16))) char smem[LDS_BYTES];
  unsigned* ctl = (unsigned*)(smem + L_CTL);
  const int lane = threadIdx.x & 63;
  const int wid = __builtin_amdgcn_readfirstlane(threadIdx.x >> 6);
  const int gx = blockIdx.x & 7;
  const int cu = blockIdx.x >> 3;
  const bool helper = cu >= 16;
  const int q = (cu >> 2) & 3;
  const rsrc_t rws = mk_rsrc(p.ws[0], WS_USED);
  const rsrc_t rx = mk_rsrc(p.x, 512u * 4096u * 4u);
  const rsrc_t rw = mk_rsrc(p.w, 4096u * 512u * 4u);
  const int vx = (lane & 15) * 16384 + (lane >> 4) * 16;
  const int vw = (32 * (lane >> 5) + 4 * ((lane >> 3) & 3)) * 2048 + (lane & 7) * 16;
  const int vl = lane * 16;
  const int v0 = lane == 0 ? 0 : OOB;

  const unsigned cprev = ld_sys32(rws, 0, OFF_CTR + blockIdx.x * 64);

  const rsrc_t rcnt = mk_rsrc(p.ws[0] + OFF_CNT, CI_END * 64);
  int gen, j, qstep, succwg;
  bool active;
  sender_role(blockIdx.x, gen, j, active, qstep, succwg);

  if (__builtin_expect(wid == 6, 1)) {
    f4 rd[8], ra[8], rb[24];
    if (active && gen == 0) {
      load_xpiece(rd, rx, vx, j);
      load_wgran(ra, rw, vw, j & 3, j >> 2);
#pragma unroll
      for (int t = 0; t < 3; ++t) load_wgran(rb + 8 * t, rw, vw, 4 * (t + 1) + (j & 3), j >> 2);
    }
    asm volatile("" :: "s"(p.ws[1]), "s"(p.ws[2]), "s"(p.ws[3]), "s"(p.ws[4]), "s"(p.ws[5]), "s"(p.ws[6]), "s"(p.ws[7]));
    const unsigned call = __builtin_amdgcn_readfirstlane(cprev) + 1u;
    asm volatile("" :: "s"(call) : "memory");
    __builtin_amdgcn_s_barrier();
    if (!active) return;
    CBAR();
    if (gen != 0) {
      for (;;) {
        const unsigned mk = __builtin_amdgcn_readfirstlane(lds_ld(ctl + C_MASK));
        if ((mk >> F_ABORT) & 1u) return;
        if ((mk >> F_GATE) & 1u) break;
        __builtin_amdgcn_s_sleep(2);
      }
    }
    __builtin_amdgcn_s_setprio(2);
    const unsigned par = call & 1u;
    rsrc_t rp[NR];
#pragma unroll
    for (int d = 1; d < NR; ++d) rp[d] = mk_rsrc(p.ws[d], WS_USED);
    const int xbase = OFF_X + par * (32u << 20);
    const int wbase = OFF_W + par * (8u << 20);
    const int vfl = lane < 8 ? lane * 1024 : OOB;
    const int vgl = lane < 8 ? lane * 128 : OOB;
    int i = gen == 0 ? j : -1, prev = -1;
    if (gen == 0 && j >= S1)
      for (int z = (j - S1) >> 3; z >= -CFG_TIER0; --z) __builtin_amdgcn_s_sleep(CFG_TIER);
    WAITV0();
    for (;;) {
      const bool quit = i >= qstep;
      int cv = CI_END;
      if (lane == 0 && !quit) cv = CI_TICK + 2 * par;
      if (lane == 1 && prev >= 0) cv = CI_SX + (prev >> 5);
      if (lane == 2 && prev >= 0 && prev < E2) cv = prev < NWST ? CI_SW + (prev >> 6) * 4 + (prev & 3) : CI_SX + ND + ((prev - NWST) >> 5);
      const int tv = __builtin_amdgcn_raw_ptr_buffer_atomic_add_i32(1, rcnt, cv * 64, 0, 0);
      int t0v, t1v, t2v;
      if (i == GATE_STEP) {
        st_sys32(call, rws, (128 + lane) * 128, OFF_PL + F_GATE * 4);
        st_sys32(call, rws, (192 + lane) * 128, OFF_PL + F_GATE * 4);
      }
      if (quit && prev < qstep && succwg >= 0) st_sys32(call, rws, v0, OFF_PL + succwg * 128 + F_GATE * 4);
      if (i < 0 || i >= NSTEP) {
        t0v = __builtin_amdgcn_readlane(tv, 0); t1v = __builtin_amdgcn_readlane(tv, 1); t2v = __builtin_amdgcn_readlane(tv, 2);
      } else if (i < NWST) {
        const int wo = wbase + (16 + (i & 3)) * 262144 + (i >> 2) * 4096;
#pragma unroll
        for (int t = 0; t < 4; ++t) {
          const u4 h = cvt8(rd[2 * t], rd[2 * t + 1]);
#pragma unroll
          for (int d = 1; d < 4; ++d) st_sys(h, rp[d], vl + t * 1024, xbase + d * (4 << 20) + i * 4096);
          st_sys(cvtcol(ra, t), rp[4], vl + t * 1024, wo);
#pragma unroll
          for (int d = 5; d < 8; ++d) st_sys(cvtcol(rb + 8 * (d - 5), t), rp[d], vl + t * 1024, wo + (d - 4) * (4 * 262144));
        }
        t0v = __builtin_amdgcn_readlane(tv, 0); t1v = __builtin_amdgcn_readlane(tv, 1); t2v = __builtin_amdgcn_readlane(tv, 2);
      } else if (i < E2) {
        const int xi = NSTEP + i - NWST;
#pragma unroll
        for (int t = 0; t < 4; ++t) {
          const u4 h = cvt8(rd[2 * t], rd[2 * t + 1]);
          const u4 g = cvt8(ra[2 * t], ra[2 * t + 1]);
#pragma unroll
          for (int d = 1; d < 4; ++d) st_sys(h, rp[d], vl + t * 1024, xbase + d * (4 << 20) + i * 4096);
#pragma unroll
          for (int d = 4; d < 8; ++d) st_sys(g, rp[d], vl + t * 1024, xbase + d * (4 << 20) + xi * 4096);
        }
        t0v = __builtin_amdgcn_readlane(tv, 0); t1v = __builtin_amdgcn_readlane(tv, 1); t2v = __builtin_amdgcn_readlane(tv, 2);
      } else {
#pragma unroll
        for (int t = 0; t < 4; ++t) {
          const u4 h = cvt8(rd[2 * t], rd[2 * t + 1]);
#pragma unroll
          for (int d = 1; d < 4; ++d) st_sys(h, rp[d], vl + t * 1024, xbase + d * (4 << 20) + i * 4096);
        }
        t0v = __builtin_amdgcn_readlane(tv, 0); t1v = __builtin_amdgcn_readlane(tv, 1); t2v = __builtin_amdgcn_readlane(tv, 2);
      }
      if (prev >= 0) {
        if ((t1v & 31) == 31) {
#pragma unroll
          for (int d = 1; d < 4; ++d) st_sys32(call, rp[d], vfl, OFF_SL + flag_off(d, prev >> 5));
        }
        if (prev < NWST) {
          if ((t2v & 15) == 15) {
#pragma unroll
            for (int d = 4; d < 8; ++d)
              st_sys32(call, rp[d], vgl, OFF_PL + (16 + 4 * (d - 4) + (prev & 3)) * 1024 + (F_W + (prev >> 6)) * 4);
          }
        } else if (prev < E2) {
          if ((t2v & 31) == 31) {
#pragma unroll
            for (int d = 4; d < 8; ++d) st_sys32(call, rp[d], vfl, OFF_SL + flag_off(d, ND + ((prev - NWST) >> 5)));
          }
        }
      }
      if (i >= NSTEP) break;
      int nxt = t0v + S0;
      if (quit || (unsigned)nxt >= (unsigned)NSTEP) nxt = NSTEP;
      if (nxt < NSTEP) {
        load_xpiece(rd, rx, vx, nxt);
        if (nxt < NWST) {
          load_wgran(ra, rw, vw, nxt & 3, nxt >> 2);
#pragma unroll
          for (int t = 0; t < 3; ++t) load_wgran(rb + 8 * t, rw, vw, 4 * (t + 1) + (nxt & 3), nxt >> 2);
        } else if (nxt < E2) {
          load_xpiece(ra, rx, vx, NSTEP + nxt - NWST);
        }
      }
      WAITV0();
      prev = i;
      i = nxt;
    }
    if (lane == 0) lds_st(ctl + C_SDONE, 1u);
    STAMP(1);
    return;
  }

  int sd, sb;
  const bool sused = slot_block(lane, gx, helper, q, sd, sb);
  const unsigned needmask = (unsigned)__ballot(sused);
  const int nneed = __builtin_popcount(needmask);

  if (wid >= 4) {
    if (wid == 4) lds_st(ctl + lane, 0u);
    const unsigned call = __builtin_amdgcn_readfirstlane(cprev) + 1u;
    asm volatile("s_waitcnt lgkmcnt(0)" :: "s"(call) : "memory");
    __builtin_amdgcn_s_barrier();
    CBAR();
    __builtin_amdgcn_s_setprio(2);
    if (wid == 4) {
      const unsigned want = needmask | (15u << F_W) | (helper || active ? 1u << F_GATE : 0u) | (helper ? 0u : 1u << F_DONE);
      bool hold = active;
      const int pv = lane < NSLOT ? (int)(OFF_SL + ((cu >> 2) * 8 + gx) * 128 + lane * 4)
                   : lane < 32 ? (int)(OFF_PL + blockIdx.x * 128 + lane * 4) : lane == 32 ? (int)OFF_TMO : OOB;
      const unsigned boff = OFF_X + (call & 1u) * (32u << 20) + sd * (4 << 20) + sb * 131072;
      unsigned seen = 0;
      const u64 t0 = __builtin_amdgcn_s_memrealtime();
      for (;;) {
        const unsigned fv = ld_sys32(rws, pv, 0);
        unsigned m = (unsigned)__ballot((int)(fv - call) >= 0) & want;
        if (hold && (gen == 0 || ((m >> F_GATE) & 1u))) {
          if (__builtin_amdgcn_readfirstlane(lds_ld(ctl + C_SDONE)) != 0u) hold = false;
          else m = seen | (m & (1u << F_GATE));
        }
        if (m != seen) {
          const unsigned fresh = m & ~seen & needmask;
          if (lane < 32 && ((fresh >> lane) & 1u)) {
            const int pos = __builtin_popcount(seen & needmask) + __builtin_popcount(fresh >> lane) - 1;
            lds_st(ctl + C_QSLOT + pos, (unsigned)lane);
            lds_st(ctl + C_QOFF + pos, boff);
          }
          if (lane == 0) lds_st(ctl + C_MASK, m);
          seen = m;
          if (m == want) break;
        }
        if (__builtin_amdgcn_readlane(fv, 32) == call || __builtin_amdgcn_s_memrealtime() - t0 > DEADLINE) {
          st_sys32(call, rws, v0, OFF_TMO);
          if (lane == 0) lds_st(ctl + C_MASK, seen | (1u << F_ABORT));
          break;
        }
        __builtin_amdgcn_s_sleep(1);
      }
      STAMP(7);
    } else if (wid == 5) {
      unsigned pending = needmask;
      const int rowoff = (((p.rank ^ sd) & 7) * 512 + sb * 16) * 2048;
      const int vo = (lane >> 3) * 2048 + (lane & 7) * 16 + (cu & 15) * 128;
      const rsrc_t ro = mk_rsrc(!helper ? p.out[0] : q == 0 ? p.out[4] : q == 1 ? p.out[5] : q == 2 ? p.out[6] : p.out[7], 4096u * 512u * 4u);
      char* wsq = q == 0 ? p.ws[4] : q == 1 ? p.ws[5] : q == 2 ? p.ws[6] : p.ws[7];
      asm volatile("" : "+s"(wsq));
      int bi = 0;
      bool abort = false;
      for (int n = 0; n < nneed + (helper ? 0 : 1); ++n) {
        if (helper) WAITV0();
        unsigned c;
        for (;;) {
          c = lds_ld(ctl + lane);
          const unsigned mk = __builtin_amdgcn_readlane(c, C_MASK);
          abort = (mk >> F_ABORT) & 1u;
          if (abort || (n == nneed && ((mk >> F_DONE) & 1u)) ||
              ((int)__builtin_amdgcn_readlane(c, C_RED) > n && (!helper || ((mk >> F_GATE) & 1u)))) break;
          __builtin_amdgcn_s_sleep(1);
        }
        if (abort || n == nneed) break;
        CBAR();
        const int sl = __builtin_amdgcn_readlane(c, C_QSLOT + n);
        const int ro_s = __builtin_amdgcn_readlane(rowoff, sl);
        pending &= ~(1u << sl);
        st_sys(*(const u4*)(smem + L_BACK + bi * 2048 + vl), ro, vo, ro_s);
        st_sys(*(const u4*)(smem + L_BACK + bi * 2048 + 1024 + vl), ro, vo, ro_s + 16384);
        CBAR();
        if (lane == 0) lds_st(ctl + C_FL, (unsigned)(n + 1));
        bi = bi + 1 == NBACK ? 0 : bi + 1;
      }
      if (abort) {
        pending |= 1u;
        WAITV0();
        while (pending != 0u) {
          const int ro_s = __builtin_amdgcn_readlane(rowoff, __builtin_ctz(pending));
          pending &= pending - 1u;
          const u4 nan4 = u4{0x7fc00000u, 0x7fc00000u, 0x7fc00000u, 0x7fc00000u};
          st_sys(nan4, ro, vo, ro_s);
          st_sys(nan4, ro, vo, ro_s + 16384);
        }
      }
      if (helper) {
        WAITV0();
        st_sys32(call, mk_rsrc(wsq, WS_USED), v0, OFF_PL + (blockIdx.x - 128) * 128 + F_DONE * 4);
      }
      STAMP(5);
    } else {
      st_sys32(call, rws, v0, OFF_CTR + blockIdx.x * 64);
      if (blockIdx.x == 0) st_sys32(0u, rcnt, v0, (CI_TICK + 2 * ((call & 1u) ^ 1u)) * 64);
      STAMP(6);
      const int wo = (lane >> 4) * 512 + (lane & 7) * 16 + ((lane >> 3) & 1) * 4;
      int bi = 0;
      for (int n = 0; n < nneed; ++n) {
        bool abort;
        for (;;) {
          const unsigned c = lds_ld(ctl + lane);
          abort = (__builtin_amdgcn_readlane(c, C_MASK) >> F_ABORT) & 1u;
          const int pc = (n & 1) ? __builtin_amdgcn_readlane(c, C_PCNT + 1) : __builtin_amdgcn_readlane(c, C_PCNT);
          if (abort || (pc >= 4 * ((n >> 1) + 1) && n - (int)__builtin_amdgcn_readlane(c, C_FL) < NBACK)) break;
          __builtin_amdgcn_s_sleep(1);
        }
        if (abort) break;
        CBAR();
        const char* pp = smem + L_PART + (n & 1) * 8192 + vl;
        float* bo = (float*)(smem + L_BACK + bi * 2048 + wo);
#pragma unroll
        for (int m = 0; m < 2; ++m) {
          const f4 v = (*(const f4*)(pp + m * 1024) + *(const f4*)(pp + 2048 + m * 1024)) +
                       (*(const f4*)(pp + 4096 + m * 1024) + *(const f4*)(pp + 6144 + m * 1024));
#pragma unroll
          for (int jj = 0; jj < 4; ++jj) bo[jj * 32 + m * 2] = v[jj] < 0.f ? 0.f : v[jj];
        }
        CBAR();
        if (lane == 0) lds_st(ctl + C_RED, (unsigned)(n + 1));
        bi = bi + 1 == NBACK ? 0 : bi + 1;
      }
    }
    return;
  }

  {
    const int gi = blockIdx.x * 4 + wid;
    const unsigned call = __builtin_amdgcn_readfirstlane(cprev) + 1u;
    asm volatile("" :: "s"(call) : "memory");
    __builtin_amdgcn_s_barrier();
    CBAR();
    __builtin_amdgcn_s_sleep(CFG_STAGE_DELAY);
    f4 sx[8], sw[8];
    load_xpiece(sx, rx, vx, gi);
    load_wgran(sw, rw, vw, gi >> 6, gi & 63);
    const unsigned par = call & 1u;
    const int xbase = OFF_X + par * (32u << 20);
    const int wbase = OFF_W + par * (8u << 20);
#pragma unroll
    for (int t = 0; t < 4; ++t) st_sys(cvtcol(sw, t), rws, vl + t * 1024, wbase + gi * 4096);
#pragma unroll
    for (int t = 0; t < 4; ++t) st_sys(cvt8(sx[2 * t], sx[2 * t + 1]), rws, vl + t * 1024, xbase + gi * 4096);
    WAITV0();
    {
      const int tv = __builtin_amdgcn_raw_ptr_buffer_atomic_add_i32(1, rcnt, (lane == 0 ? CI_LX + (gi >> 5) : lane == 1 ? CI_LW + (gi >> 4) : CI_END) * 64, 0, 0);
      if ((__builtin_amdgcn_readlane(tv, 0) & 31) == 31)
        st_sys32(call, rws, lane < 8 ? lane * 1024 : OOB, OFF_SL + flag_off(0, gi >> 5));
      if ((__builtin_amdgcn_readlane(tv, 1) & 15) == 15)
        st_sys32(call, rws, lane < 8 ? lane * 128 : OOB, OFF_PL + (gi >> 6) * 1024 + (F_W + ((gi >> 4) & 3)) * 4);
    }

    for (;;) {
      const unsigned mk = __builtin_amdgcn_readfirstlane(lds_ld(ctl + C_MASK));
      if ((mk >> F_ABORT) & 1u) return;
      if ((mk >> (F_W + wid)) & 1u) break;
      __builtin_amdgcn_s_sleep(4);
    }
    CBAR();
    int qo[8];
#pragma unroll
    for (int qd = 0; qd < 8; ++qd) qo[qd] = ((qd + cu) & 7) * 4096;
    const int vb = ((lane >> 3) & 1) * 1024 + (lane >> 4) * 128 + (lane & 7) * 16;
    u4 breg[18][2];
    char* ldsb = smem + L_B + wid * (NLDSK * 2048) + vl;
#pragma unroll
    for (int hf = 0; hf < 2; ++hf) {
      const int so = wbase + cu * 262144 + wid * 65536;
      u4 tmp[7][2];
#pragma unroll
      for (int t = 0; t < 16; ++t) {
#pragma unroll
        for (int m = 0; m < 2; ++m) {
          const u4 v = ld_sys(rws, vb + m * 2048 + (t & 1) * 512, so + 2 * qo[hf * 4 + (t >> 2)] + ((t >> 1) & 1) * 4096);
          if (k_in_lds(t)) tmp[t >> 1][m] = v; else breg[reg_index(hf * 16 + t)][m] = v;
        }
      }
#pragma unroll
      for (int t = 0; t < 7; ++t) {
#pragma unroll
        for (int m = 0; m < 2; ++m) *(u4*)(ldsb + (hf * 7 + t) * 2048 + m * 1024) = tmp[t][m];
      }
    }
    if (wid == 0) STAMP(8);

    u4 ring[16];
    u4 bt[2][2];
#pragma unroll
    for (int n = 0; n < 2; ++n) { bt[n][0] = *(const u4*)(ldsb + n * 2048); bt[n][1] = *(const u4*)(ldsb + n * 2048 + 1024); }
    int va = vl + wid * 32768;
    int cur = (int)WS_USED;
    for (int n = 0; n < nneed; ++n) {
      if (cur == (int)WS_USED) {
        for (;;) {
          const unsigned c = lds_ld(ctl + (lane == 0 ? C_QOFF + n : C_MASK));
          cur = __builtin_amdgcn_readlane(c, 0);
          if (cur != 0) break;
          if ((__builtin_amdgcn_readlane(c, 1) >> F_ABORT) & 1u) return;
          __builtin_amdgcn_s_sleep(1);
        }
        asm volatile("" : "+v"(va) :: "memory");
#pragma unroll
        for (int t = 0; t < 16; ++t) {
          ring[t] = ld_sys(rws, va + (t & 3) * 1024, cur + qo[t >> 2]);
          __builtin_amdgcn_sched_barrier(0);
        }
      }
      f4 acc0 = f4{0.f, 0.f, 0.f, 0.f}, acc1 = f4{0.f, 0.f, 0.f, 0.f};
      int nb = 0;
      unsigned r0 = 0;
      asm volatile("" : "+v"(va));
#pragma unroll
      for (int t = 0; t < 32; ++t) {
        if (t == 16) { nb = __builtin_amdgcn_readfirstlane(r0); nb = nb != 0 ? nb : (int)WS_USED; CBAR(); }
        const u4 b0 = k_in_lds(t) ? bt[lds_index(t) & 1][0] : breg[reg_index(t)][0];
        const u4 b1 = k_in_lds(t) ? bt[lds_index(t) & 1][1] : breg[reg_index(t)][1];
        const h8 a = __builtin_bit_cast(h8, ring[t & 15]);
        acc0 = __builtin_amdgcn_mfma_f32_16x16x32_f16(a, __builtin_bit_cast(h8, b0), acc0, 0, 0, 0);
        acc1 = __builtin_amdgcn_mfma_f32_16x16x32_f16(a, __builtin_bit_cast(h8, b1), acc1, 0, 0, 0);
        __builtin_amdgcn_sched_barrier(0);
        ring[t & 15] = ld_sys(rws, va + (t & 3) * 1024, (t < 16 ? cur : nb) + qo[((t + 16) & 31) >> 2]);
        if (k_in_lds(t)) {
          const int nn = (lds_index(t) + 2) % NLDSK;
          bt[nn & 1][0] = *(const u4*)(ldsb + nn * 2048);
          bt[nn & 1][1] = *(const u4*)(ldsb + nn * 2048 + 1024);
        }
        if (t == 14) r0 = lds_ld(ctl + C_QOFF + n + 1);
        if (t == 30) r0 = lds_ld(ctl + C_RED);
        __builtin_amdgcn_sched_barrier(0);
      }
      int red = __builtin_amdgcn_readfirstlane(r0);
      while (red < n - 1) {
        __builtin_amdgcn_s_sleep(1);
        const unsigned c = lds_ld(ctl + (lane == 0 ? C_RED : C_MASK));
        if ((__builtin_amdgcn_readlane(c, 1) >> F_ABORT) & 1u) return;
        red = __builtin_amdgcn_readlane(c, 0);
      }
      CBAR();
      char* pp = smem + L_PART + (n & 1) * 8192 + wid * 2048 + vl;
      *(f4*)pp = acc0;
      *(f4*)(pp + 1024) = acc1;
      CBAR();
      if (lane == 0) __hip_atomic_fetch_add(ctl + C_PCNT + (n & 1), 1u, __ATOMIC_RELAXED, __HIP_MEMORY_SCOPE_WORKGROUP);
      CBAR();
      cur = nb;
    }
  }
}

}

extern "C" void kernel_launch(int rank, int nranks, ncclComm_t comm,
                              hipStream_t stream,
                              void* const* d_in, const int* in_sizes, int n_in,
                              void* d_out, int out_size,
                              const int* in_dtypes, int out_dtype,
                              void* const* d_out_all,
                              void* const* d_ws_all, size_t ws_bytes) {
  if (nranks != NR || ws_bytes < (size_t)WS_USED + (1u << 20) || n_in < 2 || (unsigned)rank >= (unsigned)NR) {
    fprintf(stderr, "unsupported configuration\n");
    return;
  }
  Params p;
  p.x = (const float*)d_in[0];
  p.w = (const float*)d_in[1];
  for (int d = 0; d < NR; ++d) {
    p.ws[d] = (char*)d_ws_all[rank ^ d];
    p.out[d] = (float*)d_out_all[rank ^ d];
  }
  p.rank = rank;
  hipLaunchKernelGGL(ag_gemm_relu, dim3(256), dim3(512), 0, stream, p);
  HIP_CHECK(hipGetLastError());
}
```

```cpp
#include <hip/hip_runtime.h>
#include <rccl/rccl.h>
#include <cstdio>

#define HIP_CHECK(expr) do { hipError_t _e = (expr); if (_e != hipSuccess) { \
    fprintf(stderr, "HIP error %s at %s:%d\n", hipGetErrorString(_e), __FILE__, __LINE__); \
    return; } } while (0)

namespace {

typedef float f4 __attribute__((ext_vector_type(4)));
typedef unsigned u4 __attribute__((ext_vector_type(4)));
typedef _Float16 h8 __attribute__((ext_vector_type(8)));
typedef unsigned long long u64;
typedef __amdgpu_buffer_rsrc_t rsrc_t;

#define CFG_ND 22
#define CFG_S3 18
#define CFG_Q2 16
#define CFG_GATEBACK 12
#define CFG_FWIN 2
#define CFG_ONE 96
#define CFG_STAGE_DELAY 40
#define CFG_TIER 26
#define CFG_RAWTAIL 3
#define CFG_OFFS 0x65314201

constexpr int NR = 8;
constexpr int NG = 7;
constexpr int ND = CFG_ND;
constexpr int NXB = 32 - ND;
constexpr int S1 = 32;
constexpr int S3 = CFG_S3;
constexpr int ONE = CFG_ONE;
constexpr int S0 = S1 + ONE;
constexpr int NSTEP = ND * 32;
constexpr int NWST = 256;
constexpr int E2 = NWST + NXB * 32;
constexpr int GATE_STEP = E2 - CFG_GATEBACK;
constexpr int RAWTAIL = CFG_RAWTAIL;
static_assert(ND >= 20 && ND <= 24, "slot table");
static_assert(S3 > 0 && S3 <= S1 && ONE <= 112 && S0 <= NWST && GATE_STEP >= 0 && GATE_STEP < NSTEP, "sender configuration");

constexpr u64 DEADLINE = 100000000ull;

constexpr unsigned OFF_PL = 0;
constexpr unsigned OFF_TMO = 40960;
constexpr unsigned OFF_CNT = 45056;
constexpr unsigned OFF_CTR = 65536;
constexpr unsigned OFF_SL = 98304;
constexpr unsigned OFF_RP = 110592;
constexpr unsigned OFF_DIAG = 131072;
constexpr unsigned OFF_X = 1u << 20;
constexpr unsigned OFF_W = 65u << 20;
constexpr unsigned OFF_T = 81u << 20;
constexpr unsigned WS_USED = 87u << 20;
constexpr int TBYTES = 49152;
static_assert(112u * TBYTES <= WS_USED - OFF_T && 22 * 2176 <= TBYTES, "tile region");
constexpr int CI_TICK = 0;
constexpr int CI_SX = 4;
constexpr int CI_SW = 36;
constexpr int CI_LX = 52;
constexpr int CI_LW = 84;
constexpr int CI_END = 148;
constexpr int NSLOT = 25;
constexpr int F_W = 25;
constexpr int F_GATE = 29;
constexpr int F_ABORT = 31;

constexpr int NLDSK = 14;
constexpr int NBACK = 15;
constexpr int NBOWN = 8;
constexpr int BPITCH = 2144;
constexpr int L_B = 0;
constexpr int L_PART = 4 * NLDSK * 2048;
constexpr int L_BACK = L_PART + 16384;
constexpr int L_CTL = L_BACK + NBACK * BPITCH;
constexpr int C_QOFF = 0;
constexpr int C_QSLOT = 24;
constexpr int C_PCNT = 48;
constexpr int C_RED = 50;
constexpr int C_FL = 51;
constexpr int C_MASK = 52;
constexpr int C_WORDS = 64;
constexpr int L_EXP = L_BACK + NBOWN * BPITCH;
constexpr int LDS_BYTES = L_CTL + C_WORDS * 4;
static_assert(LDS_BYTES <= 163840, "LDS");

struct Params {
  const float* x;
  const float* w;
  char* ws[NR];
  float* out[NR];
  int rank;
};

#define WAITV0() do { asm volatile("" ::: "memory"); __builtin_amdgcn_s_waitcnt(0x0F70); asm volatile("" ::: "memory"); } while (0)
#define CBAR() asm volatile("" ::: "memory")
constexpr int OOB = 0x40000000;

__device__ __forceinline__ rsrc_t mk_rsrc(const void* p, unsigned bytes) {
  return __builtin_amdgcn_make_buffer_rsrc(const_cast<void*>(p), 0, bytes, 0x00020000);
}
__device__ __forceinline__ f4 ld_in(rsrc_t r, int voff, int soff) {
  return __builtin_bit_cast(f4, __builtin_amdgcn_raw_buffer_load_b128(r, voff, soff, 0));
}
__device__ __forceinline__ u4 ld_sys(rsrc_t r, int voff, int soff) {
  return __builtin_amdgcn_raw_buffer_load_b128(r, voff, soff, 17);
}
__device__ __forceinline__ unsigned ld_sys32(rsrc_t r, int voff, int soff) {
  return __builtin_amdgcn_raw_buffer_load_b32(r, voff, soff, 17);
}
__device__ __forceinline__ void st_sys(u4 v, rsrc_t r, int voff, int soff) {
  __builtin_amdgcn_raw_buffer_store_b128(v, r, voff, soff, 17);
}
__device__ __forceinline__ void st_sys32(unsigned v, rsrc_t r, int voff, int soff) {
  __builtin_amdgcn_raw_buffer_store_b32(v, r, voff, soff, 17);
}
__device__ __forceinline__ u4 cvt8(f4 a, f4 b) {
  h8 h = h8{(_Float16)a.x, (_Float16)a.y, (_Float16)a.z, (_Float16)a.w,
            (_Float16)b.x, (_Float16)b.y, (_Float16)b.z, (_Float16)b.w};
  return __builtin_bit_cast(u4, h);
}
__device__ __forceinline__ u4 cvtcol(const f4* r, int i) {
  h8 h = h8{(_Float16)r[0][i], (_Float16)r[1][i], (_Float16)r[2][i], (_Float16)r[3][i],
            (_Float16)r[4][i], (_Float16)r[5][i], (_Float16)r[6][i], (_Float16)r[7][i]};
  return __builtin_bit_cast(u4, h);
}
__device__ __forceinline__ void load_xpiece(f4* r, rsrc_t rx, int vx, int pi) {
  const int so = (pi >> 5) * (16 * 16384) + (pi & 31) * 512;
#pragma unroll
  for (int t = 0; t < 8; ++t) r[t] = ld_in(rx, vx + t * 64, so);
}
__device__ __forceinline__ void load_wgran(f4* r, rsrc_t rw, int vw, int cg, int kg) {
  const int so = kg * (64 * 2048) + cg * 128;
#pragma unroll
  for (int j = 0; j < 8; ++j) r[j] = ld_in(rw, vw + (j & 1) * 2048, so + (16 * (j >> 2) + (j & 2)) * 2048);
}
__device__ __forceinline__ unsigned lds_ld(const unsigned* p) {
  return __hip_atomic_load(p, __ATOMIC_RELAXED, __HIP_MEMORY_SCOPE_WORKGROUP);
}
__device__ __forceinline__ void lds_st(unsigned* p, unsigned v) {
  __hip_atomic_store(p, v, __ATOMIC_RELAXED, __HIP_MEMORY_SCOPE_WORKGROUP);
}

__device__ __forceinline__ int src_off(int d) { return (CFG_OFFS >> (4 * d)) & 7; }
__device__ __forceinline__ int src_sbase(int d) { return d == 0 ? 0 : d < 4 ? 1 + 4 * d : 9 + 2 * d; }
__device__ __forceinline__ int src_lo(int d) { return d < 4 ? 0 : ND; }
__device__ __forceinline__ int flag_off(int d, int b) {
  return (((b + src_off(d)) % NG) * 32 + src_sbase(d) + (b - src_lo(d)) / NG) * 4;
}
__device__ __forceinline__ bool slot_block(int slot, int g, bool helper, int q, int& d, int& b) {
  d = slot < 5 ? 0 : slot < 17 ? (slot - 1) >> 2 : ((slot - 9) >> 1) & 7;
  const int lo = src_lo(d);
  const int hi = d == 0 ? (helper ? ND : 32) : d < 4 ? ND : ((helper && d == 4 + q) ? 0 : 32);
  b = lo + (g + 70 - src_off(d) - lo) % NG + NG * (slot - src_sbase(d));
  return slot < NSLOT && b < hi;
}
__device__ __forceinline__ void stage_local(int gi, unsigned call, rsrc_t rws, rsrc_t rx, rsrc_t rw, rsrc_t rcnt, int vx, int vw, int lane) {
  f4 sx[8], sw[8];
  load_xpiece(sx, rx, vx, gi);
  load_wgran(sw, rw, vw, gi >> 6, gi & 63);
  const int xbase = OFF_X + (call & 1u) * (32u << 20);
  const int wbase = OFF_W + (call & 1u) * (8u << 20);
#pragma unroll
  for (int t = 0; t < 4; ++t) st_sys(cvtcol(sw, t), rws, lane * 16 + t * 1024, wbase + gi * 4096);
#pragma unroll
  for (int t = 0; t < 4; ++t) st_sys(cvt8(sx[2 * t], sx[2 * t + 1]), rws, lane * 16 + t * 1024, xbase + gi * 4096);
  WAITV0();
  const int tv = __builtin_amdgcn_raw_ptr_buffer_atomic_add_i32(1, rcnt, (lane == 0 ? CI_LX + (gi >> 5) : lane == 1 ? CI_LW + (gi >> 4) : CI_END) * 64, 0, 0);
  if ((__builtin_amdgcn_readlane(tv, 0) & 31) == 31)
    st_sys32(call, rws, lane < 8 ? lane * 1024 : OOB, OFF_SL + flag_off(0, gi >> 5));
  if ((__builtin_amdgcn_readlane(tv, 1) & 15) == 15)
    st_sys32(call, rws, lane < NG ? lane * 4096 : OOB, OFF_PL + (gi >> 6) * 128 + (F_W + ((gi >> 4) & 3)) * 4);
}

constexpr bool k_in_lds(int t) { return ((t & 15) & 1) && (t & 15) != 15; }
constexpr int lds_index(int t) { return (t >> 4) * 7 + ((t & 15) >> 1); }
constexpr int reg_index(int t) { return (t >> 4) * 9 + ((t & 15) == 15 ? 8 : ((t & 15) >> 1)); }

#define STAMP(idx) do {} while (0)

__global__ void __launch_bounds__(512, 2) ag_gemm_relu(Params p) {
  __shared__ __attribute__((aligned(16))) char smem[LDS_BYTES];
  unsigned* ctl = (unsigned*)(smem + L_CTL);
  const int lane = threadIdx.x & 63;
  const int wid = __builtin_amdgcn_readfirstlane(threadIdx.x >> 6);
  const bool comm = blockIdx.x < 32;
  const int cidx = (blockIdx.x & 7) * 28 + (blockIdx.x >> 3) - 4;
  const int g = cidx >> 5;
  const int cu = cidx & 31;
  const bool helper = cu >= 16;
  const int q = (cu >> 2) & 3;
  const rsrc_t rws = mk_rsrc(p.ws[0], WS_USED);
  const rsrc_t rx = mk_rsrc(p.x, 512u * 4096u * 4u);
  const rsrc_t rw = mk_rsrc(p.w, 4096u * 512u * 4u);
  const rsrc_t rcnt = mk_rsrc(p.ws[0] + OFF_CNT, CI_END * 64);
  const int vx = (lane & 15) * 16384 + (lane >> 4) * 16;
  const int vw = (32 * (lane >> 5) + 4 * ((lane >> 3) & 3)) * 2048 + (lane & 7) * 16;
  const int vl = lane * 16;
  const int v0 = lane == 0 ? 0 : OOB;

  const unsigned cprev = ld_sys32(rws, 0, OFF_CTR + blockIdx.x * 64);

  if (__builtin_expect(comm ? wid == 0 : wid == 6 && helper, 1)) {
    const int j = comm ? (int)blockIdx.x : S1 + (cu - 16) * NG + g;
    const bool active = j < S0;
    f4 rd[8], ra[8], rb[24];
    if (active) {
      load_xpiece(rd, rx, vx, j);
      load_wgran(ra, rw, vw, j & 3, j >> 2);
#pragma unroll
      for (int t = 0; t < 3; ++t) load_wgran(rb + 8 * t, rw, vw, 4 * (t + 1) + (j & 3), j >> 2);
    }
    asm volatile("" :: "s"(p.ws[1]), "s"(p.ws[2]), "s"(p.ws[3]), "s"(p.ws[4]), "s"(p.ws[5]), "s"(p.ws[6]), "s"(p.ws[7]));
    const unsigned call = __builtin_amdgcn_readfirstlane(cprev) + 1u;
    asm volatile("" :: "s"(call) : "memory");
    __builtin_amdgcn_s_barrier();
    if (!active) return;
    CBAR();
    __builtin_amdgcn_s_setprio(2);
    const unsigned par = call & 1u;
    rsrc_t rp[NR];
#pragma unroll
    for (int d = 1; d < NR; ++d) rp[d] = mk_rsrc(p.ws[d], WS_USED);
    const int xbase = OFF_X + par * (32u << 20);
    const int wbase = OFF_W + par * (8u << 20);
    const int vfl = lane < 8 ? lane * 1024 : OOB;
    const int vgl = lane < NG ? lane * 4096 : OOB;
    const int qstep = !comm ? 0 : j >= S3 ? E2 - CFG_Q2 : NSTEP;
    int i = j, prev = -1;
    if (j >= S1)
      for (int z = (j - S1) >> 3; z >= 0; --z) __builtin_amdgcn_s_sleep(CFG_TIER);
    WAITV0();
    for (;;) {
      const bool quit = i >= qstep;
      int cv = CI_END;
      if (lane == 0 && !quit) cv = CI_TICK + 2 * par;
      if (lane == 1 && prev >= 0) cv = CI_SX + (prev >> 5);
      if (lane == 2 && prev >= 0 && prev < E2) cv = prev < NWST ? CI_SW + (prev >> 6) * 4 + (prev & 3) : CI_SX + ND + ((prev - NWST) >> 5);
      const int tv = __builtin_amdgcn_raw_ptr_buffer_atomic_add_i32(1, rcnt, cv * 64, 0, 0);
      int t0v, t1v, t2v;
#define READ_BACK() do { t0v = __builtin_amdgcn_readlane(tv, 0); t1v = __builtin_amdgcn_readlane(tv, 1); t2v = __builtin_amdgcn_readlane(tv, 2); } while (0)
      if (i == GATE_STEP) {
        st_sys32(call, rws, (lane >> 4) * 4096 + (lane & 15) * 128, OFF_PL + 16 * 128 + F_GATE * 4);
        st_sys32(call, rws, lane < 48 ? (lane >> 4) * 4096 + (lane & 15) * 128 : OOB, OFF_PL + 4 * 4096 + 16 * 128 + F_GATE * 4);
      }
      if (i >= NSTEP) {
        READ_BACK();
      } else if (i < NWST) {
        const int wo = wbase + (16 + (i & 3)) * 262144 + (i >> 2) * 4096;
#pragma unroll
        for (int t = 0; t < 4; ++t) {
          const u4 h = cvt8(rd[2 * t], rd[2 * t + 1]);
#pragma unroll
          for (int d = 1; d < 4; ++d) st_sys(h, rp[d], vl + t * 1024, xbase + d * (4 << 20) + i * 4096);
          st_sys(cvtcol(ra, t), rp[4], vl + t * 1024, wo);
#pragma unroll
          for (int d = 5; d < 8; ++d) st_sys(cvtcol(rb + 8 * (d - 5), t), rp[d], vl + t * 1024, wo + (d - 4) * (4 * 262144));
        }
        READ_BACK();
      } else if (i < E2) {
        const int xi = NSTEP + i - NWST;
#pragma unroll
        for (int t = 0; t < 4; ++t) {
          const u4 h = cvt8(rd[2 * t], rd[2 * t + 1]);
          const u4 gg = cvt8(ra[2 * t], ra[2 * t + 1]);
#pragma unroll
          for (int d = 1; d < 4; ++d) st_sys(h, rp[d], vl + t * 1024, xbase + d * (4 << 20) + i * 4096);
#pragma unroll
          for (int d = 4; d < 8; ++d) st_sys(gg, rp[d], vl + t * 1024, xbase + d * (4 << 20) + xi * 4096);
        }
        READ_BACK();
      } else {
#pragma unroll
        for (int t = 0; t < 4; ++t) {
          const u4 h = cvt8(rd[2 * t], rd[2 * t + 1]);
#pragma unroll
          for (int d = 1; d < 4; ++d) st_sys(h, rp[d], vl + t * 1024, xbase + d * (4 << 20) + i * 4096);
        }
        READ_BACK();
      }
      if (prev >= 0) {
        if ((t1v & 31) == 31) {
#pragma unroll
          for (int d = 1; d < 4; ++d) st_sys32(call, rp[d], vfl, OFF_SL + flag_off(d, prev >> 5));
        }
        if (prev < NWST) {
          if ((t2v & 15) == 15) {
#pragma unroll
            for (int d = 4; d < 8; ++d)
              st_sys32(call, rp[d], vgl, OFF_PL + (16 + 4 * (d - 4) + (prev & 3)) * 128 + (F_W + (prev >> 6)) * 4);
          }
        } else if (prev < E2) {
          if ((t2v & 31) == 31) {
#pragma unroll
            for (int d = 4; d < 8; ++d) st_sys32(call, rp[d], vfl, OFF_SL + flag_off(d, ND + ((prev - NWST) >> 5)));
          }
        }
      }
      if (i >= NSTEP) break;
      int nxt = t0v + S0;
      if (quit || (unsigned)nxt >= (unsigned)NSTEP) nxt = NSTEP;
      if (nxt < NSTEP) {
        load_xpiece(rd, rx, vx, nxt);
        if (nxt < NWST) {
          load_wgran(ra, rw, vw, nxt & 3, nxt >> 2);
#pragma unroll
          for (int t = 0; t < 3; ++t) load_wgran(rb + 8 * t, rw, vw, 4 * (t + 1) + (nxt & 3), nxt >> 2);
        } else if (nxt < E2) {
          load_xpiece(ra, rx, vx, NSTEP + nxt - NWST);
        }
      }
      WAITV0();
      prev = i;
      i = nxt;
    }
    return;
  }

  if (comm || wid == 6) {
    const unsigned call = __builtin_amdgcn_readfirstlane(cprev) + 1u;
    asm volatile("" :: "s"(call) : "memory");
    __builtin_amdgcn_s_barrier();
    CBAR();
    if (comm) {
      if (wid == 7) {
        st_sys32(call, rws, v0, OFF_CTR + blockIdx.x * 64);
        if (blockIdx.x == 31) st_sys32(0u, rcnt, v0, (CI_TICK + 2 * ((call & 1u) ^ 1u)) * 64);
        STAMP(6);
      }
      if (wid >= 4) {
        __builtin_amdgcn_s_sleep(CFG_STAGE_DELAY);
        stage_local(896 + blockIdx.x * 4 + wid - 4, call, rws, rx, rw, rcnt, vx, vw, lane);
      }
      return;
    }
    __builtin_amdgcn_s_setprio(2);
    const int z = q, sub = cu & 3, eg = g, e = z * 28 + sub * NG + g;
    int pd, pb;
    unsigned ppend = (unsigned)__ballot(slot_block(lane, eg, true, z, pd, pb));
    const int prow = (((p.rank ^ (4 + z) ^ pd) & 7) * 512 + pb * 16) * 2048;
    const rsrc_t ro = mk_rsrc(p.out[0], 4096u * 512u * 4u);
    const int vo = (lane >> 3) * 2048 + (lane & 7) * 16 + (4 * z + sub) * 128;
    const int tb = OFF_T + e * TBYTES;
    char* sc = smem + L_EXP;
    const int jr = (lane >> 3) & 3;
    const int sl0 = (lane >> 5) * 16 + (lane & 7);
    const int pv = lane == 0 ? (int)(OFF_RP + e * 128) : lane == 2 ? (int)OFF_TMO : OOB;
    int rl = 0, tried = 0, oc = 0;
    bool abort = false;
    const u64 t0 = __builtin_amdgcn_s_memrealtime();
    for (;;) {
      const unsigned fv = ld_sys32(rws, pv, 0);
      const unsigned ow = __builtin_amdgcn_readlane(fv, 0) - (call << 14);
      if (ow < 1024u && (int)ow > oc) oc = (int)ow;
      const int avail = (oc & 511) - rl;
      if (avail > tried) {
        const int got = avail < 64 ? avail : 64;
        u4 t[8];
#pragma unroll
        for (int y = 0; y < 8; ++y) t[y] = ld_sys(rws, (lane >> 3) + 8 * y < got ? vl : OOB, tb + rl * 128 + y * 1024);
#pragma unroll
        for (int y = 0; y < 8; ++y) *(u4*)(sc + y * 1024 + vl) = t[y];
        CBAR();
        int at = 0;
        while (at < got) {
          const char* h = sc + at * 128;
          const unsigned hd = *(const unsigned*)(h + 80 + (lane & 1) * 4);
          const u64 mk0 = *(const u64*)(h + jr * 8), mk1 = *(const u64*)(h + 32 + jr * 8);
          const int b0 = *(const unsigned short*)(h + 64 + jr * 2), b1 = *(const unsigned short*)(h + 72 + jr * 2);
          const int nl = (96 + 127 + 4 * (__builtin_amdgcn_readlane(hd, 1) & 1023)) >> 7;
          if (at + nl > got) break;
          const int sl = __builtin_amdgcn_readlane(hd, 0) & 31;
          const int ro_s = __builtin_amdgcn_readlane(prow, sl);
          ppend &= ~(1u << sl);
          unsigned o[2][4];
#pragma unroll
          for (int m = 0; m < 2; ++m) {
            const u64 mk = m ? mk1 : mk0;
            const int base = m ? b1 : b0;
#pragma unroll
            for (int hh = 0; hh < 2; ++hh) {
#pragma unroll
              for (int i1 = 0; i1 < 2; ++i1) {
                const int s = sl0 + hh * 32 + i1 * 8;
                const int idx = (base + __builtin_popcountll(mk & ((1ull << s) - 1ull))) & 511;
                const unsigned v = *(const unsigned*)(h + 96 + idx * 4);
                o[hh][m * 2 + i1] = ((mk >> s) & 1ull) ? v : 0u;
              }
            }
          }
          st_sys(u4{o[0][0], o[0][1], o[0][2], o[0][3]}, ro, vo, ro_s);
          st_sys(u4{o[1][0], o[1][1], o[1][2], o[1][3]}, ro, vo, ro_s + 16384);
          at += nl;
        }
        CBAR();
        tried = got == avail ? got - at : 0;
        if (tried == 0 && at == 0) { abort = true; break; }
        rl += at;
        continue;
      }
      if (avail == 0 && (oc & 512) != 0) break;
      if (__builtin_amdgcn_readlane(fv, 2) == call || __builtin_amdgcn_s_memrealtime() - t0 > DEADLINE) { abort = true; break; }
      if (oc == 0) __builtin_amdgcn_s_sleep(16); else __builtin_amdgcn_s_sleep(2);
    }
    if (abort) {
      st_sys32(call, rws, v0, OFF_TMO);
      ppend |= 1u;
      WAITV0();
      while (ppend != 0u) {
        const int ro_s = __builtin_amdgcn_readlane(prow, __builtin_ctz(ppend));
        ppend &= ppend - 1u;
        const u4 nan4 = u4{0x7fc00000u, 0x7fc00000u, 0x7fc00000u, 0x7fc00000u};
        st_sys(nan4, ro, vo, ro_s);
        st_sys(nan4, ro, vo, ro_s + 16384);
      }
    }
    STAMP(12);
    return;
  }

  int sd, sb;
  const bool sused = slot_block(lane, g, helper, q, sd, sb);
  const unsigned needmask = (unsigned)__ballot(sused);
  const int nneed = __builtin_popcount(needmask);

  if (wid >= 4) {
    if (wid == 4) lds_st(ctl + lane, 0u);
    const unsigned call = __builtin_amdgcn_readfirstlane(cprev) + 1u;
    asm volatile("s_waitcnt lgkmcnt(0)" :: "s"(call) : "memory");
    __builtin_amdgcn_s_barrier();
    CBAR();
    __builtin_amdgcn_s_setprio(2);
    const int npack = !helper ? 0 : nneed > RAWTAIL ? nneed - RAWTAIL : 0;

    if (wid == 4) {
      const unsigned want = needmask | (15u << F_W) | (helper ? 1u << F_GATE : 0u);
      const int pv = lane < NSLOT ? (int)(OFF_SL + ((cu >> 2) * 8 + g) * 128 + lane * 4)
                   : lane < 32 ? (int)(OFF_PL + cidx * 128 + lane * 4) : lane == 32 ? (int)OFF_TMO : OOB;
      const unsigned boff = OFF_X + (call & 1u) * (32u << 20) + sd * (4 << 20) + sb * 131072;
      unsigned seen = 0;
      const u64 t0 = __builtin_amdgcn_s_memrealtime();
      for (;;) {
        const unsigned fv = ld_sys32(rws, pv, 0);
        const unsigned m = (unsigned)__ballot((int)(fv - call) >= 0) & want;
        if (m != seen) {
          const unsigned fresh = m & ~seen & needmask;
          if (lane < 32 && ((fresh >> lane) & 1u)) {
            const int pos = __builtin_popcount(seen & needmask) + __builtin_popcount(fresh >> lane) - 1;
            lds_st(ctl + C_QSLOT + pos, (unsigned)lane);
            lds_st(ctl + C_QOFF + pos, boff);
          }
          if (lane == 0) lds_st(ctl + C_MASK, m);
          seen = m;
          if (m == want) break;
        }
        if (__builtin_amdgcn_readlane(fv, 32) == call || __builtin_amdgcn_s_memrealtime() - t0 > DEADLINE) {
          st_sys32(call, rws, v0, OFF_TMO);
          if (lane == 0) lds_st(ctl + C_MASK, seen | (1u << F_ABORT));
          break;
        }
        __builtin_amdgcn_s_sleep(1);
      }
      STAMP(4);
    } else if (wid == 5) {
      unsigned pending = needmask;
      const int rowoff = (((p.rank ^ sd) & 7) * 512 + sb * 16) * 2048;
      const int vo = (lane >> 3) * 2048 + (lane & 7) * 16 + (cu & 15) * 128;
      const rsrc_t ro = mk_rsrc(!helper ? p.out[0] : q == 0 ? p.out[4] : q == 1 ? p.out[5] : q == 2 ? p.out[6] : p.out[7], 4096u * 512u * 4u);
      const int e = q * 28 + (cu & 3) * NG + g;
      const rsrc_t rq = mk_rsrc(q == 0 ? p.ws[4] : q == 1 ? p.ws[5] : q == 2 ? p.ws[6] : p.ws[7], WS_USED);
      const int fo = OFF_RP + e * 128;
      const int tb = OFF_T + e * TBYTES;
      const int nb = helper ? NBACK : NBOWN;
      int bi = 0, wl = 0, told = 0;
      bool abort = false;
      st_sys32(0u, rq, OOB, 0);
      for (int n = 0; n < nneed;) {
        if (helper) asm volatile("s_waitcnt vmcnt(1)" ::: "memory");
        const int arrived = wl;
        unsigned c;
        int red;
        for (;;) {
          c = lds_ld(ctl + lane);
          const unsigned mk = __builtin_amdgcn_readlane(c, C_MASK);
          red = __builtin_amdgcn_readlane(c, C_RED);
          abort = (mk >> F_ABORT) & 1u;
          if (abort || (red > n && (!helper || ((mk >> F_GATE) & 1u)))) break;
          if (told != arrived) { st_sys32((call << 14) | (unsigned)arrived, rq, v0, fo); told = arrived; }
          __builtin_amdgcn_s_sleep(1);
        }
        if (abort) break;
        CBAR();
        const int nend = red < n + CFG_FWIN ? red : n + CFG_FWIN;
        for (; n < nend; ++n) {
          const int sl = __builtin_amdgcn_readlane(c, C_QSLOT + n);
          pending &= ~(1u << sl);
          const char* eb = smem + L_BACK + bi * BPITCH;
          const u4 o0 = *(const u4*)(eb + vl);
          const u4 o1 = *(const u4*)(eb + 1024 + vl);
          if (n >= npack) {
            const int ro_s = __builtin_amdgcn_readlane(rowoff, sl);
            st_sys(o0, ro, vo, ro_s);
            st_sys(o1, ro, vo, ro_s + 16384);
          } else {
            const u4 o2 = *(const u4*)(eb + 2048 + (lane & 7) * 16);
            const int nl = (96 + 127 + 4 * __builtin_amdgcn_readlane(o0.y, 5)) >> 7;
            st_sys(o0, rq, lane < 8 * nl ? vl : OOB, tb + wl * 128);
            if (nl > 8) st_sys(o1, rq, lane + 64 < 8 * nl ? vl : OOB, tb + wl * 128 + 1024);
            if (nl > 16) st_sys(o2, rq, lane < 8 ? vl : OOB, tb + wl * 128 + 2048);
            wl += nl;
          }
          bi = bi + 1 == nb ? 0 : bi + 1;
        }
        CBAR();
        if (helper) { st_sys32((call << 14) | (unsigned)arrived, rq, told != arrived ? v0 : OOB, fo); told = arrived; }
        if (lane == 0) lds_st(ctl + C_FL, (unsigned)n);
      }
      if (helper || abort) WAITV0();
      if (abort) {
        pending |= 1u;
        while (pending != 0u) {
          const int ro_s = __builtin_amdgcn_readlane(rowoff, __builtin_ctz(pending));
          pending &= pending - 1u;
          const u4 nan4 = u4{0x7fc00000u, 0x7fc00000u, 0x7fc00000u, 0x7fc00000u};
          st_sys(nan4, ro, vo, ro_s);
          st_sys(nan4, ro, vo, ro_s + 16384);
        }
      } else if (helper) {
        st_sys32((call << 14) | 512u | (unsigned)wl, rq, v0, fo);
      }
      STAMP(5);
    } else {
      st_sys32(call, rws, v0, OFF_CTR + blockIdx.x * 64);
      STAMP(6);
      const int wo = (lane >> 4) * 512 + (lane & 7) * 16 + ((lane >> 3) & 1) * 4;
      const int nb = helper ? NBACK : NBOWN;
      int bi = 0;
      for (int n = 0; n < nneed; ++n) {
        bool abort;
        unsigned c;
        for (;;) {
          c = lds_ld(ctl + lane);
          abort = (__builtin_amdgcn_readlane(c, C_MASK) >> F_ABORT) & 1u;
          const int pc = (n & 1) ? __builtin_amdgcn_readlane(c, C_PCNT + 1) : __builtin_amdgcn_readlane(c, C_PCNT);
          if (abort || (pc >= 4 * ((n >> 1) + 1) && n - (int)__builtin_amdgcn_readlane(c, C_FL) < nb)) break;
          __builtin_amdgcn_s_sleep(1);
        }
        if (abort) break;
        CBAR();
        const char* pp = smem + L_PART + (n & 1) * 8192 + vl;
        char* e = smem + L_BACK + bi * BPITCH;
        float v[8];
#pragma unroll
        for (int m = 0; m < 2; ++m) {
          const f4 s = (*(const f4*)(pp + m * 1024) + *(const f4*)(pp + 2048 + m * 1024)) +
                       (*(const f4*)(pp + 4096 + m * 1024) + *(const f4*)(pp + 6144 + m * 1024));
#pragma unroll
          for (int jj = 0; jj < 4; ++jj) v[m * 4 + jj] = s[jj] < 0.f ? 0.f : s[jj];
        }
        if (n >= npack) {
#pragma unroll
          for (int r = 0; r < 8; ++r) *(float*)(e + wo + (r & 3) * 128 + (r >> 2) * 8) = v[r];
        } else {
          int base = 0, hb = 0;
          u64 hm = 0;
#pragma unroll
          for (int r = 0; r < 8; ++r) {
            const bool nz = __builtin_bit_cast(unsigned, v[r]) != 0u;
            const u64 mk = __ballot(nz);
            const int pos = base + __builtin_amdgcn_mbcnt_hi((unsigned)(mk >> 32), __builtin_amdgcn_mbcnt_lo((unsigned)mk, 0u));
            if (nz) *(float*)(e + 96 + pos * 4) = v[r];
            if (lane == r) { hm = mk; hb = base; }
            base += __builtin_popcountll(mk);
          }
          if (lane < 8) { *(u64*)(e + lane * 8) = hm; *(unsigned short*)(e + 64 + lane * 2) = (unsigned short)hb; }
          if (lane < 2) *(unsigned*)(e + 80 + lane * 4) = lane == 0 ? (unsigned)__builtin_amdgcn_readlane(c, C_QSLOT + n) : (unsigned)base;
        }
        CBAR();
        if (lane == 0) lds_st(ctl + C_RED, (unsigned)(n + 1));
        bi = bi + 1 == nb ? 0 : bi + 1;
      }
    }
    return;
  }

  {
    const unsigned call = __builtin_amdgcn_readfirstlane(cprev) + 1u;
    asm volatile("" :: "s"(call) : "memory");
    __builtin_amdgcn_s_barrier();
    CBAR();
    __builtin_amdgcn_s_sleep(CFG_STAGE_DELAY);
    stage_local(cidx * 4 + wid, call, rws, rx, rw, rcnt, vx, vw, lane);
    const int wbase = OFF_W + (call & 1u) * (8u << 20);

    for (;;) {
      const unsigned mk = __builtin_amdgcn_readfirstlane(lds_ld(ctl + C_MASK));
      if ((mk >> F_ABORT) & 1u) return;
      if ((mk >> (F_W + wid)) & 1u) break;
      __builtin_amdgcn_s_sleep(4);
    }
    CBAR();
    int qo[8];
#pragma unroll
    for (int qd = 0; qd < 8; ++qd) qo[qd] = ((qd + cu) & 7) * 4096;
    const int vb = ((lane >> 3) & 1) * 1024 + (lane >> 4) * 128 + (lane & 7) * 16;
    u4 breg[18][2];
    char* ldsb = smem + L_B + wid * (NLDSK * 2048) + vl;
#pragma unroll
    for (int hf = 0; hf < 2; ++hf) {
      const int so = wbase + cu * 262144 + wid * 65536;
      u4 tmp[7][2];
#pragma unroll
      for (int t = 0; t < 16; ++t) {
#pragma unroll
        for (int m = 0; m < 2; ++m) {
          const u4 v = ld_sys(rws, vb + m * 2048 + (t & 1) * 512, so + 2 * qo[hf * 4 + (t >> 2)] + ((t >> 1) & 1) * 4096);
          if (k_in_lds(t)) tmp[t >> 1][m] = v; else breg[reg_index(hf * 16 + t)][m] = v;
        }
      }
#pragma unroll
      for (int t = 0; t < 7; ++t) {
#pragma unroll
        for (int m = 0; m < 2; ++m) *(u4*)(ldsb + (hf * 7 + t) * 2048 + m * 1024) = tmp[t][m];
      }
    }

    u4 ring[16];
    u4 bt[2][2];
#pragma unroll
    for (int n = 0; n < 2; ++n) { bt[n][0] = *(const u4*)(ldsb + n * 2048); bt[n][1] = *(const u4*)(ldsb + n * 2048 + 1024); }
    int va = vl + wid * 32768;
    int cur = (int)WS_USED;
    for (int n = 0; n < nneed; ++n) {
      if (cur == (int)WS_USED) {
        for (;;) {
          cur = __builtin_amdgcn_readfirstlane(lds_ld(ctl + C_QOFF + n));
          if (cur != 0) break;
          if ((__builtin_amdgcn_readfirstlane(lds_ld(ctl + C_MASK)) >> F_ABORT) & 1u) return;
          __builtin_amdgcn_s_sleep(1);
        }
        asm volatile("" : "+v"(va) :: "memory");
#pragma unroll
        for (int t = 0; t < 16; ++t) {
          ring[t] = ld_sys(rws, va + (t & 3) * 1024, cur + qo[t >> 2]);
          __builtin_amdgcn_sched_barrier(0);
        }
      }
      f4 acc0 = f4{0.f, 0.f, 0.f, 0.f}, acc1 = f4{0.f, 0.f, 0.f, 0.f};
      int nb = 0;
      unsigned r0 = 0;
      asm volatile("" : "+v"(va));
#pragma unroll
      for (int t = 0; t < 32; ++t) {
        if (t == 16) { nb = __builtin_amdgcn_readfirstlane(r0); nb = nb != 0 ? nb : (int)WS_USED; CBAR(); }
        const u4 b0 = k_in_lds(t) ? bt[lds_index(t) & 1][0] : breg[reg_index(t)][0];
        const u4 b1 = k_in_lds(t) ? bt[lds_index(t) & 1][1] : breg[reg_index(t)][1];
        const h8 a = __builtin_bit_cast(h8, ring[t & 15]);
        acc0 = __builtin_amdgcn_mfma_f32_16x16x32_f16(a, __builtin_bit_cast(h8, b0), acc0, 0, 0, 0);
        acc1 = __builtin_amdgcn_mfma_f32_16x16x32_f16(a, __builtin_bit_cast(h8, b1), acc1, 0, 0, 0);
        __builtin_amdgcn_sched_barrier(0);
        ring[t & 15] = ld_sys(rws, va + (t & 3) * 1024, (t < 16 ? cur : nb) + qo[((t + 16) & 31) >> 2]);
        if (k_in_lds(t)) {
          const int nn = (lds_index(t) + 2) % NLDSK;
          bt[nn & 1][0] = *(const u4*)(ldsb + nn * 2048);
          bt[nn & 1][1] = *(const u4*)(ldsb + nn * 2048 + 1024);
        }
        if (t == 14) r0 = lds_ld(ctl + C_QOFF + n + 1);
        if (t == 30) r0 = lds_ld(ctl + C_RED);
        __builtin_amdgcn_sched_barrier(0);
      }
      int red = __builtin_amdgcn_readfirstlane(r0);
      while (red < n - 1) {
        __builtin_amdgcn_s_sleep(1);
        if ((__builtin_amdgcn_readfirstlane(lds_ld(ctl + C_MASK)) >> F_ABORT) & 1u) return;
        red = __builtin_amdgcn_readfirstlane(lds_ld(ctl + C_RED));
      }
      CBAR();
      char* pp = smem + L_PART + (n & 1) * 8192 + wid * 2048 + vl;
      *(f4*)pp = acc0;
      *(f4*)(pp + 1024) = acc1;
      CBAR();
      if (lane == 0) __hip_atomic_fetch_add(ctl + C_PCNT + (n & 1), 1u, __ATOMIC_RELAXED, __HIP_MEMORY_SCOPE_WORKGROUP);
      CBAR();
      cur = nb;
    }
  }
}

}

extern "C" void kernel_launch(int rank, int nranks, ncclComm_t comm,
                              hipStream_t stream,
                              void* const* d_in, const int* in_sizes, int n_in,
                              void* d_out, int out_size,
                              const int* in_dtypes, int out_dtype,
                              void* const* d_out_all,
                              void* const* d_ws_all, size_t ws_bytes) {
  if (nranks != NR || ws_bytes < (size_t)WS_USED + (1u << 20) || n_in < 2 || (unsigned)rank >= (unsigned)NR) {
    fprintf(stderr, "unsupported configuration\n");
    return;
  }
  Params p;
  p.x = (const float*)d_in[0];
  p.w = (const float*)d_in[1];
  for (int d = 0; d < NR; ++d) {
    p.ws[d] = (char*)d_ws_all[rank ^ d];
    p.out[d] = (float*)d_out_all[rank ^ d];
  }
  p.rank = rank;
  hipLaunchKernelGGL(ag_gemm_relu, dim3(256), dim3(512), 0, stream, p);
  HIP_CHECK(hipGetLastError());
}
```

```cpp
#include <hip/hip_runtime.h>
#include <rccl/rccl.h>
#include <cstdio>

#define HIP_CHECK(expr) do { hipError_t _e = (expr); if (_e != hipSuccess) { \
    fprintf(stderr, "HIP error %s at %s:%d\n", hipGetErrorString(_e), __FILE__, __LINE__); \
    return; } } while (0)

namespace {

typedef float f4 __attribute__((ext_vector_type(4)));
typedef unsigned u4 __attribute__((ext_vector_type(4)));
typedef _Float16 h8 __attribute__((ext_vector_type(8)));
typedef unsigned long long u64;
typedef __amdgpu_buffer_rsrc_t rsrc_t;

constexpr int NR = 8;
constexpr int NG = 7;
constexpr int ND = 22;
constexpr int NXB = 32 - ND;
constexpr int S1 = 32;
constexpr int S3 = 18;
constexpr int ONE = 96;
constexpr int S0 = S1 + ONE;
constexpr int NSTEP = ND * 32;
constexpr int NWST = 256;
constexpr int E2 = NWST + NXB * 32;
constexpr int GATE_STEP = E2 - 12;
constexpr int QUIT_STEP = E2 - 16;
constexpr int FWIN = 2;
constexpr int SLP_STAGE = 40, SLP_STIER = 14, SLP_TIER = 48;
constexpr int RAWTAIL = 3;
static_assert(ND >= 20 && ND <= 24, "slot table");
static_assert(S3 > 0 && S3 <= S1 && ONE <= 112 && S0 <= NWST && GATE_STEP >= 0 && GATE_STEP < NSTEP, "sender configuration");

constexpr u64 DEADLINE = 100000000ull;

constexpr unsigned OFF_PL = 0;
constexpr unsigned OFF_TMO = 40960;
constexpr unsigned OFF_CNT = 45056;
constexpr unsigned OFF_CTR = 65536;
constexpr unsigned OFF_SL = 98304;
constexpr unsigned OFF_RP = 110592;
constexpr unsigned OFF_X = 1u << 20;
constexpr unsigned OFF_W = 65u << 20;
constexpr unsigned OFF_T = 81u << 20;
constexpr unsigned WS_USED = 87u << 20;
constexpr int TBYTES = 49152;
static_assert(112u * TBYTES <= WS_USED - OFF_T && 22 * 2176 <= TBYTES, "tile region");
constexpr int CI_TICK = 0;
constexpr int CI_SX = 4;
constexpr int CI_SW = 36;
constexpr int CI_LX = 52;
constexpr int CI_LW = 84;
constexpr int CI_END = 148;
constexpr int NSLOT = 25;
constexpr int F_W = 25;
constexpr int F_GATE = 29;
constexpr int F_ABORT = 31;

constexpr int NLDSK = 14;
constexpr int NBACK = 15;
constexpr int NBOWN = 8;
constexpr int BPITCH = 2144;
constexpr int L_B = 0;
constexpr int L_PART = 4 * NLDSK * 2048;
constexpr int L_BACK = L_PART + 16384;
constexpr int L_CTL = L_BACK + NBACK * BPITCH;
constexpr int C_QOFF = 0;
constexpr int C_QSLOT = 24;
constexpr int C_PCNT = 48;
constexpr int C_RED = 50;
constexpr int C_FL = 51;
constexpr int C_MASK = 52;
constexpr int C_WORDS = 64;
constexpr int L_EXP = L_BACK + NBOWN * BPITCH;
constexpr int LDS_BYTES = L_CTL + C_WORDS * 4;
static_assert(LDS_BYTES <= 163840, "LDS");

struct Params {
  const float* x;
  const float* w;
  char* ws[NR];
  float* out[NR];
  int rank;
};

#define WAITV0() do { asm volatile("" ::: "memory"); __builtin_amdgcn_s_waitcnt(0x0F70); asm volatile("" ::: "memory"); } while (0)
#define CBAR() asm volatile("" ::: "memory")
constexpr int OOB = 0x40000000;

__device__ __forceinline__ rsrc_t mk_rsrc(const void* p, unsigned bytes) {
  return __builtin_amdgcn_make_buffer_rsrc(const_cast<void*>(p), 0, bytes, 0x00020000);
}
__device__ __forceinline__ f4 ld_in(rsrc_t r, int voff, int soff) {
  return __builtin_bit_cast(f4, __builtin_amdgcn_raw_buffer_load_b128(r, voff, soff, 0));
}
__device__ __forceinline__ u4 ld_sys(rsrc_t r, int voff, int soff) {
  return __builtin_amdgcn_raw_buffer_load_b128(r, voff, soff, 17);
}
__device__ __forceinline__ unsigned ld_sys32(rsrc_t r, int voff, int soff) {
  return __builtin_amdgcn_raw_buffer_load_b32(r, voff, soff, 17);
}
__device__ __forceinline__ void st_sys(u4 v, rsrc_t r, int voff, int soff) {
  __builtin_amdgcn_raw_buffer_store_b128(v, r, voff, soff, 17);
}
__device__ __forceinline__ void st_sys32(unsigned v, rsrc_t r, int voff, int soff) {
  __builtin_amdgcn_raw_buffer_store_b32(v, r, voff, soff, 17);
}
__device__ __forceinline__ u4 cvt8(f4 a, f4 b) {
  h8 h = h8{(_Float16)a.x, (_Float16)a.y, (_Float16)a.z, (_Float16)a.w,
            (_Float16)b.x, (_Float16)b.y, (_Float16)b.z, (_Float16)b.w};
  return __builtin_bit_cast(u4, h);
}
__device__ __forceinline__ u4 cvtcol(const f4* r, int i) {
  h8 h = h8{(_Float16)r[0][i], (_Float16)r[1][i], (_Float16)r[2][i], (_Float16)r[3][i],
            (_Float16)r[4][i], (_Float16)r[5][i], (_Float16)r[6][i], (_Float16)r[7][i]};
  return __builtin_bit_cast(u4, h);
}
__device__ __forceinline__ void load_xpiece(f4* r, rsrc_t rx, int vx, int pi) {
  const int so = (pi >> 5) * (16 * 16384) + (pi & 31) * 512;
#pragma unroll
  for (int t = 0; t < 8; ++t) r[t] = ld_in(rx, vx + t * 64, so);
}
__device__ __forceinline__ void load_wgran(f4* r, rsrc_t rw, int vw, int cg, int kg) {
  const int so = kg * (64 * 2048) + cg * 128;
#pragma unroll
  for (int j = 0; j < 8; ++j) r[j] = ld_in(rw, vw + (j & 1) * 2048, so + (16 * (j >> 2) + (j & 2)) * 2048);
}
__device__ __forceinline__ unsigned lds_ld(const unsigned* p) {
  return __hip_atomic_load(p, __ATOMIC_RELAXED, __HIP_MEMORY_SCOPE_WORKGROUP);
}
__device__ __forceinline__ void lds_st(unsigned* p, unsigned v) {
  __hip_atomic_store(p, v, __ATOMIC_RELAXED, __HIP_MEMORY_SCOPE_WORKGROUP);
}

__device__ __forceinline__ int src_off(int d) { return (0x65314201 >> (4 * d)) & 7; }
__device__ __forceinline__ int src_sbase(int d) { return d == 0 ? 0 : d < 4 ? 1 + 4 * d : 9 + 2 * d; }
__device__ __forceinline__ int src_lo(int d) { return d < 4 ? 0 : ND; }
__device__ __forceinline__ int flag_off(int d, int b) {
  return (((b + src_off(d)) % NG) * 32 + src_sbase(d) + (b - src_lo(d)) / NG) * 4;
}
__device__ __forceinline__ bool slot_block(int slot, int g, bool helper, int q, int& d, int& b) {
  d = slot < 5 ? 0 : slot < 17 ? (slot - 1) >> 2 : ((slot - 9) >> 1) & 7;
  const int lo = src_lo(d);
  const int hi = d == 0 ? (helper ? ND : 32) : d < 4 ? ND : ((helper && d == 4 + q) ? 0 : 32);
  b = lo + (g + 70 - src_off(d) - lo) % NG + NG * (slot - src_sbase(d));
  return slot < NSLOT && b < hi;
}
__device__ __forceinline__ void stage_local(int gi, unsigned call, rsrc_t rws, rsrc_t rx, rsrc_t rw, rsrc_t rcnt, int vx, int vw, int lane) {
  f4 sx[8], sw[8];
  load_xpiece(sx, rx, vx, gi);
  load_wgran(sw, rw, vw, gi >> 6, gi & 63);
  const int xbase = OFF_X + (call & 1u) * (32u << 20);
  const int wbase = OFF_W + (call & 1u) * (8u << 20);
#pragma unroll
  for (int t = 0; t < 4; ++t) st_sys(cvtcol(sw, t), rws, lane * 16 + t * 1024, wbase + gi * 4096);
#pragma unroll
  for (int t = 0; t < 4; ++t) st_sys(cvt8(sx[2 * t], sx[2 * t + 1]), rws, lane * 16 + t * 1024, xbase + gi * 4096);
  WAITV0();
  const int tv = __builtin_amdgcn_raw_ptr_buffer_atomic_add_i32(1, rcnt, (lane == 0 ? CI_LX + (gi >> 5) : lane == 1 ? CI_LW + (gi >> 4) : CI_END) * 64, 0, 0);
  if ((__builtin_amdgcn_readlane(tv, 0) & 31) == 31)
    st_sys32(call, rws, lane < 8 ? lane * 1024 : OOB, OFF_SL + flag_off(0, gi >> 5));
  if ((__builtin_amdgcn_readlane(tv, 1) & 15) == 15)
    st_sys32(call, rws, lane < NG ? lane * 4096 : OOB, OFF_PL + (gi >> 6) * 128 + (F_W + ((gi >> 4) & 3)) * 4);
}

constexpr bool k_in_lds(int t) { return ((t & 15) & 1) && (t & 15) != 15; }
constexpr int lds_index(int t) { return (t >> 4) * 7 + ((t & 15) >> 1); }
constexpr int reg_index(int t) { return (t >> 4) * 9 + ((t & 15) == 15 ? 8 : ((t & 15) >> 1)); }

__global__ void __launch_bounds__(512, 2) ag_gemm_relu(Params p) {
  __shared__ __attribute__((aligned(16))) char smem[LDS_BYTES];
  unsigned* ctl = (unsigned*)(smem + L_CTL);
  const int lane = threadIdx.x & 63;
  const int wid = __builtin_amdgcn_readfirstlane(threadIdx.x >> 6);
  const bool comm = blockIdx.x < 32;
  const int cidx = (blockIdx.x & 7) * 28 + (blockIdx.x >> 3) - 4;
  const int g = cidx >> 5;
  const int cu = cidx & 31;
  const bool helper = cu >= 16;
  const int q = (cu >> 2) & 3;
  const rsrc_t rws = mk_rsrc(p.ws[0], WS_USED);
  const rsrc_t rx = mk_rsrc(p.x, 512u * 4096u * 4u);
  const rsrc_t rw = mk_rsrc(p.w, 4096u * 512u * 4u);
  const rsrc_t rcnt = mk_rsrc(p.ws[0] + OFF_CNT, CI_END * 64);
  const int vx = (lane & 15) * 16384 + (lane >> 4) * 16;
  const int vw = (32 * (lane >> 5) + 4 * ((lane >> 3) & 3)) * 2048 + (lane & 7) * 16;
  const int vl = lane * 16;
  const int v0 = lane == 0 ? 0 : OOB;

  const unsigned cprev = ld_sys32(rws, 0, OFF_CTR + blockIdx.x * 64);

  const int j = comm ? (int)blockIdx.x : S1 + ((cu + 16) & 31) * NG + g;
  if (__builtin_expect(comm ? wid == 0 : wid == 6 && j < S0, 1)) {
    f4 rd[8], ra[8], rb[24];
    load_xpiece(rd, rx, vx, j);
    load_wgran(ra, rw, vw, j & 3, j >> 2);
#pragma unroll
    for (int t = 0; t < 3; ++t) load_wgran(rb + 8 * t, rw, vw, 4 * (t + 1) + (j & 3), j >> 2);
    asm volatile("" :: "s"(p.ws[1]), "s"(p.ws[2]), "s"(p.ws[3]), "s"(p.ws[4]), "s"(p.ws[5]), "s"(p.ws[6]), "s"(p.ws[7]));
    const unsigned call = __builtin_amdgcn_readfirstlane(cprev) + 1u;
    asm volatile("" :: "s"(call) : "memory");
    __builtin_amdgcn_s_barrier();
    CBAR();
    __builtin_amdgcn_s_setprio(2);
    const unsigned par = call & 1u;
    rsrc_t rp[NR];
#pragma unroll
    for (int d = 1; d < NR; ++d) rp[d] = mk_rsrc(p.ws[d], WS_USED);
    const int xbase = OFF_X + par * (32u << 20);
    const int wbase = OFF_W + par * (8u << 20);
    const int vfl = lane < 8 ? lane * 1024 : OOB;
    const int vgl = lane < NG ? lane * 4096 : OOB;
    const int qstep = !comm ? 0 : j >= S3 ? QUIT_STEP : NSTEP;
    int i = j, prev = -1;
    WAITV0();
    for (int z = (j - S1) >> 3; z > 0; --z) __builtin_amdgcn_s_sleep(SLP_TIER);
    for (;;) {
      const bool quit = i >= qstep;
      int cv = CI_END;
      if (lane == 0 && !quit) cv = CI_TICK + 2 * par;
      if (lane == 1 && prev >= 0) cv = CI_SX + (prev >> 5);
      if (lane == 2 && prev >= 0 && prev < E2) cv = prev < NWST ? CI_SW + (prev >> 6) * 4 + (prev & 3) : CI_SX + ND + ((prev - NWST) >> 5);
      const int tv = __builtin_amdgcn_raw_ptr_buffer_atomic_add_i32(1, rcnt, cv * 64, 0, 0);
      int t0v, t1v, t2v;
#define READ_BACK() do { t0v = __builtin_amdgcn_readlane(tv, 0); t1v = __builtin_amdgcn_readlane(tv, 1); t2v = __builtin_amdgcn_readlane(tv, 2); } while (0)
      if (i == GATE_STEP) {
        st_sys32(call, rws, (lane >> 4) * 4096 + (lane & 15) * 128, OFF_PL + 16 * 128 + F_GATE * 4);
        st_sys32(call, rws, lane < 48 ? (lane >> 4) * 4096 + (lane & 15) * 128 : OOB, OFF_PL + 4 * 4096 + 16 * 128 + F_GATE * 4);
      }
      if (i >= NSTEP) {
        READ_BACK();
      } else if (i < NWST) {
        const int wo = wbase + (16 + (i & 3)) * 262144 + (i >> 2) * 4096;
#pragma unroll
        for (int t = 0; t < 4; ++t) {
          const u4 h = cvt8(rd[2 * t], rd[2 * t + 1]);
#pragma unroll
          for (int d = 1; d < 4; ++d) st_sys(h, rp[d], vl + t * 1024, xbase + d * (4 << 20) + i * 4096);
          st_sys(cvtcol(ra, t), rp[4], vl + t * 1024, wo);
#pragma unroll
          for (int d = 5; d < 8; ++d) st_sys(cvtcol(rb + 8 * (d - 5), t), rp[d], vl + t * 1024, wo + (d - 4) * (4 * 262144));
        }
        READ_BACK();
      } else if (i < E2) {
        const int xi = NSTEP + i - NWST;
#pragma unroll
        for (int t = 0; t < 4; ++t) {
          const u4 h = cvt8(rd[2 * t], rd[2 * t + 1]);
          const u4 gg = cvt8(ra[2 * t], ra[2 * t + 1]);
#pragma unroll
          for (int d = 1; d < 4; ++d) st_sys(h, rp[d], vl + t * 1024, xbase + d * (4 << 20) + i * 4096);
#pragma unroll
          for (int d = 4; d < 8; ++d) st_sys(gg, rp[d], vl + t * 1024, xbase + d * (4 << 20) + xi * 4096);
        }
        READ_BACK();
      } else {
#pragma unroll
        for (int t = 0; t < 4; ++t) {
          const u4 h = cvt8(rd[2 * t], rd[2 * t + 1]);
#pragma unroll
          for (int d = 1; d < 4; ++d) st_sys(h, rp[d], vl + t * 1024, xbase + d * (4 << 20) + i * 4096);
        }
        READ_BACK();
      }
      if (prev >= 0) {
        if ((t1v & 31) == 31) {
#pragma unroll
          for (int d = 1; d < 4; ++d) st_sys32(call, rp[d], vfl, OFF_SL + flag_off(d, prev >> 5));
        }
        if (prev < NWST) {
          if ((t2v & 15) == 15) {
#pragma unroll
            for (int d = 4; d < 8; ++d)
              st_sys32(call, rp[d], vgl, OFF_PL + (16 + 4 * (d - 4) + (prev & 3)) * 128 + (F_W + (prev >> 6)) * 4);
          }
        } else if (prev < E2) {
          if ((t2v & 31) == 31) {
#pragma unroll
            for (int d = 4; d < 8; ++d) st_sys32(call, rp[d], vfl, OFF_SL + flag_off(d, ND + ((prev - NWST) >> 5)));
          }
        }
      }
      if (i >= NSTEP) break;
      int nxt = t0v + S0;
      if (quit || (unsigned)nxt >= (unsigned)NSTEP) nxt = NSTEP;
      if (nxt < NSTEP) {
        load_xpiece(rd, rx, vx, nxt);
        if (nxt < NWST) {
          load_wgran(ra, rw, vw, nxt & 3, nxt >> 2);
#pragma unroll
          for (int t = 0; t < 3; ++t) load_wgran(rb + 8 * t, rw, vw, 4 * (t + 1) + (nxt & 3), nxt >> 2);
        } else if (nxt < E2) {
          load_xpiece(ra, rx, vx, NSTEP + nxt - NWST);
        }
      }
      WAITV0();
      prev = i;
      i = nxt;
    }
    return;
  }

  if (comm || wid == 6) {
    const unsigned call = __builtin_amdgcn_readfirstlane(cprev) + 1u;
    asm volatile("" :: "s"(call) : "memory");
    __builtin_amdgcn_s_barrier();
    CBAR();
    if (comm) {
      if (wid == 7) {
        st_sys32(call, rws, v0, OFF_CTR + blockIdx.x * 64);
        if (blockIdx.x == 31) st_sys32(0u, rcnt, v0, (CI_TICK + 2 * ((call & 1u) ^ 1u)) * 64);
      }
      return;
    }
    __builtin_amdgcn_s_sleep(SLP_STAGE);
    for (int z = (blockIdx.x >> 3) - 4; z > 0; --z) __builtin_amdgcn_s_sleep(SLP_STIER);
    stage_local(896 + j - S0, call, rws, rx, rw, rcnt, vx, vw, lane);
    if (helper) return;
    __builtin_amdgcn_s_setprio(2);
    const int z = q, sub = cu & 3, eg = g, e = z * 28 + sub * NG + g;
    int pd, pb;
    unsigned ppend = (unsigned)__ballot(slot_block(lane, eg, true, z, pd, pb));
    const int prow = (((p.rank ^ (4 + z) ^ pd) & 7) * 512 + pb * 16) * 2048;
    const rsrc_t ro = mk_rsrc(p.out[0], 4096u * 512u * 4u);
    const int vo = (lane >> 3) * 2048 + (lane & 7) * 16 + (4 * z + sub) * 128;
    const int tb = OFF_T + e * TBYTES;
    char* sc = smem + L_EXP;
    const int jr = (lane >> 3) & 3;
    const int sl0 = (lane >> 5) * 16 + (lane & 7);
    const int pv = lane == 0 ? (int)(OFF_RP + e * 128) : lane == 2 ? (int)OFF_TMO : OOB;
    int rl = 0, tried = 0, oc = 0;
    bool abort = false;
    const u64 t0 = __builtin_amdgcn_s_memrealtime();
    for (;;) {
      const unsigned fv = ld_sys32(rws, pv, 0);
      const unsigned ow = __builtin_amdgcn_readlane(fv, 0) - (call << 14);
      if (ow < 1024u && (int)ow > oc) oc = (int)ow;
      const int avail = (oc & 511) - rl;
      if (avail > tried) {
        const int got = avail < 64 ? avail : 64;
        u4 t[8];
#pragma unroll
        for (int y = 0; y < 8; ++y) t[y] = ld_sys(rws, (lane >> 3) + 8 * y < got ? vl : OOB, tb + rl * 128 + y * 1024);
#pragma unroll
        for (int y = 0; y < 8; ++y) *(u4*)(sc + y * 1024 + vl) = t[y];
        CBAR();
        int at = 0;
        while (at < got) {
          const char* h = sc + at * 128;
          const unsigned hd = *(const unsigned*)(h + 80 + (lane & 1) * 4);
          const u64 mk0 = *(const u64*)(h + jr * 8), mk1 = *(const u64*)(h + 32 + jr * 8);
          const int b0 = *(const unsigned short*)(h + 64 + jr * 2), b1 = *(const unsigned short*)(h + 72 + jr * 2);
          const int nl = (96 + 127 + 4 * (__builtin_amdgcn_readlane(hd, 1) & 1023)) >> 7;
          if (at + nl > got) break;
          const int sl = __builtin_amdgcn_readlane(hd, 0) & 31;
          const int ro_s = __builtin_amdgcn_readlane(prow, sl);
          ppend &= ~(1u << sl);
          unsigned o[2][4];
#pragma unroll
          for (int m = 0; m < 2; ++m) {
            const u64 mk = m ? mk1 : mk0;
            const int base = m ? b1 : b0;
#pragma unroll
            for (int hh = 0; hh < 2; ++hh) {
#pragma unroll
              for (int i1 = 0; i1 < 2; ++i1) {
                const int s = sl0 + hh * 32 + i1 * 8;
                const int idx = (base + __builtin_popcountll(mk & ((1ull << s) - 1ull))) & 511;
                const unsigned v = *(const unsigned*)(h + 96 + idx * 4);
                o[hh][m * 2 + i1] = ((mk >> s) & 1ull) ? v : 0u;
              }
            }
          }
          st_sys(u4{o[0][0], o[0][1], o[0][2], o[0][3]}, ro, vo, ro_s);
          st_sys(u4{o[1][0], o[1][1], o[1][2], o[1][3]}, ro, vo, ro_s + 16384);
          at += nl;
        }
        CBAR();
        tried = got == avail ? got - at : 0;
        if (tried == 0 && at == 0) { abort = true; break; }
        rl += at;
        continue;
      }
      if (avail == 0 && (oc & 512) != 0) break;
      if (__builtin_amdgcn_readlane(fv, 2) == call || __builtin_amdgcn_s_memrealtime() - t0 > DEADLINE) { abort = true; break; }
      if (oc == 0) __builtin_amdgcn_s_sleep(16); else __builtin_amdgcn_s_sleep(2);
    }
    if (abort) {
      st_sys32(call, rws, v0, OFF_TMO);
      ppend |= 1u;
      WAITV0();
      while (ppend != 0u) {
        const int ro_s = __builtin_amdgcn_readlane(prow, __builtin_ctz(ppend));
        ppend &= ppend - 1u;
        const u4 nan4 = u4{0x7fc00000u, 0x7fc00000u, 0x7fc00000u, 0x7fc00000u};
        st_sys(nan4, ro, vo, ro_s);
        st_sys(nan4, ro, vo, ro_s + 16384);
      }
    }
    return;
  }

  int sd, sb;
  const bool sused = slot_block(lane, g, helper, q, sd, sb);
  const unsigned needmask = (unsigned)__ballot(sused);
  const int nneed = __builtin_popcount(needmask);

  if (wid >= 4) {
    if (wid == 4) lds_st(ctl + lane, 0u);
    const unsigned call = __builtin_amdgcn_readfirstlane(cprev) + 1u;
    asm volatile("s_waitcnt lgkmcnt(0)" :: "s"(call) : "memory");
    __builtin_amdgcn_s_barrier();
    CBAR();
    __builtin_amdgcn_s_setprio(2);
    const int npack = !helper ? 0 : nneed > RAWTAIL ? nneed - RAWTAIL : 0;

    if (wid == 4) {
      const unsigned want = needmask | (15u << F_W) | (helper ? 1u << F_GATE : 0u);
      const int pv = lane < NSLOT ? (int)(OFF_SL + ((cu >> 2) * 8 + g) * 128 + lane * 4)
                   : lane < 32 ? (int)(OFF_PL + cidx * 128 + lane * 4) : lane == 32 ? (int)OFF_TMO : OOB;
      const unsigned boff = OFF_X + (call & 1u) * (32u << 20) + sd * (4 << 20) + sb * 131072;
      unsigned seen = 0;
      const u64 t0 = __builtin_amdgcn_s_memrealtime();
      for (;;) {
        const unsigned fv = ld_sys32(rws, pv, 0);
        const unsigned m = (unsigned)__ballot((int)(fv - call) >= 0) & want;
        if (m != seen) {
          const unsigned fresh = m & ~seen & needmask;
          if (lane < 32 && ((fresh >> lane) & 1u)) {
            const int pos = __builtin_popcount(seen & needmask) + __builtin_popcount(fresh >> lane) - 1;
            lds_st(ctl + C_QSLOT + pos, (unsigned)lane);
            lds_st(ctl + C_QOFF + pos, boff);
          }
          if (lane == 0) lds_st(ctl + C_MASK, m);
          seen = m;
          if (m == want) break;
        }
        if (__builtin_amdgcn_readlane(fv, 32) == call || __builtin_amdgcn_s_memrealtime() - t0 > DEADLINE) {
          st_sys32(call, rws, v0, OFF_TMO);
          if (lane == 0) lds_st(ctl + C_MASK, seen | (1u << F_ABORT));
          break;
        }
        __builtin_amdgcn_s_sleep(1);
      }
    } else if (wid == 5) {
      unsigned pending = needmask;
      const int rowoff = (((p.rank ^ sd) & 7) * 512 + sb * 16) * 2048;
      const int vo = (lane >> 3) * 2048 + (lane & 7) * 16 + (cu & 15) * 128;
      const rsrc_t ro = mk_rsrc(!helper ? p.out[0] : q == 0 ? p.out[4] : q == 1 ? p.out[5] : q == 2 ? p.out[6] : p.out[7], 4096u * 512u * 4u);
      const int e = q * 28 + (cu & 3) * NG + g;
      const rsrc_t rq = mk_rsrc(q == 0 ? p.ws[4] : q == 1 ? p.ws[5] : q == 2 ? p.ws[6] : p.ws[7], WS_USED);
      const int fo = OFF_RP + e * 128;
      const int tb = OFF_T + e * TBYTES;
      const int nb = helper ? NBACK : NBOWN;
      int bi = 0, wl = 0, told = 0;
      bool abort = false;
      st_sys32(0u, rq, OOB, 0);
      for (int n = 0; n < nneed;) {
        if (helper) asm volatile("s_waitcnt vmcnt(1)" ::: "memory");
        const int arrived = wl;
        unsigned c;
        int red;
        for (;;) {
          c = lds_ld(ctl + lane);
          const unsigned mk = __builtin_amdgcn_readlane(c, C_MASK);
          red = __builtin_amdgcn_readlane(c, C_RED);
          abort = (mk >> F_ABORT) & 1u;
          const bool hold = n >= npack && red < nneed && (mk & needmask) == needmask;
          if (abort || (red > n && (!helper || (((mk >> F_GATE) & 1u) && !hold)))) break;
          if (told != arrived) { st_sys32((call << 14) | (unsigned)arrived, rq, v0, fo); told = arrived; }
          __builtin_amdgcn_s_sleep(1);
        }
        if (abort) break;
        CBAR();
        const int nend = red < n + FWIN || n >= npack ? red : n + FWIN;
        for (; n < nend; ++n) {
          const int sl = __builtin_amdgcn_readlane(c, C_QSLOT + n);
          pending &= ~(1u << sl);
          const char* eb = smem + L_BACK + bi * BPITCH;
          const u4 o0 = *(const u4*)(eb + vl);
          const u4 o1 = *(const u4*)(eb + 1024 + vl);
          if (n >= npack) {
            const int ro_s = __builtin_amdgcn_readlane(rowoff, sl);
            st_sys(o0, ro, vo, ro_s);
            st_sys(o1, ro, vo, ro_s + 16384);
          } else {
            const u4 o2 = *(const u4*)(eb + 2048 + (lane & 7) * 16);
            const int nl = (96 + 127 + 4 * __builtin_amdgcn_readlane(o0.y, 5)) >> 7;
            st_sys(o0, rq, lane < 8 * nl ? vl : OOB, tb + wl * 128);
            if (nl > 8) st_sys(o1, rq, lane + 64 < 8 * nl ? vl : OOB, tb + wl * 128 + 1024);
            if (nl > 16) st_sys(o2, rq, lane < 8 ? vl : OOB, tb + wl * 128 + 2048);
            wl += nl;
          }
          bi = bi + 1 == nb ? 0 : bi + 1;
        }
        CBAR();
        if (helper) { st_sys32((call << 14) | (unsigned)arrived, rq, told != arrived ? v0 : OOB, fo); told = arrived; }
        if (lane == 0) lds_st(ctl + C_FL, (unsigned)n);
      }
      if (helper || abort) WAITV0();
      if (abort) {
        pending |= 1u;
        while (pending != 0u) {
          const int ro_s = __builtin_amdgcn_readlane(rowoff, __builtin_ctz(pending));
          pending &= pending - 1u;
          const u4 nan4 = u4{0x7fc00000u, 0x7fc00000u, 0x7fc00000u, 0x7fc00000u};
          st_sys(nan4, ro, vo, ro_s);
          st_sys(nan4, ro, vo, ro_s + 16384);
        }
      } else if (helper) {
        st_sys32((call << 14) | 512u | (unsigned)wl, rq, v0, fo);
      }
    } else {
      st_sys32(call, rws, v0, OFF_CTR + blockIdx.x * 64);
      const int wo = (lane >> 4) * 512 + (lane & 7) * 16 + ((lane >> 3) & 1) * 4;
      const int nb = helper ? NBACK : NBOWN;
      int bi = 0;
      for (int n = 0; n < nneed; ++n) {
        bool abort;
        unsigned c;
        for (;;) {
          c = lds_ld(ctl + lane);
          abort = (__builtin_amdgcn_readlane(c, C_MASK) >> F_ABORT) & 1u;
          const int pc = (n & 1) ? __builtin_amdgcn_readlane(c, C_PCNT + 1) : __builtin_amdgcn_readlane(c, C_PCNT);
          if (abort || (pc >= 4 * ((n >> 1) + 1) && n - (int)__builtin_amdgcn_readlane(c, C_FL) < nb)) break;
          __builtin_amdgcn_s_sleep(1);
        }
        if (abort) break;
        CBAR();
        const char* pp = smem + L_PART + (n & 1) * 8192 + vl;
        char* e = smem + L_BACK + bi * BPITCH;
        float v[8];
#pragma unroll
        for (int m = 0; m < 2; ++m) {
          const f4 s = (*(const f4*)(pp + m * 1024) + *(const f4*)(pp + 2048 + m * 1024)) +
                       (*(const f4*)(pp + 4096 + m * 1024) + *(const f4*)(pp + 6144 + m * 1024));
#pragma unroll
          for (int jj = 0; jj < 4; ++jj) v[m * 4 + jj] = s[jj] < 0.f ? 0.f : s[jj];
        }
        if (n >= npack) {
#pragma unroll
          for (int r = 0; r < 8; ++r) *(float*)(e + wo + (r & 3) * 128 + (r >> 2) * 8) = v[r];
        } else {
          int base = 0, hb = 0;
          u64 hm = 0;
#pragma unroll
          for (int r = 0; r < 8; ++r) {
            const bool nz = __builtin_bit_cast(unsigned, v[r]) != 0u;
            const u64 mk = __ballot(nz);
            const int pos = base + __builtin_amdgcn_mbcnt_hi((unsigned)(mk >> 32), __builtin_amdgcn_mbcnt_lo((unsigned)mk, 0u));
            if (nz) *(float*)(e + 96 + pos * 4) = v[r];
            if (lane == r) { hm = mk; hb = base; }
            base += __builtin_popcountll(mk);
          }
          if (lane < 8) { *(u64*)(e + lane * 8) = hm; *(unsigned short*)(e + 64 + lane * 2) = (unsigned short)hb; }
          if (lane < 2) *(unsigned*)(e + 80 + lane * 4) = lane == 0 ? (unsigned)__builtin_amdgcn_readlane(c, C_QSLOT + n) : (unsigned)base;
        }
        CBAR();
        if (lane == 0) lds_st(ctl + C_RED, (unsigned)(n + 1));
        bi = bi + 1 == nb ? 0 : bi + 1;
      }
    }
    return;
  }

  {
    const unsigned call = __builtin_amdgcn_readfirstlane(cprev) + 1u;
    asm volatile("" :: "s"(call) : "memory");
    __builtin_amdgcn_s_barrier();
    CBAR();
    __builtin_amdgcn_s_sleep(SLP_STAGE);
    for (int z = (blockIdx.x >> 3) - 4; z > 0; --z) __builtin_amdgcn_s_sleep(SLP_STIER);
    stage_local(cidx * 4 + wid, call, rws, rx, rw, rcnt, vx, vw, lane);
    const int wbase = OFF_W + (call & 1u) * (8u << 20);

    for (;;) {
      const unsigned mk = __builtin_amdgcn_readfirstlane(lds_ld(ctl + C_MASK));
      if ((mk >> F_ABORT) & 1u) return;
      if ((mk >> (F_W + wid)) & 1u) break;
      __builtin_amdgcn_s_sleep(4);
    }
    CBAR();
    int qo[8];
#pragma unroll
    for (int qd = 0; qd < 8; ++qd) qo[qd] = ((qd + cu) & 7) * 4096;
    const int vb = ((lane >> 3) & 1) * 1024 + (lane >> 4) * 128 + (lane & 7) * 16;
    u4 breg[18][2];
    char* ldsb = smem + L_B + wid * (NLDSK * 2048) + vl;
#pragma unroll
    for (int hf = 0; hf < 2; ++hf) {
      const int so = wbase + cu * 262144 + wid * 65536;
      u4 tmp[7][2];
#pragma unroll
      for (int t = 0; t < 16; ++t) {
#pragma unroll
        for (int m = 0; m < 2; ++m) {
          const u4 v = ld_sys(rws, vb + m * 2048 + (t & 1) * 512, so + 2 * qo[hf * 4 + (t >> 2)] + ((t >> 1) & 1) * 4096);
          if (k_in_lds(t)) tmp[t >> 1][m] = v; else breg[reg_index(hf * 16 + t)][m] = v;
        }
      }
#pragma unroll
      for (int t = 0; t < 7; ++t) {
#pragma unroll
        for (int m = 0; m < 2; ++m) *(u4*)(ldsb + (hf * 7 + t) * 2048 + m * 1024) = tmp[t][m];
      }
    }

    u4 ring[16];
    u4 bt0 = *(const u4*)ldsb, bt1 = *(const u4*)(ldsb + 1024);
    int va = vl + wid * 32768;
    int cur = (int)WS_USED;
    for (int n = 0; n < nneed; ++n) {
      if (cur == (int)WS_USED) {
        for (;;) {
          cur = __builtin_amdgcn_readfirstlane(lds_ld(ctl + C_QOFF + n));
          if (cur != 0) break;
          if ((__builtin_amdgcn_readfirstlane(lds_ld(ctl + C_MASK)) >> F_ABORT) & 1u) return;
          __builtin_amdgcn_s_sleep(1);
        }
        asm volatile("" : "+v"(va) :: "memory");
#pragma unroll
        for (int t = 0; t < 16; ++t) {
          ring[t] = ld_sys(rws, va + (t & 3) * 1024, cur + qo[t >> 2]);
          __builtin_amdgcn_sched_barrier(0);
        }
      }
      f4 acc0 = f4{0.f, 0.f, 0.f, 0.f}, acc1 = f4{0.f, 0.f, 0.f, 0.f};
      int nb = 0;
      unsigned r0 = 0;
      asm volatile("" : "+v"(va));
#pragma unroll
      for (int t = 0; t < 32; ++t) {
        if (t == 16) { nb = __builtin_amdgcn_readfirstlane(r0); nb = nb != 0 ? nb : (int)WS_USED; CBAR(); }
        const u4 b0 = k_in_lds(t) ? bt0 : breg[reg_index(t)][0];
        const u4 b1 = k_in_lds(t) ? bt1 : breg[reg_index(t)][1];
        const h8 a = __builtin_bit_cast(h8, ring[t & 15]);
        acc0 = __builtin_amdgcn_mfma_f32_16x16x32_f16(a, __builtin_bit_cast(h8, b0), acc0, 0, 0, 0);
        acc1 = __builtin_amdgcn_mfma_f32_16x16x32_f16(a, __builtin_bit_cast(h8, b1), acc1, 0, 0, 0);
        __builtin_amdgcn_sched_barrier(0);
        ring[t & 15] = ld_sys(rws, va + (t & 3) * 1024, (t < 16 ? cur : nb) + qo[((t + 16) & 31) >> 2]);
        if (k_in_lds(t)) {
          const int nn = (lds_index(t) + 1) % NLDSK;
          bt0 = *(const u4*)(ldsb + nn * 2048);
          bt1 = *(const u4*)(ldsb + nn * 2048 + 1024);
        }
        if (t == 14) r0 = lds_ld(ctl + C_QOFF + n + 1);
        if (t == 30) r0 = lds_ld(ctl + C_RED);
        __builtin_amdgcn_sched_barrier(0);
      }
      int red = __builtin_amdgcn_readfirstlane(r0);
      while (red < n - 1) {
        __builtin_amdgcn_s_sleep(1);
        if ((__builtin_amdgcn_readfirstlane(lds_ld(ctl + C_MASK)) >> F_ABORT) & 1u) return;
        red = __builtin_amdgcn_readfirstlane(lds_ld(ctl + C_RED));
      }
      CBAR();
      char* pp = smem + L_PART + (n & 1) * 8192 + wid * 2048 + vl;
      *(f4*)pp = acc0;
      *(f4*)(pp + 1024) = acc1;
      CBAR();
      if (lane == 0) __hip_atomic_fetch_add(ctl + C_PCNT + (n & 1), 1u, __ATOMIC_RELAXED, __HIP_MEMORY_SCOPE_WORKGROUP);
      CBAR();
      cur = nb;
    }
  }
}

}

extern "C" void kernel_launch(int rank, int nranks, ncclComm_t comm,
                              hipStream_t stream,
                              void* const* d_in, const int* in_sizes, int n_in,
                              void* d_out, int out_size,
                              const int* in_dtypes, int out_dtype,
                              void* const* d_out_all,
                              void* const* d_ws_all, size_t ws_bytes) {
  if (nranks != NR || ws_bytes < (size_t)WS_USED + (1u << 20) || n_in < 2 || (unsigned)rank >= (unsigned)NR) {
    fprintf(stderr, "unsupported configuration\n");
    return;
  }
  Params p;
  p.x = (const float*)d_in[0];
  p.w = (const float*)d_in[1];
  for (int d = 0; d < NR; ++d) {
    p.ws[d] = (char*)d_ws_all[rank ^ d];
    p.out[d] = (float*)d_out_all[rank ^ d];
  }
  p.rank = rank;
  hipLaunchKernelGGL(ag_gemm_relu, dim3(256), dim3(512), 0, stream, p);
  HIP_CHECK(hipGetLastError());
}
```

```cpp
#include <hip/hip_runtime.h>
#include <rccl/rccl.h>
#include <cstdio>

#define HIP_CHECK(expr) do { hipError_t _e = (expr); if (_e != hipSuccess) { \
    fprintf(stderr, "HIP error %s at %s:%d\n", hipGetErrorString(_e), __FILE__, __LINE__); \
    return; } } while (0)

namespace {

typedef float f4 __attribute__((ext_vector_type(4)));
typedef unsigned u4 __attribute__((ext_vector_type(4)));
typedef _Float16 h8 __attribute__((ext_vector_type(8)));
typedef unsigned long long u64;
typedef __amdgpu_buffer_rsrc_t rsrc_t;

constexpr int NR = 8;
constexpr int NG = 7;
constexpr int ND = 22;
constexpr int NXB = 32 - ND;
constexpr int S1 = 32;
constexpr int S3 = 18;
constexpr int ONE = 96;
constexpr int S0 = S1 + ONE;
constexpr int NSTEP = ND * 32;
constexpr int NWST = 256;
constexpr int E2 = NWST + NXB * 32;
constexpr int GATE_STEP = E2 - 12;
constexpr int QUIT_STEP = E2 - 16;
constexpr int FWIN = 2;
constexpr int T_STAGE = 110, T_STIER = 42, T_TIER = 135;
constexpr int RAWTAIL = 3;
static_assert(ND >= 20 && ND <= 24, "slot table");
static_assert(S3 > 0 && S3 <= S1 && ONE <= 112 && S0 <= NWST && GATE_STEP >= 0 && GATE_STEP < NSTEP, "sender configuration");

constexpr u64 DEADLINE = 100000000ull;

constexpr unsigned OFF_PL = 0;
constexpr unsigned OFF_TMO = 40960;
constexpr unsigned OFF_CNT = 45056;
constexpr unsigned OFF_CTR = 65536;
constexpr unsigned OFF_SL = 98304;
constexpr unsigned OFF_RP = 110592;
constexpr unsigned OFF_X = 1u << 20;
constexpr unsigned OFF_W = 65u << 20;
constexpr unsigned OFF_T = 81u << 20;
constexpr unsigned WS_USED = 87u << 20;
constexpr int TBYTES = 49152;
static_assert(112u * TBYTES <= WS_USED - OFF_T && 22 * 2176 <= TBYTES, "tile region");
constexpr int CI_TICK = 0;
constexpr int CI_SX = 4;
constexpr int CI_SW = 36;
constexpr int CI_LX = 52;
constexpr int CI_LW = 84;
constexpr int CI_END = 148;
constexpr int NSLOT = 25;
constexpr int F_W = 25;
constexpr int F_GATE = 29;
constexpr int F_ABORT = 31;

constexpr int NLDSK = 14;
constexpr int NBACK = 15;
constexpr int NBOWN = 8;
constexpr int BPITCH = 2144;
constexpr int L_B = 0;
constexpr int L_PART = 4 * NLDSK * 2048;
constexpr int L_BACK = L_PART + 16384;
constexpr int L_CTL = L_BACK + NBACK * BPITCH;
constexpr int C_QOFF = 0;
constexpr int C_QSLOT = 24;
constexpr int C_PCNT = 48;
constexpr int C_RED = 50;
constexpr int C_FL = 51;
constexpr int C_MASK = 52;
constexpr int C_WORDS = 64;
constexpr int L_EXP = L_BACK + NBOWN * BPITCH;
constexpr int LDS_BYTES = L_CTL + C_WORDS * 4;
static_assert(LDS_BYTES <= 163840, "LDS");

struct Params {
  const float* x;
  const float* w;
  char* ws[NR];
  float* out[NR];
  int rank;
};

#define WAITV0() do { asm volatile("" ::: "memory"); __builtin_amdgcn_s_waitcnt(0x0F70); asm volatile("" ::: "memory"); } while (0)
#define CBAR() asm volatile("" ::: "memory")
constexpr int OOB = 0x40000000;

__device__ __forceinline__ rsrc_t mk_rsrc(const void* p, unsigned bytes) {
  return __builtin_amdgcn_make_buffer_rsrc(const_cast<void*>(p), 0, bytes, 0x00020000);
}
__device__ __forceinline__ f4 ld_in(rsrc_t r, int voff, int soff) {
  return __builtin_bit_cast(f4, __builtin_amdgcn_raw_buffer_load_b128(r, voff, soff, 0));
}
__device__ __forceinline__ u4 ld_sys(rsrc_t r, int voff, int soff) {
  return __builtin_amdgcn_raw_buffer_load_b128(r, voff, soff, 17);
}
__device__ __forceinline__ unsigned ld_sys32(rsrc_t r, int voff, int soff) {
  return __builtin_amdgcn_raw_buffer_load_b32(r, voff, soff, 17);
}
__device__ __forceinline__ void st_sys(u4 v, rsrc_t r, int voff, int soff) {
  __builtin_amdgcn_raw_buffer_store_b128(v, r, voff, soff, 17);
}
__device__ __forceinline__ void st_sys32(unsigned v, rsrc_t r, int voff, int soff) {
  __builtin_amdgcn_raw_buffer_store_b32(v, r, voff, soff, 17);
}
__device__ __forceinline__ u4 cvt8(f4 a, f4 b) {
  h8 h = h8{(_Float16)a.x, (_Float16)a.y, (_Float16)a.z, (_Float16)a.w,
            (_Float16)b.x, (_Float16)b.y, (_Float16)b.z, (_Float16)b.w};
  return __builtin_bit_cast(u4, h);
}
__device__ __forceinline__ u4 cvtcol(const f4* r, int i) {
  h8 h = h8{(_Float16)r[0][i], (_Float16)r[1][i], (_Float16)r[2][i], (_Float16)r[3][i],
            (_Float16)r[4][i], (_Float16)r[5][i], (_Float16)r[6][i], (_Float16)r[7][i]};
  return __builtin_bit_cast(u4, h);
}
__device__ __forceinline__ void load_xpiece(f4* r, rsrc_t rx, int vx, int pi) {
  const int so = (pi >> 5) * (16 * 16384) + (pi & 31) * 512;
#pragma unroll
  for (int t = 0; t < 8; ++t) r[t] = ld_in(rx, vx + t * 64, so);
}
__device__ __forceinline__ void load_wgran(f4* r, rsrc_t rw, int vw, int cg, int kg) {
  const int so = kg * (64 * 2048) + cg * 128;
#pragma unroll
  for (int j = 0; j < 8; ++j) r[j] = ld_in(rw, vw + (j & 1) * 2048, so + (16 * (j >> 2) + (j & 2)) * 2048);
}
__device__ __forceinline__ void pause_ticks(int ticks) {
  const u64 t0 = __builtin_amdgcn_s_memrealtime();
  while (__builtin_amdgcn_s_memrealtime() - t0 < (u64)ticks) __builtin_amdgcn_s_sleep(2);
}
__device__ __forceinline__ unsigned lds_ld(const unsigned* p) {
  return __hip_atomic_load(p, __ATOMIC_RELAXED, __HIP_MEMORY_SCOPE_WORKGROUP);
}
__device__ __forceinline__ void lds_st(unsigned* p, unsigned v) {
  __hip_atomic_store(p, v, __ATOMIC_RELAXED, __HIP_MEMORY_SCOPE_WORKGROUP);
}

__device__ __forceinline__ int src_off(int d) { return (0x65314201 >> (4 * d)) & 7; }
__device__ __forceinline__ int src_sbase(int d) { return d == 0 ? 0 : d < 4 ? 1 + 4 * d : 9 + 2 * d; }
__device__ __forceinline__ int src_lo(int d) { return d < 4 ? 0 : ND; }
__device__ __forceinline__ int flag_off(int d, int b) {
  return (((b + src_off(d)) % NG) * 32 + src_sbase(d) + (b - src_lo(d)) / NG) * 4;
}
__device__ __forceinline__ bool slot_block(int slot, int g, bool helper, int q, int& d, int& b) {
  d = slot < 5 ? 0 : slot < 17 ? (slot - 1) >> 2 : ((slot - 9) >> 1) & 7;
  const int lo = src_lo(d);
  const int hi = d == 0 ? (helper ? ND : 32) : d < 4 ? ND : ((helper && d == 4 + q) ? 0 : 32);
  b = lo + (g + 70 - src_off(d) - lo) % NG + NG * (slot - src_sbase(d));
  return slot < NSLOT && b < hi;
}
__device__ __forceinline__ void stage_local(int gi, unsigned call, rsrc_t rws, rsrc_t rx, rsrc_t rw, rsrc_t rcnt, int vx, int vw, int lane) {
  f4 sx[8], sw[8];
  load_xpiece(sx, rx, vx, gi);
  load_wgran(sw, rw, vw, gi >> 6, gi & 63);
  const int xbase = OFF_X + (call & 1u) * (32u << 20);
  const int wbase = OFF_W + (call & 1u) * (8u << 20);
#pragma unroll
  for (int t = 0; t < 4; ++t) st_sys(cvtcol(sw, t), rws, lane * 16 + t * 1024, wbase + gi * 4096);
#pragma unroll
  for (int t = 0; t < 4; ++t) st_sys(cvt8(sx[2 * t], sx[2 * t + 1]), rws, lane * 16 + t * 1024, xbase + gi * 4096);
  WAITV0();
  const int tv = __builtin_amdgcn_raw_ptr_buffer_atomic_add_i32(1, rcnt, (lane == 0 ? CI_LX + (gi >> 5) : lane == 1 ? CI_LW + (gi >> 4) : CI_END) * 64, 0, 0);
  if ((__builtin_amdgcn_readlane(tv, 0) & 31) == 31)
    st_sys32(call, rws, lane < 8 ? lane * 1024 : OOB, OFF_SL + flag_off(0, gi >> 5));
  if ((__builtin_amdgcn_readlane(tv, 1) & 15) == 15)
    st_sys32(call, rws, lane < NG ? lane * 4096 : OOB, OFF_PL + (gi >> 6) * 128 + (F_W + ((gi >> 4) & 3)) * 4);
}

constexpr bool k_in_lds(int t) { return ((t & 15) & 1) && (t & 15) != 15; }
constexpr int lds_index(int t) { return (t >> 4) * 7 + ((t & 15) >> 1); }
constexpr int reg_index(int t) { return (t >> 4) * 9 + ((t & 15) == 15 ? 8 : ((t & 15) >> 1)); }

__global__ void __launch_bounds__(512, 2) ag_gemm_relu(Params p) {
  __shared__ __attribute__((aligned(16))) char smem[LDS_BYTES];
  unsigned* ctl = (unsigned*)(smem + L_CTL);
  const int lane = threadIdx.x & 63;
  const int wid = __builtin_amdgcn_readfirstlane(threadIdx.x >> 6);
  const bool comm = blockIdx.x < 32;
  const int cidx = (blockIdx.x & 7) * 28 + (blockIdx.x >> 3) - 4;
  const int g = cidx >> 5;
  const int cu = cidx & 31;
  const bool helper = cu >= 16;
  const int q = (cu >> 2) & 3;
  const rsrc_t rws = mk_rsrc(p.ws[0], WS_USED);
  const rsrc_t rx = mk_rsrc(p.x, 512u * 4096u * 4u);
  const rsrc_t rw = mk_rsrc(p.w, 4096u * 512u * 4u);
  const rsrc_t rcnt = mk_rsrc(p.ws[0] + OFF_CNT, CI_END * 64);
  const int vx = (lane & 15) * 16384 + (lane >> 4) * 16;
  const int vw = (32 * (lane >> 5) + 4 * ((lane >> 3) & 3)) * 2048 + (lane & 7) * 16;
  const int vl = lane * 16;
  const int v0 = lane == 0 ? 0 : OOB;

  const unsigned cprev = ld_sys32(rws, 0, OFF_CTR + blockIdx.x * 64);

  const int j = comm ? (int)blockIdx.x : S1 + ((cu + 16) & 31) * NG + g;
  if (__builtin_expect(comm ? wid == 0 : wid == 6 && j < S0, 1)) {
    f4 rd[8], ra[8], rb[24];
    load_xpiece(rd, rx, vx, j);
    load_wgran(ra, rw, vw, j & 3, j >> 2);
#pragma unroll
    for (int t = 0; t < 3; ++t) load_wgran(rb + 8 * t, rw, vw, 4 * (t + 1) + (j & 3), j >> 2);
    asm volatile("" :: "s"(p.ws[1]), "s"(p.ws[2]), "s"(p.ws[3]), "s"(p.ws[4]), "s"(p.ws[5]), "s"(p.ws[6]), "s"(p.ws[7]));
    const unsigned call = __builtin_amdgcn_readfirstlane(cprev) + 1u;
    asm volatile("" :: "s"(call) : "memory");
    __builtin_amdgcn_s_barrier();
    CBAR();
    __builtin_amdgcn_s_setprio(2);
    const unsigned par = call & 1u;
    rsrc_t rp[NR];
#pragma unroll
    for (int d = 1; d < NR; ++d) rp[d] = mk_rsrc(p.ws[d], WS_USED);
    const int xbase = OFF_X + par * (32u << 20);
    const int wbase = OFF_W + par * (8u << 20);
    const int vfl = lane < 8 ? lane * 1024 : OOB;
    const int vgl = lane < NG ? lane * 4096 : OOB;
    const int qstep = !comm ? 0 : j >= S3 ? QUIT_STEP : NSTEP;
    int i = j, prev = -1;
    WAITV0();
    if (j >= S1 + 8) pause_ticks(((j - S1) >> 3) * T_TIER);
    for (;;) {
      const bool quit = i >= qstep;
      int cv = CI_END;
      if (lane == 0 && !quit) cv = CI_TICK + 2 * par;
      if (lane == 1 && prev >= 0) cv = CI_SX + (prev >> 5);
      if (lane == 2 && prev >= 0 && prev < E2) cv = prev < NWST ? CI_SW + (prev >> 6) * 4 + (prev & 3) : CI_SX + ND + ((prev - NWST) >> 5);
      const int tv = __builtin_amdgcn_raw_ptr_buffer_atomic_add_i32(1, rcnt, cv * 64, 0, 0);
      int t0v, t1v, t2v;
#define READ_BACK() do { t0v = __builtin_amdgcn_readlane(tv, 0); t1v = __builtin_amdgcn_readlane(tv, 1); t2v = __builtin_amdgcn_readlane(tv, 2); } while (0)
      if (i == GATE_STEP) {
        st_sys32(call, rws, (lane >> 4) * 4096 + (lane & 15) * 128, OFF_PL + 16 * 128 + F_GATE * 4);
        st_sys32(call, rws, lane < 48 ? (lane >> 4) * 4096 + (lane & 15) * 128 : OOB, OFF_PL + 4 * 4096 + 16 * 128 + F_GATE * 4);
      }
      if (i >= NSTEP) {
        READ_BACK();
      } else if (i < NWST) {
        const int wo = wbase + (16 + (i & 3)) * 262144 + (i >> 2) * 4096;
#pragma unroll
        for (int t = 0; t < 4; ++t) {
          const u4 h = cvt8(rd[2 * t], rd[2 * t + 1]);
#pragma unroll
          for (int d = 1; d < 4; ++d) st_sys(h, rp[d], vl + t * 1024, xbase + d * (4 << 20) + i * 4096);
          st_sys(cvtcol(ra, t), rp[4], vl + t * 1024, wo);
#pragma unroll
          for (int d = 5; d < 8; ++d) st_sys(cvtcol(rb + 8 * (d - 5), t), rp[d], vl + t * 1024, wo + (d - 4) * (4 * 262144));
        }
        READ_BACK();
      } else if (i < E2) {
        const int xi = NSTEP + i - NWST;
#pragma unroll
        for (int t = 0; t < 4; ++t) {
          const u4 h = cvt8(rd[2 * t], rd[2 * t + 1]);
          const u4 gg = cvt8(ra[2 * t], ra[2 * t + 1]);
#pragma unroll
          for (int d = 1; d < 4; ++d) st_sys(h, rp[d], vl + t * 1024, xbase + d * (4 << 20) + i * 4096);
#pragma unroll
          for (int d = 4; d < 8; ++d) st_sys(gg, rp[d], vl + t * 1024, xbase + d * (4 << 20) + xi * 4096);
        }
        READ_BACK();
      } else {
#pragma unroll
        for (int t = 0; t < 4; ++t) {
          const u4 h = cvt8(rd[2 * t], rd[2 * t + 1]);
#pragma unroll
          for (int d = 1; d < 4; ++d) st_sys(h, rp[d], vl + t * 1024, xbase + d * (4 << 20) + i * 4096);
        }
        READ_BACK();
      }
      if (prev >= 0) {
        if ((t1v & 31) == 31) {
#pragma unroll
          for (int d = 1; d < 4; ++d) st_sys32(call, rp[d], vfl, OFF_SL + flag_off(d, prev >> 5));
        }
        if (prev < NWST) {
          if ((t2v & 15) == 15) {
#pragma unroll
            for (int d = 4; d < 8; ++d)
              st_sys32(call, rp[d], vgl, OFF_PL + (16 + 4 * (d - 4) + (prev & 3)) * 128 + (F_W + (prev >> 6)) * 4);
          }
        } else if (prev < E2) {
          if ((t2v & 31) == 31) {
#pragma unroll
            for (int d = 4; d < 8; ++d) st_sys32(call, rp[d], vfl, OFF_SL + flag_off(d, ND + ((prev - NWST) >> 5)));
          }
        }
      }
      if (i >= NSTEP) break;
      int nxt = t0v + S0;
      if (quit || (unsigned)nxt >= (unsigned)NSTEP) nxt = NSTEP;
      if (nxt < NSTEP) {
        load_xpiece(rd, rx, vx, nxt);
        if (nxt < NWST) {
          load_wgran(ra, rw, vw, nxt & 3, nxt >> 2);
#pragma unroll
          for (int t = 0; t < 3; ++t) load_wgran(rb + 8 * t, rw, vw, 4 * (t + 1) + (nxt & 3), nxt >> 2);
        } else if (nxt < E2) {
          load_xpiece(ra, rx, vx, NSTEP + nxt - NWST);
        }
      }
      WAITV0();
      prev = i;
      i = nxt;
    }
    return;
  }

  if (comm || wid == 6) {
    const unsigned call = __builtin_amdgcn_readfirstlane(cprev) + 1u;
    asm volatile("" :: "s"(call) : "memory");
    __builtin_amdgcn_s_barrier();
    CBAR();
    if (comm) {
      if (wid == 7) {
        st_sys32(call, rws, v0, OFF_CTR + blockIdx.x * 64);
        if (blockIdx.x == 31) st_sys32(0u, rcnt, v0, (CI_TICK + 2 * ((call & 1u) ^ 1u)) * 64);
      }
      return;
    }
    pause_ticks(T_STAGE + ((blockIdx.x >> 3) - 4) * T_STIER);
    stage_local(896 + j - S0, call, rws, rx, rw, rcnt, vx, vw, lane);
    if (helper) return;
    __builtin_amdgcn_s_setprio(2);
    const int z = q, sub = cu & 3, eg = g, e = z * 28 + sub * NG + g;
    int pd, pb;
    unsigned ppend = (unsigned)__ballot(slot_block(lane, eg, true, z, pd, pb));
    const int prow = (((p.rank ^ (4 + z) ^ pd) & 7) * 512 + pb * 16) * 2048;
    const rsrc_t ro = mk_rsrc(p.out[0], 4096u * 512u * 4u);
    const int vo = (lane >> 3) * 2048 + (lane & 7) * 16 + (4 * z + sub) * 128;
    const int tb = OFF_T + e * TBYTES;
    char* sc = smem + L_EXP;
    const int jr = (lane >> 3) & 3;
    const int sl0 = (lane >> 5) * 16 + (lane & 7);
    const int pv = lane == 0 ? (int)(OFF_RP + e * 128) : lane == 2 ? (int)OFF_TMO : OOB;
    int rl = 0, tried = 0, oc = 0;
    bool abort = false;
    const u64 t0 = __builtin_amdgcn_s_memrealtime();
    for (;;) {
      const unsigned fv = ld_sys32(rws, pv, 0);
      const unsigned ow = __builtin_amdgcn_readlane(fv, 0) - (call << 14);
      if (ow < 1024u && (int)ow > oc) oc = (int)ow;
      const int avail = (oc & 511) - rl;
      if (avail > tried) {
        const int got = avail < 64 ? avail : 64;
        u4 t[8];
#pragma unroll
        for (int y = 0; y < 8; ++y) t[y] = ld_sys(rws, (lane >> 3) + 8 * y < got ? vl : OOB, tb + rl * 128 + y * 1024);
#pragma unroll
        for (int y = 0; y < 8; ++y) *(u4*)(sc + y * 1024 + vl) = t[y];
        CBAR();
        int at = 0;
        while (at < got) {
          const char* h = sc + at * 128;
          const unsigned hd = *(const unsigned*)(h + 80 + (lane & 1) * 4);
          const u64 mk0 = *(const u64*)(h + jr * 8), mk1 = *(const u64*)(h + 32 + jr * 8);
          const int b0 = *(const unsigned short*)(h + 64 + jr * 2), b1 = *(const unsigned short*)(h + 72 + jr * 2);
          const int nl = (96 + 127 + 4 * (__builtin_amdgcn_readlane(hd, 1) & 1023)) >> 7;
          if (at + nl > got) break;
          const int sl = __builtin_amdgcn_readlane(hd, 0) & 31;
          const int ro_s = __builtin_amdgcn_readlane(prow, sl);
          ppend &= ~(1u << sl);
          unsigned o[2][4];
#pragma unroll
          for (int m = 0; m < 2; ++m) {
            const u64 mk = m ? mk1 : mk0;
            const int base = m ? b1 : b0;
#pragma unroll
            for (int hh = 0; hh < 2; ++hh) {
#pragma unroll
              for (int i1 = 0; i1 < 2; ++i1) {
                const int s = sl0 + hh * 32 + i1 * 8;
                const int idx = (base + __builtin_popcountll(mk & ((1ull << s) - 1ull))) & 511;
                const unsigned v = *(const unsigned*)(h + 96 + idx * 4);
                o[hh][m * 2 + i1] = ((mk >> s) & 1ull) ? v : 0u;
              }
            }
          }
          st_sys(u4{o[0][0], o[0][1], o[0][2], o[0][3]}, ro, vo, ro_s);
          st_sys(u4{o[1][0], o[1][1], o[1][2], o[1][3]}, ro, vo, ro_s + 16384);
          at += nl;
        }
        CBAR();
        tried = got == avail ? got - at : 0;
        if (tried == 0 && at == 0) { abort = true; break; }
        rl += at;
        continue;
      }
      if (avail == 0 && (oc & 512) != 0) break;
      if (__builtin_amdgcn_readlane(fv, 2) == call || __builtin_amdgcn_s_memrealtime() - t0 > DEADLINE) { abort = true; break; }
      if (oc == 0) __builtin_amdgcn_s_sleep(16); else __builtin_amdgcn_s_sleep(2);
    }
    if (abort) {
      st_sys32(call, rws, v0, OFF_TMO);
      ppend |= 1u;
      WAITV0();
      while (ppend != 0u) {
        const int ro_s = __builtin_amdgcn_readlane(prow, __builtin_ctz(ppend));
        ppend &= ppend - 1u;
        const u4 nan4 = u4{0x7fc00000u, 0x7fc00000u, 0x7fc00000u, 0x7fc00000u};
        st_sys(nan4, ro, vo, ro_s);
        st_sys(nan4, ro, vo, ro_s + 16384);
      }
    }
    return;
  }

  int sd, sb;
  const bool sused = slot_block(lane, g, helper, q, sd, sb);
  const unsigned needmask = (unsigned)__ballot(sused);
  const int nneed = __builtin_popcount(needmask);

  if (wid >= 4) {
    if (wid == 4) lds_st(ctl + lane, 0u);
    const unsigned call = __builtin_amdgcn_readfirstlane(cprev) + 1u;
    asm volatile("s_waitcnt lgkmcnt(0)" :: "s"(call) : "memory");
    __builtin_amdgcn_s_barrier();
    CBAR();
    __builtin_amdgcn_s_setprio(2);
    const int npack = !helper ? 0 : nneed > RAWTAIL ? nneed - RAWTAIL : 0;

    if (wid == 4) {
      const unsigned want = needmask | (15u << F_W) | (helper ? 1u << F_GATE : 0u);
      const int pv = lane < NSLOT ? (int)(OFF_SL + ((cu >> 2) * 8 + g) * 128 + lane * 4)
                   : lane < 32 ? (int)(OFF_PL + cidx * 128 + lane * 4) : lane == 32 ? (int)OFF_TMO : OOB;
      const unsigned boff = OFF_X + (call & 1u) * (32u << 20) + sd * (4 << 20) + sb * 131072;
      unsigned seen = 0;
      const u64 t0 = __builtin_amdgcn_s_memrealtime();
      for (;;) {
        const unsigned fv = ld_sys32(rws, pv, 0);
        const unsigned m = (unsigned)__ballot((int)(fv - call) >= 0) & want;
        if (m != seen) {
          const unsigned fresh = m & ~seen & needmask;
          if (lane < 32 && ((fresh >> lane) & 1u)) {
            const int pos = __builtin_popcount(seen & needmask) + __builtin_popcount(fresh >> lane) - 1;
            lds_st(ctl + C_QSLOT + pos, (unsigned)lane);
            lds_st(ctl + C_QOFF + pos, boff);
          }
          if (lane == 0) lds_st(ctl + C_MASK, m);
          seen = m;
          if (m == want) break;
        }
        if (__builtin_amdgcn_readlane(fv, 32) == call || __builtin_amdgcn_s_memrealtime() - t0 > DEADLINE) {
          st_sys32(call, rws, v0, OFF_TMO);
          if (lane == 0) lds_st(ctl + C_MASK, seen | (1u << F_ABORT));
          break;
        }
        __builtin_amdgcn_s_sleep(1);
      }
    } else if (wid == 5) {
      unsigned pending = needmask;
      const int rowoff = (((p.rank ^ sd) & 7) * 512 + sb * 16) * 2048;
      const int vo = (lane >> 3) * 2048 + (lane & 7) * 16 + (cu & 15) * 128;
      const rsrc_t ro = mk_rsrc(!helper ? p.out[0] : q == 0 ? p.out[4] : q == 1 ? p.out[5] : q == 2 ? p.out[6] : p.out[7], 4096u * 512u * 4u);
      const int e = q * 28 + (cu & 3) * NG + g;
      const rsrc_t rq = mk_rsrc(q == 0 ? p.ws[4] : q == 1 ? p.ws[5] : q == 2 ? p.ws[6] : p.ws[7], WS_USED);
      const int fo = OFF_RP + e * 128;
      const int tb = OFF_T + e * TBYTES;
      const int nb = helper ? NBACK : NBOWN;
      int bi = 0, wl = 0, told = 0;
      bool abort = false, word_last = false;
      for (int n = 0; n < nneed;) {
        if (word_last) asm volatile("s_waitcnt vmcnt(1)" ::: "memory");
        else if (helper) asm volatile("s_waitcnt vmcnt(0)" ::: "memory");
        const int arrived = wl;
        unsigned c;
        int red;
        for (;;) {
          c = lds_ld(ctl + lane);
          const unsigned mk = __builtin_amdgcn_readlane(c, C_MASK);
          red = __builtin_amdgcn_readlane(c, C_RED);
          abort = (mk >> F_ABORT) & 1u;
          const bool hold = n >= npack && red < nneed && (mk & needmask) == needmask;
          if (abort || (red > n && (!helper || (((mk >> F_GATE) & 1u) && !hold)))) break;
          if (told != arrived) { st_sys32((call << 14) | (unsigned)arrived, rq, v0, fo); told = arrived; }
          __builtin_amdgcn_s_sleep(1);
        }
        if (abort) break;
        CBAR();
        const int nend = red < n + FWIN || n >= npack ? red : n + FWIN;
        for (; n < nend; ++n) {
          const int sl = __builtin_amdgcn_readlane(c, C_QSLOT + n);
          pending &= ~(1u << sl);
          const char* eb = smem + L_BACK + bi * BPITCH;
          const u4 o0 = *(const u4*)(eb + vl);
          const u4 o1 = *(const u4*)(eb + 1024 + vl);
          if (n >= npack) {
            const int ro_s = __builtin_amdgcn_readlane(rowoff, sl);
            st_sys(o0, ro, vo, ro_s);
            st_sys(o1, ro, vo, ro_s + 16384);
          } else {
            const u4 o2 = *(const u4*)(eb + 2048 + (lane & 7) * 16);
            const int nl = (96 + 127 + 4 * __builtin_amdgcn_readlane(o0.y, 5)) >> 7;
            st_sys(o0, rq, lane < 8 * nl ? vl : OOB, tb + wl * 128);
            if (nl > 8) st_sys(o1, rq, lane + 64 < 8 * nl ? vl : OOB, tb + wl * 128 + 1024);
            if (nl > 16) st_sys(o2, rq, lane < 8 ? vl : OOB, tb + wl * 128 + 2048);
            wl += nl;
          }
          bi = bi + 1 == nb ? 0 : bi + 1;
        }
        CBAR();
        word_last = told != arrived;
        if (word_last) { st_sys32((call << 14) | (unsigned)arrived, rq, v0, fo); told = arrived; }
        if (lane == 0) lds_st(ctl + C_FL, (unsigned)n);
      }
      if (helper || abort) WAITV0();
      if (abort) {
        pending |= 1u;
        while (pending != 0u) {
          const int ro_s = __builtin_amdgcn_readlane(rowoff, __builtin_ctz(pending));
          pending &= pending - 1u;
          const u4 nan4 = u4{0x7fc00000u, 0x7fc00000u, 0x7fc00000u, 0x7fc00000u};
          st_sys(nan4, ro, vo, ro_s);
          st_sys(nan4, ro, vo, ro_s + 16384);
        }
      } else if (helper) {
        st_sys32((call << 14) | 512u | (unsigned)wl, rq, v0, fo);
      }
    } else {
      st_sys32(call, rws, v0, OFF_CTR + blockIdx.x * 64);
      const int wo = (lane >> 4) * 512 + (lane & 7) * 16 + ((lane >> 3) & 1) * 4;
      const int nb = helper ? NBACK : NBOWN;
      int bi = 0;
      for (int n = 0; n < nneed; ++n) {
        bool abort;
        unsigned c;
        for (;;) {
          c = lds_ld(ctl + lane);
          abort = (__builtin_amdgcn_readlane(c, C_MASK) >> F_ABORT) & 1u;
          const int pc = (n & 1) ? __builtin_amdgcn_readlane(c, C_PCNT + 1) : __builtin_amdgcn_readlane(c, C_PCNT);
          if (abort || (pc >= 4 * ((n >> 1) + 1) && n - (int)__builtin_amdgcn_readlane(c, C_FL) < nb)) break;
          __builtin_amdgcn_s_sleep(1);
        }
        if (abort) break;
        CBAR();
        const char* pp = smem + L_PART + (n & 1) * 8192 + vl;
        char* e = smem + L_BACK + bi * BPITCH;
        float v[8];
#pragma unroll
        for (int m = 0; m < 2; ++m) {
          const f4 s = (*(const f4*)(pp + m * 1024) + *(const f4*)(pp + 2048 + m * 1024)) +
                       (*(const f4*)(pp + 4096 + m * 1024) + *(const f4*)(pp + 6144 + m * 1024));
#pragma unroll
          for (int jj = 0; jj < 4; ++jj) v[m * 4 + jj] = s[jj] < 0.f ? 0.f : s[jj];
        }
        if (n >= npack) {
#pragma unroll
          for (int r = 0; r < 8; ++r) *(float*)(e + wo + (r & 3) * 128 + (r >> 2) * 8) = v[r];
        } else {
          int base = 0, hb = 0;
          u64 hm = 0;
#pragma unroll
          for (int r = 0; r < 8; ++r) {
            const bool nz = __builtin_bit_cast(unsigned, v[r]) != 0u;
            const u64 mk = __ballot(nz);
            const int pos = base + __builtin_amdgcn_mbcnt_hi((unsigned)(mk >> 32), __builtin_amdgcn_mbcnt_lo((unsigned)mk, 0u));
            if (nz) *(float*)(e + 96 + pos * 4) = v[r];
            if (lane == r) { hm = mk; hb = base; }
            base += __builtin_popcountll(mk);
          }
          if (lane < 8) { *(u64*)(e + lane * 8) = hm; *(unsigned short*)(e + 64 + lane * 2) = (unsigned short)hb; }
          if (lane < 2) *(unsigned*)(e + 80 + lane * 4) = lane == 0 ? (unsigned)__builtin_amdgcn_readlane(c, C_QSLOT + n) : (unsigned)base;
        }
        CBAR();
        if (lane == 0) lds_st(ctl + C_RED, (unsigned)(n + 1));
        bi = bi + 1 == nb ? 0 : bi + 1;
      }
    }
    return;
  }

  {
    const unsigned call = __builtin_amdgcn_readfirstlane(cprev) + 1u;
    asm volatile("" :: "s"(call) : "memory");
    __builtin_amdgcn_s_barrier();
    CBAR();
    pause_ticks(T_STAGE + ((blockIdx.x >> 3) - 4) * T_STIER);
    stage_local(cidx * 4 + wid, call, rws, rx, rw, rcnt, vx, vw, lane);
    const int wbase = OFF_W + (call & 1u) * (8u << 20);

    for (;;) {
      const unsigned mk = __builtin_amdgcn_readfirstlane(lds_ld(ctl + C_MASK));
      if ((mk >> F_ABORT) & 1u) return;
      if ((mk >> (F_W + wid)) & 1u) break;
      __builtin_amdgcn_s_sleep(4);
    }
    CBAR();
    int qo[8];
#pragma unroll
    for (int qd = 0; qd < 8; ++qd) qo[qd] = ((qd + cu) & 7) * 4096;
    const int vb = ((lane >> 3) & 1) * 1024 + (lane >> 4) * 128 + (lane & 7) * 16;
    u4 breg[18][2];
    char* ldsb = smem + L_B + wid * (NLDSK * 2048) + vl;
#pragma unroll
    for (int hf = 0; hf < 2; ++hf) {
      const int so = wbase + cu * 262144 + wid * 65536;
      u4 tmp[7][2];
#pragma unroll
      for (int t = 0; t < 16; ++t) {
#pragma unroll
        for (int m = 0; m < 2; ++m) {
          const u4 v = ld_sys(rws, vb + m * 2048 + (t & 1) * 512, so + 2 * qo[hf * 4 + (t >> 2)] + ((t >> 1) & 1) * 4096);
          if (k_in_lds(t)) tmp[t >> 1][m] = v; else breg[reg_index(hf * 16 + t)][m] = v;
        }
      }
#pragma unroll
      for (int t = 0; t < 7; ++t) {
#pragma unroll
        for (int m = 0; m < 2; ++m) *(u4*)(ldsb + (hf * 7 + t) * 2048 + m * 1024) = tmp[t][m];
      }
    }

    u4 ring[16];
    u4 bt0 = *(const u4*)ldsb, bt1 = *(const u4*)(ldsb + 1024);
    int va = vl + wid * 32768;
    int cur = (int)WS_USED;
    for (int n = 0; n < nneed; ++n) {
      if (cur == (int)WS_USED) {
        for (;;) {
          cur = __builtin_amdgcn_readfirstlane(lds_ld(ctl + C_QOFF + n));
          if (cur != 0) break;
          if ((__builtin_amdgcn_readfirstlane(lds_ld(ctl + C_MASK)) >> F_ABORT) & 1u) return;
          __builtin_amdgcn_s_sleep(1);
        }
        asm volatile("" : "+v"(va) :: "memory");
#pragma unroll
        for (int t = 0; t < 16; ++t) {
          ring[t] = ld_sys(rws, va + (t & 3) * 1024, cur + qo[t >> 2]);
          __builtin_amdgcn_sched_barrier(0);
        }
      }
      f4 acc0 = f4{0.f, 0.f, 0.f, 0.f}, acc1 = f4{0.f, 0.f, 0.f, 0.f};
      int nb = 0;
      unsigned r0 = 0;
      asm volatile("" : "+v"(va));
#pragma unroll
      for (int t = 0; t < 32; ++t) {
        if (t == 16) { nb = __builtin_amdgcn_readfirstlane(r0); nb = nb != 0 ? nb : (int)WS_USED; CBAR(); }
        const u4 b0 = k_in_lds(t) ? bt0 : breg[reg_index(t)][0];
        const u4 b1 = k_in_lds(t) ? bt1 : breg[reg_index(t)][1];
        const h8 a = __builtin_bit_cast(h8, ring[t & 15]);
        acc0 = __builtin_amdgcn_mfma_f32_16x16x32_f16(a, __builtin_bit_cast(h8, b0), acc0, 0, 0, 0);
        acc1 = __builtin_amdgcn_mfma_f32_16x16x32_f16(a, __builtin_bit_cast(h8, b1), acc1, 0, 0, 0);
        __builtin_amdgcn_sched_barrier(0);
        ring[t & 15] = ld_sys(rws, va + (t & 3) * 1024, (t < 16 ? cur : nb) + qo[((t + 16) & 31) >> 2]);
        if (k_in_lds(t)) {
          const int nn = (lds_index(t) + 1) % NLDSK;
          bt0 = *(const u4*)(ldsb + nn * 2048);
          bt1 = *(const u4*)(ldsb + nn * 2048 + 1024);
        }
        if (t == 14) r0 = lds_ld(ctl + C_QOFF + n + 1);
        if (t == 30) r0 = lds_ld(ctl + C_RED);
        __builtin_amdgcn_sched_barrier(0);
      }
      int red = __builtin_amdgcn_readfirstlane(r0);
      while (red < n - 1) {
        __builtin_amdgcn_s_sleep(1);
        if ((__builtin_amdgcn_readfirstlane(lds_ld(ctl + C_MASK)) >> F_ABORT) & 1u) return;
        red = __builtin_amdgcn_readfirstlane(lds_ld(ctl + C_RED));
      }
      CBAR();
      char* pp = smem + L_PART + (n & 1) * 8192 + wid * 2048 + vl;
      *(f4*)pp = acc0;
      *(f4*)(pp + 1024) = acc1;
      CBAR();
      if (lane == 0) __hip_atomic_fetch_add(ctl + C_PCNT + (n & 1), 1u, __ATOMIC_RELAXED, __HIP_MEMORY_SCOPE_WORKGROUP);
      CBAR();
      cur = nb;
    }
  }
}

}

extern "C" void kernel_launch(int rank, int nranks, ncclComm_t comm,
                              hipStream_t stream,
                              void* const* d_in, const int* in_sizes, int n_in,
                              void* d_out, int out_size,
                              const int* in_dtypes, int out_dtype,
                              void* const* d_out_all,
                              void* const* d_ws_all, size_t ws_bytes) {
  if (nranks != NR || ws_bytes < (size_t)WS_USED + (1u << 20) || n_in < 2 || (unsigned)rank >= (unsigned)NR) {
    fprintf(stderr, "unsupported configuration\n");
    return;
  }
  Params p;
  p.x = (const float*)d_in[0];
  p.w = (const float*)d_in[1];
  for (int d = 0; d < NR; ++d) {
    p.ws[d] = (char*)d_ws_all[rank ^ d];
    p.out[d] = (float*)d_out_all[rank ^ d];
  }
  p.rank = rank;
  hipLaunchKernelGGL(ag_gemm_relu, dim3(256), dim3(512), 0, stream, p);
  HIP_CHECK(hipGetLastError());
}
```

```cpp
#include <hip/hip_runtime.h>
#include <rccl/rccl.h>
#include <cstdio>

#define HIP_CHECK(expr) do { hipError_t _e = (expr); if (_e != hipSuccess) { \
    fprintf(stderr, "HIP error %s at %s:%d\n", hipGetErrorString(_e), __FILE__, __LINE__); \
    return; } } while (0)

namespace {

typedef float f4 __attribute__((ext_vector_type(4)));
typedef unsigned u4 __attribute__((ext_vector_type(4)));
typedef _Float16 h8 __attribute__((ext_vector_type(8)));
typedef unsigned long long u64;
typedef __amdgpu_buffer_rsrc_t rsrc_t;

constexpr int NR = 8;
constexpr int NG = 7;
constexpr int ND = 22;
constexpr int NXB = 32 - ND;
constexpr int S1 = 32;
constexpr int S3 = 16;
constexpr int ONE = 96;
constexpr int S0 = S1 + ONE;
constexpr int NSTEP = ND * 32;
constexpr int NWST = 256;
constexpr int E2 = NWST + NXB * 32;
constexpr int GATE_STEP = E2 - 12;
constexpr int QUIT_STEP = E2 - 16;
constexpr int FWIN = 2;
constexpr int T_STAGE = 110, T_STIER = 42, T_TIER = 120;
constexpr int RAWTAIL = 3;
static_assert(ND >= 20 && ND <= 24, "slot table");
static_assert(S3 > 0 && S3 <= S1 && ONE == 96 && S0 <= NWST && GATE_STEP >= 0 && GATE_STEP < NSTEP, "sender configuration");

constexpr u64 DEADLINE = 100000000ull;

constexpr unsigned OFF_PL = 0;
constexpr unsigned OFF_TMO = 40960;
constexpr unsigned OFF_CNT = 45056;
constexpr unsigned OFF_CTR = 65536;
constexpr unsigned OFF_SL = 98304;
constexpr unsigned OFF_RP = 110592;
constexpr unsigned OFF_X = 1u << 20;
constexpr unsigned OFF_W = 65u << 20;
constexpr unsigned OFF_T = 81u << 20;
constexpr unsigned WS_USED = 87u << 20;
constexpr int TBYTES = 49152;
static_assert(112u * TBYTES <= WS_USED - OFF_T && 22 * 2176 <= TBYTES, "tile region");
constexpr int CI_TICK = 0;
constexpr int CI_SX = 4;
constexpr int CI_SW = 36;
constexpr int CI_LX = 52;
constexpr int CI_LW = 84;
constexpr int CI_END = 148;
constexpr int NSLOT = 25;
constexpr int F_W = 25;
constexpr int F_GATE = 29;
constexpr int F_ABORT = 31;

constexpr int NLDSK = 14;
constexpr int NBACK = 15;
constexpr int NBOWN = 8;
constexpr int BPITCH = 2144;
constexpr int L_B = 0;
constexpr int L_PART = 4 * NLDSK * 2048;
constexpr int L_BACK = L_PART + 16384;
constexpr int L_CTL = L_BACK + NBACK * BPITCH;
constexpr int C_QOFF = 0;
constexpr int C_QSLOT = 24;
constexpr int C_PCNT = 48;
constexpr int C_RED = 50;
constexpr int C_FL = 51;
constexpr int C_MASK = 52;
constexpr int C_WORDS = 64;
constexpr int L_EXP = L_BACK + NBOWN * BPITCH;
constexpr int LDS_BYTES = L_CTL + C_WORDS * 4;
static_assert(LDS_BYTES <= 163840, "LDS");

__host__ __device__ constexpr int phys(int l) { return (0x75416320 >> (4 * l)) & 7; }

struct Params {
  const float* x;
  const float* w;
  char* ws[NR];
  float* out[NR];
  int rank;
};

#define WAITV0() do { asm volatile("" ::: "memory"); __builtin_amdgcn_s_waitcnt(0x0F70); asm volatile("" ::: "memory"); } while (0)
#define CBAR() asm volatile("" ::: "memory")
constexpr int OOB = 0x40000000;

__device__ __forceinline__ rsrc_t mk_rsrc(const void* p, unsigned bytes) {
  return __builtin_amdgcn_make_buffer_rsrc(const_cast<void*>(p), 0, bytes, 0x00020000);
}
__device__ __forceinline__ f4 ld_in(rsrc_t r, int voff, int soff) {
  return __builtin_bit_cast(f4, __builtin_amdgcn_raw_buffer_load_b128(r, voff, soff, 0));
}
__device__ __forceinline__ u4 ld_sys(rsrc_t r, int voff, int soff) {
  return __builtin_amdgcn_raw_buffer_load_b128(r, voff, soff, 17);
}
__device__ __forceinline__ unsigned ld_sys32(rsrc_t r, int voff, int soff) {
  return __builtin_amdgcn_raw_buffer_load_b32(r, voff, soff, 17);
}
__device__ __forceinline__ void st_sys(u4 v, rsrc_t r, int voff, int soff) {
  __builtin_amdgcn_raw_buffer_store_b128(v, r, voff, soff, 17);
}
__device__ __forceinline__ void st_sys32(unsigned v, rsrc_t r, int voff, int soff) {
  __builtin_amdgcn_raw_buffer_store_b32(v, r, voff, soff, 17);
}
__device__ __forceinline__ u4 cvt8(f4 a, f4 b) {
  h8 h = h8{(_Float16)a.x, (_Float16)a.y, (_Float16)a.z, (_Float16)a.w,
            (_Float16)b.x, (_Float16)b.y, (_Float16)b.z, (_Float16)b.w};
  return __builtin_bit_cast(u4, h);
}
__device__ __forceinline__ u4 cvtcol(const f4* r, int i) {
  h8 h = h8{(_Float16)r[0][i], (_Float16)r[1][i], (_Float16)r[2][i], (_Float16)r[3][i],
            (_Float16)r[4][i], (_Float16)r[5][i], (_Float16)r[6][i], (_Float16)r[7][i]};
  return __builtin_bit_cast(u4, h);
}
__device__ __forceinline__ void load_xpiece(f4* r, rsrc_t rx, int vx, int pi) {
  const int so = (pi >> 5) * (16 * 16384) + (pi & 31) * 512;
#pragma unroll
  for (int t = 0; t < 8; ++t) r[t] = ld_in(rx, vx + t * 64, so);
}
__device__ __forceinline__ void load_wgran(f4* r, rsrc_t rw, int vw, int cg, int kg) {
  const int so = kg * (64 * 2048) + cg * 128;
#pragma unroll
  for (int j = 0; j < 8; ++j) r[j] = ld_in(rw, vw + (j & 1) * 2048, so + (16 * (j >> 2) + (j & 2)) * 2048);
}
__device__ __forceinline__ void pause_ticks(int ticks) {
  const u64 t0 = __builtin_amdgcn_s_memrealtime();
  while (__builtin_amdgcn_s_memrealtime() - t0 < (u64)ticks) __builtin_amdgcn_s_sleep(2);
}
__device__ __forceinline__ unsigned lds_ld(const unsigned* p) {
  return __hip_atomic_load(p, __ATOMIC_RELAXED, __HIP_MEMORY_SCOPE_WORKGROUP);
}
__device__ __forceinline__ void lds_st(unsigned* p, unsigned v) {
  __hip_atomic_store(p, v, __ATOMIC_RELAXED, __HIP_MEMORY_SCOPE_WORKGROUP);
}

__device__ __forceinline__ int src_off(int d) { return (0x65314201 >> (4 * d)) & 7; }
__device__ __forceinline__ int src_sbase(int d) { return d == 0 ? 0 : d < 4 ? 1 + 4 * d : 9 + 2 * d; }
__device__ __forceinline__ int src_lo(int d) { return d < 4 ? 0 : ND; }
__device__ __forceinline__ int flag_off(int d, int b) {
  return (((b + src_off(d)) % NG) * 32 + src_sbase(d) + (b - src_lo(d)) / NG) * 4;
}
__device__ __forceinline__ bool slot_block(int slot, int g, bool helper, int q, int& d, int& b) {
  d = slot < 5 ? 0 : slot < 17 ? (slot - 1) >> 2 : ((slot - 9) >> 1) & 7;
  const int lo = src_lo(d);
  const int hi = d == 0 ? (helper ? ND : 32) : d < 4 ? ND : ((helper && d == 4 + q) ? 0 : 32);
  b = lo + (g + 70 - src_off(d) - lo) % NG + NG * (slot - src_sbase(d));
  return slot < NSLOT && b < hi;
}
__device__ __forceinline__ void stage_local(int gi, unsigned call, rsrc_t rws, rsrc_t rx, rsrc_t rw, rsrc_t rcnt, int vx, int vw, int lane) {
  f4 sx[8], sw[8];
  load_xpiece(sx, rx, vx, gi);
  load_wgran(sw, rw, vw, gi >> 6, gi & 63);
  const int xbase = OFF_X + (call & 1u) * (32u << 20);
  const int wbase = OFF_W + (call & 1u) * (8u << 20);
#pragma unroll
  for (int t = 0; t < 4; ++t) st_sys(cvtcol(sw, t), rws, lane * 16 + t * 1024, wbase + gi * 4096);
#pragma unroll
  for (int t = 0; t < 4; ++t) st_sys(cvt8(sx[2 * t], sx[2 * t + 1]), rws, lane * 16 + t * 1024, xbase + gi * 4096);
  WAITV0();
  const int tv = __builtin_amdgcn_raw_ptr_buffer_atomic_add_i32(1, rcnt, (lane == 0 ? CI_LX + (gi >> 5) : lane == 1 ? CI_LW + (gi >> 4) : CI_END) * 64, 0, 0);
  if ((__builtin_amdgcn_readlane(tv, 0) & 31) == 31)
    st_sys32(call, rws, lane < 8 ? lane * 1024 : OOB, OFF_SL + flag_off(0, gi >> 5));
  if ((__builtin_amdgcn_readlane(tv, 1) & 15) == 15)
    st_sys32(call, rws, lane < NG ? lane * 4096 : OOB, OFF_PL + (gi >> 6) * 128 + (F_W + ((gi >> 4) & 3)) * 4);
}

constexpr bool k_in_lds(int t) { return ((t & 15) & 1) && (t & 15) != 15; }
constexpr int lds_index(int t) { return (t >> 4) * 7 + ((t & 15) >> 1); }
constexpr int reg_index(int t) { return (t >> 4) * 9 + ((t & 15) == 15 ? 8 : ((t & 15) >> 1)); }

__global__ void __launch_bounds__(512, 2) ag_gemm_relu(Params p) {
  __shared__ __attribute__((aligned(16))) char smem[LDS_BYTES];
  unsigned* ctl = (unsigned*)(smem + L_CTL);
  const int lane = threadIdx.x & 63;
  const int wid = __builtin_amdgcn_readfirstlane(threadIdx.x >> 6);
  const bool comm = blockIdx.x < 32;
  const int hidx = (blockIdx.x & 3) * 28 + (blockIdx.x >> 3) - 4;
  const int cidx = (hidx >> 4) * 32 + (hidx & 15) + (blockIdx.x & 4) * 4;
  const int g = cidx >> 5;
  const int cu = cidx & 31;
  const bool helper = cu >= 16;
  const int q = (cu >> 2) & 3;
  const rsrc_t rws = mk_rsrc(p.ws[0], WS_USED);
  const rsrc_t rx = mk_rsrc(p.x, 512u * 4096u * 4u);
  const rsrc_t rw = mk_rsrc(p.w, 4096u * 512u * 4u);
  const rsrc_t rcnt = mk_rsrc(p.ws[0] + OFF_CNT, CI_END * 64);
  const int vx = (lane & 15) * 16384 + (lane >> 4) * 16;
  const int vw = (32 * (lane >> 5) + 4 * ((lane >> 3) & 3)) * 2048 + (lane & 7) * 16;
  const int vl = lane * 16;
  const int v0 = lane == 0 ? 0 : OOB;

  const unsigned cprev = ld_sys32(rws, 0, OFF_CTR + blockIdx.x * 64);

  const int j = blockIdx.x;
  bool sent = false;
  if (__builtin_expect(comm ? wid == 0 : wid == 6 && j < S0, 1)) {
    f4 rd[8], ra[8], rb[24];
    load_xpiece(rd, rx, vx, j);
    load_wgran(ra, rw, vw, j & 3, j >> 2);
#pragma unroll
    for (int t = 0; t < 3; ++t) load_wgran(rb + 8 * t, rw, vw, 4 * (t + 1) + (j & 3), j >> 2);
    asm volatile("" :: "s"(p.ws[1]), "s"(p.ws[2]), "s"(p.ws[3]), "s"(p.ws[4]), "s"(p.ws[5]), "s"(p.ws[6]), "s"(p.ws[7]));
    const unsigned call = __builtin_amdgcn_readfirstlane(cprev) + 1u;
    asm volatile("" :: "s"(call) : "memory");
    __builtin_amdgcn_s_barrier();
    CBAR();
    __builtin_amdgcn_s_setprio(2);
    const unsigned par = call & 1u;
    rsrc_t rp[NR];
#pragma unroll
    for (int d = 1; d < NR; ++d) rp[d] = mk_rsrc(p.ws[d], WS_USED);
    const int xbase = OFF_X + par * (32u << 20);
    const int wbase = OFF_W + par * (8u << 20);
    const int vfl = lane < 8 ? lane * 1024 : OOB;
    const int vgl = lane < NG ? lane * 4096 : OOB;
    const int js = (j & 4) * 4 + (j >> 3) * 4 + (j & 3);
    const int qstep = !comm ? 0 : js >= S3 ? QUIT_STEP : NSTEP;
    int i = j, prev = -1;
    WAITV0();
    if (j >= S1 + 8) pause_ticks(((j - S1) >> 3) * T_TIER);
    for (;;) {
      const bool quit = i >= qstep;
      int cv = CI_END;
      if (lane == 0 && !quit) cv = CI_TICK + 2 * par;
      if (lane == 1 && prev >= 0) cv = CI_SX + (prev >> 5);
      if (lane == 2 && prev >= 0 && prev < E2) cv = prev < NWST ? CI_SW + (prev >> 6) * 4 + (prev & 3) : CI_SX + ND + ((prev - NWST) >> 5);
      const int tv = __builtin_amdgcn_raw_ptr_buffer_atomic_add_i32(1, rcnt, cv * 64, 0, 0);
      int t0v, t1v, t2v;
#define READ_BACK() do { t0v = __builtin_amdgcn_readlane(tv, 0); t1v = __builtin_amdgcn_readlane(tv, 1); t2v = __builtin_amdgcn_readlane(tv, 2); } while (0)
      if (i == GATE_STEP) {
        st_sys32(call, rws, (lane >> 4) * 4096 + (lane & 15) * 128, OFF_PL + 16 * 128 + F_GATE * 4);
        st_sys32(call, rws, lane < 48 ? (lane >> 4) * 4096 + (lane & 15) * 128 : OOB, OFF_PL + 4 * 4096 + 16 * 128 + F_GATE * 4);
      }
      if (i >= NSTEP) {
        READ_BACK();
      } else if (i < NWST) {
        const int wo = wbase + (16 + (i & 3)) * 262144 + (i >> 2) * 4096;
#pragma unroll
        for (int t = 0; t < 4; ++t) {
          const u4 h = cvt8(rd[2 * t], rd[2 * t + 1]);
#pragma unroll
          for (int d = 1; d < 4; ++d) st_sys(h, rp[d], vl + t * 1024, xbase + d * (4 << 20) + i * 4096);
          st_sys(cvtcol(ra, t), rp[4], vl + t * 1024, wo);
#pragma unroll
          for (int d = 5; d < 8; ++d) st_sys(cvtcol(rb + 8 * (d - 5), t), rp[d], vl + t * 1024, wo + (d - 4) * (4 * 262144));
        }
        READ_BACK();
      } else if (i < E2) {
        const int xi = NSTEP + i - NWST;
#pragma unroll
        for (int t = 0; t < 4; ++t) {
          const u4 h = cvt8(rd[2 * t], rd[2 * t + 1]);
          const u4 gg = cvt8(ra[2 * t], ra[2 * t + 1]);
#pragma unroll
          for (int d = 1; d < 4; ++d) st_sys(h, rp[d], vl + t * 1024, xbase + d * (4 << 20) + i * 4096);
#pragma unroll
          for (int d = 4; d < 8; ++d) st_sys(gg, rp[d], vl + t * 1024, xbase + d * (4 << 20) + xi * 4096);
        }
        READ_BACK();
      } else {
#pragma unroll
        for (int t = 0; t < 4; ++t) {
          const u4 h = cvt8(rd[2 * t], rd[2 * t + 1]);
#pragma unroll
          for (int d = 1; d < 4; ++d) st_sys(h, rp[d], vl + t * 1024, xbase + d * (4 << 20) + i * 4096);
        }
        READ_BACK();
      }
      if (prev >= 0) {
        if ((t1v & 31) == 31) {
#pragma unroll
          for (int d = 1; d < 4; ++d) st_sys32(call, rp[d], vfl, OFF_SL + flag_off(d, prev >> 5));
        }
        if (prev < NWST) {
          if ((t2v & 15) == 15) {
#pragma unroll
            for (int d = 4; d < 8; ++d)
              st_sys32(call, rp[d], vgl, OFF_PL + (16 + 4 * (d - 4) + (prev & 3)) * 128 + (F_W + (prev >> 6)) * 4);
          }
        } else if (prev < E2) {
          if ((t2v & 31) == 31) {
#pragma unroll
            for (int d = 4; d < 8; ++d) st_sys32(call, rp[d], vfl, OFF_SL + flag_off(d, ND + ((prev - NWST) >> 5)));
          }
        }
      }
      if (i >= NSTEP) break;
      int nxt = t0v + S0;
      if (quit || (unsigned)nxt >= (unsigned)NSTEP) nxt = NSTEP;
      if (nxt < NSTEP) {
        load_xpiece(rd, rx, vx, nxt);
        if (nxt < NWST) {
          load_wgran(ra, rw, vw, nxt & 3, nxt >> 2);
#pragma unroll
          for (int t = 0; t < 3; ++t) load_wgran(rb + 8 * t, rw, vw, 4 * (t + 1) + (nxt & 3), nxt >> 2);
        } else if (nxt < E2) {
          load_xpiece(ra, rx, vx, NSTEP + nxt - NWST);
        }
      }
      WAITV0();
      prev = i;
      i = nxt;
    }
    if (comm || helper) return;
    sent = true;
  }

  if (comm || wid == 6) {
    const unsigned call = __builtin_amdgcn_readfirstlane(cprev) + 1u;
    asm volatile("" :: "s"(call) : "memory");
    if (!sent) __builtin_amdgcn_s_barrier();
    CBAR();
    if (comm) {
      if (wid == 7) {
        st_sys32(call, rws, v0, OFF_CTR + blockIdx.x * 64);
        if (blockIdx.x == 31) st_sys32(0u, rcnt, v0, (CI_TICK + 2 * ((call & 1u) ^ 1u)) * 64);
      }
      return;
    }
    if (!sent) {
      pause_ticks(T_STAGE + ((blockIdx.x >> 3) - 4) * T_STIER);
      stage_local(896 + j - S0, call, rws, rx, rw, rcnt, vx, vw, lane);
    }
    if (helper) return;
    __builtin_amdgcn_s_setprio(2);
    const int z = q, sub = cu & 3, eg = g, e = z * 28 + sub * NG + g;
    int pd, pb;
    unsigned ppend = (unsigned)__ballot(slot_block(lane, eg, true, z, pd, pb));
    const int prow = (phys((p.rank ^ (4 + z) ^ pd) & 7) * 512 + pb * 16) * 2048;
    const rsrc_t ro = mk_rsrc(p.out[0], 4096u * 512u * 4u);
    const int vo = (lane >> 3) * 2048 + (lane & 7) * 16 + (4 * z + sub) * 128;
    const int tb = OFF_T + e * TBYTES;
    char* sc = smem + L_EXP;
    const int jr = (lane >> 3) & 3;
    const int sl0 = (lane >> 5) * 16 + (lane & 7);
    const int pv = lane == 0 ? (int)(OFF_RP + e * 128) : lane == 2 ? (int)OFF_TMO : OOB;
    int rl = 0, tried = 0, oc = 0;
    bool abort = false;
    const u64 t0 = __builtin_amdgcn_s_memrealtime();
    for (;;) {
      const unsigned fv = ld_sys32(rws, pv, 0);
      const unsigned ow = __builtin_amdgcn_readlane(fv, 0) - (call << 14);
      if (ow < 1024u && (int)ow > oc) oc = (int)ow;
      const int avail = (oc & 511) - rl;
      if (avail > tried) {
        const int got = avail < 64 ? avail : 64;
        u4 t[8];
#pragma unroll
        for (int y = 0; y < 8; ++y) t[y] = ld_sys(rws, (lane >> 3) + 8 * y < got ? vl : OOB, tb + rl * 128 + y * 1024);
#pragma unroll
        for (int y = 0; y < 8; ++y) *(u4*)(sc + y * 1024 + vl) = t[y];
        CBAR();
        int at = 0;
        while (at < got) {
          const char* h = sc + at * 128;
          const unsigned hd = *(const unsigned*)(h + 80 + (lane & 1) * 4);
          const u64 mk0 = *(const u64*)(h + jr * 8), mk1 = *(const u64*)(h + 32 + jr * 8);
          const int b0 = *(const unsigned short*)(h + 64 + jr * 2), b1 = *(const unsigned short*)(h + 72 + jr * 2);
          const int nl = (96 + 127 + 4 * (__builtin_amdgcn_readlane(hd, 1) & 1023)) >> 7;
          if (at + nl > got) break;
          const int sl = __builtin_amdgcn_readlane(hd, 0) & 31;
          const int ro_s = __builtin_amdgcn_readlane(prow, sl);
          ppend &= ~(1u << sl);
          unsigned o[2][4];
#pragma unroll
          for (int m = 0; m < 2; ++m) {
            const u64 mk = m ? mk1 : mk0;
            const int base = m ? b1 : b0;
#pragma unroll
            for (int hh = 0; hh < 2; ++hh) {
#pragma unroll
              for (int i1 = 0; i1 < 2; ++i1) {
                const int s = sl0 + hh * 32 + i1 * 8;
                const int idx = (base + __builtin_popcountll(mk & ((1ull << s) - 1ull))) & 511;
                const unsigned v = *(const unsigned*)(h + 96 + idx * 4);
                o[hh][m * 2 + i1] = ((mk >> s) & 1ull) ? v : 0u;
              }
            }
          }
          st_sys(u4{o[0][0], o[0][1], o[0][2], o[0][3]}, ro, vo, ro_s);
          st_sys(u4{o[1][0], o[1][1], o[1][2], o[1][3]}, ro, vo, ro_s + 16384);
          at += nl;
        }
        CBAR();
        tried = got == avail ? got - at : 0;
        if (tried == 0 && at == 0) { abort = true; break; }
        rl += at;
        continue;
      }
      if (avail == 0 && (oc & 512) != 0) break;
      if (__builtin_amdgcn_readlane(fv, 2) == call || __builtin_amdgcn_s_memrealtime() - t0 > DEADLINE) { abort = true; break; }
      if (oc == 0) __builtin_amdgcn_s_sleep(16); else __builtin_amdgcn_s_sleep(2);
    }
    if (abort) {
      st_sys32(call, rws, v0, OFF_TMO);
      ppend |= 1u;
      WAITV0();
      while (ppend != 0u) {
        const int ro_s = __builtin_amdgcn_readlane(prow, __builtin_ctz(ppend));
        ppend &= ppend - 1u;
        const u4 nan4 = u4{0x7fc00000u, 0x7fc00000u, 0x7fc00000u, 0x7fc00000u};
        st_sys(nan4, ro, vo, ro_s);
        st_sys(nan4, ro, vo, ro_s + 16384);
      }
    }
    return;
  }

  int sd, sb;
  const bool sused = slot_block(lane, g, helper, q, sd, sb);
  const unsigned needmask = (unsigned)__ballot(sused);
  const int nneed = __builtin_popcount(needmask);

  if (wid >= 4) {
    if (wid == 4) lds_st(ctl + lane, 0u);
    const unsigned call = __builtin_amdgcn_readfirstlane(cprev) + 1u;
    asm volatile("s_waitcnt lgkmcnt(0)" :: "s"(call) : "memory");
    __builtin_amdgcn_s_barrier();
    CBAR();
    __builtin_amdgcn_s_setprio(2);
    const int npack = !helper ? 0 : nneed > RAWTAIL ? nneed - RAWTAIL : 0;

    if (wid == 4) {
      const unsigned want = needmask | (15u << F_W) | (helper ? 1u << F_GATE : 0u);
      const int pv = lane < NSLOT ? (int)(OFF_SL + ((cu >> 2) * 8 + g) * 128 + lane * 4)
                   : lane < 32 ? (int)(OFF_PL + cidx * 128 + lane * 4) : lane == 32 ? (int)OFF_TMO : OOB;
      const unsigned boff = OFF_X + (call & 1u) * (32u << 20) + sd * (4 << 20) + sb * 131072;
      unsigned seen = 0;
      const u64 t0 = __builtin_amdgcn_s_memrealtime();
      for (;;) {
        const unsigned fv = ld_sys32(rws, pv, 0);
        const unsigned m = (unsigned)__ballot((int)(fv - call) >= 0) & want;
        if (m != seen) {
          const unsigned fresh = m & ~seen & needmask;
          if (lane < 32 && ((fresh >> lane) & 1u)) {
            const int pos = __builtin_popcount(seen & needmask) + __builtin_popcount(fresh >> lane) - 1;
            lds_st(ctl + C_QSLOT + pos, (unsigned)lane);
            lds_st(ctl + C_QOFF + pos, boff);
          }
          if (lane == 0) lds_st(ctl + C_MASK, m);
          seen = m;
          if (m == want) break;
        }
        if (__builtin_amdgcn_readlane(fv, 32) == call || __builtin_amdgcn_s_memrealtime() - t0 > DEADLINE) {
          st_sys32(call, rws, v0, OFF_TMO);
          if (lane == 0) lds_st(ctl + C_MASK, seen | (1u << F_ABORT));
          break;
        }
        __builtin_amdgcn_s_sleep(1);
      }
    } else if (wid == 5) {
      unsigned pending = needmask;
      const int rowoff = (phys((p.rank ^ sd) & 7) * 512 + sb * 16) * 2048;
      const int vo = (lane >> 3) * 2048 + (lane & 7) * 16 + (cu & 15) * 128;
      const rsrc_t ro = mk_rsrc(!helper ? p.out[0] : q == 0 ? p.out[4] : q == 1 ? p.out[5] : q == 2 ? p.out[6] : p.out[7], 4096u * 512u * 4u);
      const int e = q * 28 + (cu & 3) * NG + g;
      const rsrc_t rq = mk_rsrc(q == 0 ? p.ws[4] : q == 1 ? p.ws[5] : q == 2 ? p.ws[6] : p.ws[7], WS_USED);
      const int fo = OFF_RP + e * 128;
      const int tb = OFF_T + e * TBYTES;
      const int nb = helper ? NBACK : NBOWN;
      int bi = 0, wl = 0, told = 0;
      bool abort = false, word_last = false;
      for (int n = 0; n < nneed;) {
        if (word_last) asm volatile("s_waitcnt vmcnt(1)" ::: "memory");
        else if (helper) asm volatile("s_waitcnt vmcnt(0)" ::: "memory");
        const int arrived = wl;
        unsigned c;
        int red;
        for (;;) {
          c = lds_ld(ctl + lane);
          const unsigned mk = __builtin_amdgcn_readlane(c, C_MASK);
          red = __builtin_amdgcn_readlane(c, C_RED);
          abort = (mk >> F_ABORT) & 1u;
          const bool hold = n >= npack && red < nneed && (mk & needmask) == needmask;
          if (abort || (red > n && (!helper || (((mk >> F_GATE) & 1u) && !hold)))) break;
          if (told != arrived) { st_sys32((call << 14) | (unsigned)arrived, rq, v0, fo); told = arrived; }
          __builtin_amdgcn_s_sleep(1);
        }
        if (abort) break;
        CBAR();
        const int nend = red < n + FWIN || n >= npack ? red : n + FWIN;
        for (; n < nend; ++n) {
          const int sl = __builtin_amdgcn_readlane(c, C_QSLOT + n);
          pending &= ~(1u << sl);
          const char* eb = smem + L_BACK + bi * BPITCH;
          const u4 o0 = *(const u4*)(eb + vl);
          const u4 o1 = *(const u4*)(eb + 1024 + vl);
          if (n >= npack) {
            const int ro_s = __builtin_amdgcn_readlane(rowoff, sl);
            st_sys(o0, ro, vo, ro_s);
            st_sys(o1, ro, vo, ro_s + 16384);
          } else {
            const u4 o2 = *(const u4*)(eb + 2048 + (lane & 7) * 16);
            const int nl = (96 + 127 + 4 * __builtin_amdgcn_readlane(o0.y, 5)) >> 7;
            st_sys(o0, rq, lane < 8 * nl ? vl : OOB, tb + wl * 128);
            if (nl > 8) st_sys(o1, rq, lane + 64 < 8 * nl ? vl : OOB, tb + wl * 128 + 1024);
            if (nl > 16) st_sys(o2, rq, lane < 8 ? vl : OOB, tb + wl * 128 + 2048);
            wl += nl;
          }
          bi = bi + 1 == nb ? 0 : bi + 1;
        }
        CBAR();
        word_last = told != arrived;
        if (word_last) { st_sys32((call << 14) | (unsigned)arrived, rq, v0, fo); told = arrived; }
        if (lane == 0) lds_st(ctl + C_FL, (unsigned)n);
      }
      if (helper || abort) WAITV0();
      if (abort) {
        pending |= 1u;
        while (pending != 0u) {
          const int ro_s = __builtin_amdgcn_readlane(rowoff, __builtin_ctz(pending));
          pending &= pending - 1u;
          const u4 nan4 = u4{0x7fc00000u, 0x7fc00000u, 0x7fc00000u, 0x7fc00000u};
          st_sys(nan4, ro, vo, ro_s);
          st_sys(nan4, ro, vo, ro_s + 16384);
        }
      } else if (helper) {
        st_sys32((call << 14) | 512u | (unsigned)wl, rq, v0, fo);
      }
    } else {
      st_sys32(call, rws, v0, OFF_CTR + blockIdx.x * 64);
      const int wo = (lane >> 4) * 512 + (lane & 7) * 16 + ((lane >> 3) & 1) * 4;
      const int nb = helper ? NBACK : NBOWN;
      int bi = 0;
      for (int n = 0; n < nneed; ++n) {
        bool abort;
        unsigned c;
        for (;;) {
          c = lds_ld(ctl + lane);
          abort = (__builtin_amdgcn_readlane(c, C_MASK) >> F_ABORT) & 1u;
          const int pc = (n & 1) ? __builtin_amdgcn_readlane(c, C_PCNT + 1) : __builtin_amdgcn_readlane(c, C_PCNT);
          if (abort || (pc >= 4 * ((n >> 1) + 1) && n - (int)__builtin_amdgcn_readlane(c, C_FL) < nb)) break;
          __builtin_amdgcn_s_sleep(1);
        }
        if (abort) break;
        CBAR();
        const char* pp = smem + L_PART + (n & 1) * 8192 + vl;
        char* e = smem + L_BACK + bi * BPITCH;
        float v[8];
#pragma unroll
        for (int m = 0; m < 2; ++m) {
          const f4 s = (*(const f4*)(pp + m * 1024) + *(const f4*)(pp + 2048 + m * 1024)) +
                       (*(const f4*)(pp + 4096 + m * 1024) + *(const f4*)(pp + 6144 + m * 1024));
#pragma unroll
          for (int jj = 0; jj < 4; ++jj) v[m * 4 + jj] = s[jj] < 0.f ? 0.f : s[jj];
        }
        if (n >= npack) {
#pragma unroll
          for (int r = 0; r < 8; ++r) *(float*)(e + wo + (r & 3) * 128 + (r >> 2) * 8) = v[r];
        } else {
          int base = 0, hb = 0;
          u64 hm = 0;
#pragma unroll
          for (int r = 0; r < 8; ++r) {
            const bool nz = __builtin_bit_cast(unsigned, v[r]) != 0u;
            const u64 mk = __ballot(nz);
            const int pos = base + __builtin_amdgcn_mbcnt_hi((unsigned)(mk >> 32), __builtin_amdgcn_mbcnt_lo((unsigned)mk, 0u));
            if (nz) *(float*)(e + 96 + pos * 4) = v[r];
            if (lane == r) { hm = mk; hb = base; }
            base += __builtin_popcountll(mk);
          }
          if (lane < 8) { *(u64*)(e + lane * 8) = hm; *(unsigned short*)(e + 64 + lane * 2) = (unsigned short)hb; }
          if (lane < 2) *(unsigned*)(e + 80 + lane * 4) = lane == 0 ? (unsigned)__builtin_amdgcn_readlane(c, C_QSLOT + n) : (unsigned)base;
        }
        CBAR();
        if (lane == 0) lds_st(ctl + C_RED, (unsigned)(n + 1));
        bi = bi + 1 == nb ? 0 : bi + 1;
      }
    }
    return;
  }

  {
    const unsigned call = __builtin_amdgcn_readfirstlane(cprev) + 1u;
    asm volatile("" :: "s"(call) : "memory");
    __builtin_amdgcn_s_barrier();
    CBAR();
    pause_ticks(T_STAGE + ((blockIdx.x >> 3) - 4) * T_STIER);
    stage_local(cidx * 4 + wid, call, rws, rx, rw, rcnt, vx, vw, lane);
    const int wbase = OFF_W + (call & 1u) * (8u << 20);

    for (;;) {
      const unsigned mk = __builtin_amdgcn_readfirstlane(lds_ld(ctl + C_MASK));
      if ((mk >> F_ABORT) & 1u) return;
      if ((mk >> (F_W + wid)) & 1u) break;
      __builtin_amdgcn_s_sleep(4);
    }
    CBAR();
    int qo[8];
#pragma unroll
    for (int qd = 0; qd < 8; ++qd) qo[qd] = ((qd + cu) & 7) * 4096;
    const int vb = ((lane >> 3) & 1) * 1024 + (lane >> 4) * 128 + (lane & 7) * 16;
    u4 breg[18][2];
    char* ldsb = smem + L_B + wid * (NLDSK * 2048) + vl;
#pragma unroll
    for (int hf = 0; hf < 2; ++hf) {
      const int so = wbase + cu * 262144 + wid * 65536;
      u4 tmp[7][2];
#pragma unroll
      for (int t = 0; t < 16; ++t) {
#pragma unroll
        for (int m = 0; m < 2; ++m) {
          const u4 v = ld_sys(rws, vb + m * 2048 + (t & 1) * 512, so + 2 * qo[hf * 4 + (t >> 2)] + ((t >> 1) & 1) * 4096);
          if (k_in_lds(t)) tmp[t >> 1][m] = v; else breg[reg_index(hf * 16 + t)][m] = v;
        }
      }
#pragma unroll
      for (int t = 0; t < 7; ++t) {
#pragma unroll
        for (int m = 0; m < 2; ++m) *(u4*)(ldsb + (hf * 7 + t) * 2048 + m * 1024) = tmp[t][m];
      }
    }

    u4 ring[16];
    u4 bt0 = *(const u4*)ldsb, bt1 = *(const u4*)(ldsb + 1024);
    int va = vl + wid * 32768;
    int cur = (int)WS_USED;
    for (int n = 0; n < nneed; ++n) {
      if (cur == (int)WS_USED) {
        for (;;) {
          cur = __builtin_amdgcn_readfirstlane(lds_ld(ctl + C_QOFF + n));
          if (cur != 0) break;
          if ((__builtin_amdgcn_readfirstlane(lds_ld(ctl + C_MASK)) >> F_ABORT) & 1u) return;
          __builtin_amdgcn_s_sleep(1);
        }
        asm volatile("" : "+v"(va) :: "memory");
#pragma unroll
        for (int t = 0; t < 16; ++t) {
          ring[t] = ld_sys(rws, va + (t & 3) * 1024, cur + qo[t >> 2]);
          __builtin_amdgcn_sched_barrier(0);
        }
      }
      f4 acc0 = f4{0.f, 0.f, 0.f, 0.f}, acc1 = f4{0.f, 0.f, 0.f, 0.f};
      int nb = 0;
      unsigned r0 = 0;
      asm volatile("" : "+v"(va));
#pragma unroll
      for (int t = 0; t < 32; ++t) {
        if (t == 16) { nb = __builtin_amdgcn_readfirstlane(r0); nb = nb != 0 ? nb : (int)WS_USED; CBAR(); }
        const u4 b0 = k_in_lds(t) ? bt0 : breg[reg_index(t)][0];
        const u4 b1 = k_in_lds(t) ? bt1 : breg[reg_index(t)][1];
        const h8 a = __builtin_bit_cast(h8, ring[t & 15]);
        acc0 = __builtin_amdgcn_mfma_f32_16x16x32_f16(a, __builtin_bit_cast(h8, b0), acc0, 0, 0, 0);
        acc1 = __builtin_amdgcn_mfma_f32_16x16x32_f16(a, __builtin_bit_cast(h8, b1), acc1, 0, 0, 0);
        __builtin_amdgcn_sched_barrier(0);
        ring[t & 15] = ld_sys(rws, va + (t & 3) * 1024, (t < 16 ? cur : nb) + qo[((t + 16) & 31) >> 2]);
        if (k_in_lds(t)) {
          const int nn = (lds_index(t) + 1) % NLDSK;
          bt0 = *(const u4*)(ldsb + nn * 2048);
          bt1 = *(const u4*)(ldsb + nn * 2048 + 1024);
        }
        if (t == 14) r0 = lds_ld(ctl + C_QOFF + n + 1);
        if (t == 30) r0 = lds_ld(ctl + C_RED);
        __builtin_amdgcn_sched_barrier(0);
      }
      int red = __builtin_amdgcn_readfirstlane(r0);
      while (red < n - 1) {
        __builtin_amdgcn_s_sleep(1);
        if ((__builtin_amdgcn_readfirstlane(lds_ld(ctl + C_MASK)) >> F_ABORT) & 1u) return;
        red = __builtin_amdgcn_readfirstlane(lds_ld(ctl + C_RED));
      }
      CBAR();
      char* pp = smem + L_PART + (n & 1) * 8192 + wid * 2048 + vl;
      *(f4*)pp = acc0;
      *(f4*)(pp + 1024) = acc1;
      CBAR();
      if (lane == 0) __hip_atomic_fetch_add(ctl + C_PCNT + (n & 1), 1u, __ATOMIC_RELAXED, __HIP_MEMORY_SCOPE_WORKGROUP);
      CBAR();
      cur = nb;
    }
  }
}

}

extern "C" void kernel_launch(int rank, int nranks, ncclComm_t comm,
                              hipStream_t stream,
                              void* const* d_in, const int* in_sizes, int n_in,
                              void* d_out, int out_size,
                              const int* in_dtypes, int out_dtype,
                              void* const* d_out_all,
                              void* const* d_ws_all, size_t ws_bytes) {
  if (nranks != NR || ws_bytes < (size_t)WS_USED + (1u << 20) || n_in < 2 || (unsigned)rank >= (unsigned)NR) {
    fprintf(stderr, "unsupported configuration\n");
    return;
  }
  Params p;
  p.x = (const float*)d_in[0];
  p.w = (const float*)d_in[1];
  int pos = 0;
  while (phys(pos) != rank) ++pos;
  for (int d = 0; d < NR; ++d) {
    p.ws[d] = (char*)d_ws_all[phys(pos ^ d)];
    p.out[d] = (float*)d_out_all[phys(pos ^ d)];
  }
  p.rank = pos;
  hipLaunchKernelGGL(ag_gemm_relu, dim3(256), dim3(512), 0, stream, p);
  HIP_CHECK(hipGetLastError());
}
```

```cpp
#include <hip/hip_runtime.h>
#include <rccl/rccl.h>
#include <cstdio>

#define HIP_CHECK(expr) do { hipError_t _e = (expr); if (_e != hipSuccess) { \
    fprintf(stderr, "HIP error %s at %s:%d\n", hipGetErrorString(_e), __FILE__, __LINE__); \
    return; } } while (0)

namespace {

typedef float f4 __attribute__((ext_vector_type(4)));
typedef unsigned u4 __attribute__((ext_vector_type(4)));
typedef _Float16 h8 __attribute__((ext_vector_type(8)));
typedef unsigned long long u64;
typedef __amdgpu_buffer_rsrc_t rsrc_t;

constexpr int NR = 8;
constexpr int NG = 7;
constexpr int ND = 22;
constexpr int NXB = 32 - ND;
constexpr int S1 = 32;
constexpr int ONE = 96;
constexpr int S0 = S1 + ONE;
constexpr int NSTEP = ND * 32;
constexpr int NWST = 256;
constexpr int E2 = NWST + NXB * 32;
constexpr int GATE_STEP = E2 - 12;
constexpr int QUIT_STEP = E2 - 16;
constexpr int FWIN = 2;
constexpr int T_STAGE = 50, T_STIER = 42, T_TIER = 120;
constexpr int RAWTAIL = 3;
static_assert(ND >= 20 && ND <= 24, "slot table");
static_assert(ONE == 96 && S0 <= NWST && GATE_STEP >= 0 && GATE_STEP < NSTEP, "sender configuration");

constexpr u64 DEADLINE = 100000000ull;

constexpr unsigned OFF_PL = 0;
constexpr unsigned OFF_TMO = 40960;
constexpr unsigned OFF_CNT = 45056;
constexpr unsigned OFF_CTR = 65536;
constexpr unsigned OFF_SL = 98304;
constexpr unsigned OFF_RP = 110592;
constexpr unsigned OFF_X = 1u << 20;
constexpr unsigned OFF_W = 65u << 20;
constexpr unsigned OFF_T = 81u << 20;
constexpr unsigned WS_USED = 87u << 20;
constexpr int TBYTES = 49152;
static_assert(112u * TBYTES <= WS_USED - OFF_T && 22 * 2176 <= TBYTES, "tile region");
constexpr int CI_TICK = 0;
constexpr int CI_SX = 4;
constexpr int CI_SW = 36;
constexpr int CI_LX = 52;
constexpr int CI_LW = 84;
constexpr int CI_ROLE = 148;
constexpr int CI_END = 164;
constexpr int NSLOT = 25;
constexpr int F_W = 25;
constexpr int F_GATE = 29;
constexpr int F_ABORT = 31;

constexpr int NLDSK = 14;
constexpr int NBACK = 15;
constexpr int NBOWN = 8;
constexpr int BPITCH = 2144;
constexpr int L_B = 0;
constexpr int L_PART = 4 * NLDSK * 2048;
constexpr int L_BACK = L_PART + 16384;
constexpr int L_CTL = L_BACK + NBACK * BPITCH;
constexpr int C_QOFF = 0;
constexpr int C_QSLOT = 24;
constexpr int C_PCNT = 48;
constexpr int C_RED = 50;
constexpr int C_FL = 51;
constexpr int C_MASK = 52;
constexpr int C_ROLE = 53;
constexpr int C_WORDS = 64;
constexpr int L_EXP = L_BACK + NBOWN * BPITCH;
constexpr int LDS_BYTES = L_CTL + C_WORDS * 4;
static_assert(LDS_BYTES <= 163840, "LDS");

__host__ __device__ constexpr int phys(int l) { return (0x75416320 >> (4 * l)) & 7; }

struct Params {
  const float* x;
  const float* w;
  char* ws[NR];
  float* out[NR];
  int rank;
};

#define WAITV0() do { asm volatile("" ::: "memory"); __builtin_amdgcn_s_waitcnt(0x0F70); asm volatile("" ::: "memory"); } while (0)
#define CBAR() asm volatile("" ::: "memory")
constexpr int OOB = 0x40000000;

__device__ __forceinline__ rsrc_t mk_rsrc(const void* p, unsigned bytes) {
  return __builtin_amdgcn_make_buffer_rsrc(const_cast<void*>(p), 0, bytes, 0x00020000);
}
__device__ __forceinline__ f4 ld_in(rsrc_t r, int voff, int soff) {
  return __builtin_bit_cast(f4, __builtin_amdgcn_raw_buffer_load_b128(r, voff, soff, 0));
}
__device__ __forceinline__ u4 ld_sys(rsrc_t r, int voff, int soff) {
  return __builtin_amdgcn_raw_buffer_load_b128(r, voff, soff, 17);
}
__device__ __forceinline__ unsigned ld_sys32(rsrc_t r, int voff, int soff) {
  return __builtin_amdgcn_raw_buffer_load_b32(r, voff, soff, 17);
}
__device__ __forceinline__ void st_sys(u4 v, rsrc_t r, int voff, int soff) {
  __builtin_amdgcn_raw_buffer_store_b128(v, r, voff, soff, 17);
}
__device__ __forceinline__ void st_sys32(unsigned v, rsrc_t r, int voff, int soff) {
  __builtin_amdgcn_raw_buffer_store_b32(v, r, voff, soff, 17);
}
__device__ __forceinline__ u4 cvt8(f4 a, f4 b) {
  h8 h = h8{(_Float16)a.x, (_Float16)a.y, (_Float16)a.z, (_Float16)a.w,
            (_Float16)b.x, (_Float16)b.y, (_Float16)b.z, (_Float16)b.w};
  return __builtin_bit_cast(u4, h);
}
__device__ __forceinline__ u4 cvtcol(const f4* r, int i) {
  h8 h = h8{(_Float16)r[0][i], (_Float16)r[1][i], (_Float16)r[2][i], (_Float16)r[3][i],
            (_Float16)r[4][i], (_Float16)r[5][i], (_Float16)r[6][i], (_Float16)r[7][i]};
  return __builtin_bit_cast(u4, h);
}
__device__ __forceinline__ void load_xpiece(f4* r, rsrc_t rx, int vx, int pi) {
  const int so = (pi >> 5) * (16 * 16384) + (pi & 31) * 512;
#pragma unroll
  for (int t = 0; t < 8; ++t) r[t] = ld_in(rx, vx + t * 64, so);
}
__device__ __forceinline__ void load_wgran(f4* r, rsrc_t rw, int vw, int cg, int kg) {
  const int so = kg * (64 * 2048) + cg * 128;
#pragma unroll
  for (int j = 0; j < 8; ++j) r[j] = ld_in(rw, vw + (j & 1) * 2048, so + (16 * (j >> 2) + (j & 2)) * 2048);
}
__device__ __forceinline__ void pause_ticks(int ticks) {
  const u64 t0 = __builtin_amdgcn_s_memrealtime();
  while (__builtin_amdgcn_s_memrealtime() - t0 < (u64)ticks) __builtin_amdgcn_s_sleep(2);
}
__device__ __forceinline__ unsigned lds_ld(const unsigned* p) {
  return __hip_atomic_load(p, __ATOMIC_RELAXED, __HIP_MEMORY_SCOPE_WORKGROUP);
}
__device__ __forceinline__ void lds_st(unsigned* p, unsigned v) {
  __hip_atomic_store(p, v, __ATOMIC_RELAXED, __HIP_MEMORY_SCOPE_WORKGROUP);
}

__device__ __forceinline__ int src_off(int d) { return (0x65314201 >> (4 * d)) & 7; }
__device__ __forceinline__ int src_sbase(int d) { return d == 0 ? 0 : d < 4 ? 1 + 4 * d : 9 + 2 * d; }
__device__ __forceinline__ int src_lo(int d) { return d < 4 ? 0 : ND; }
__device__ __forceinline__ int flag_off(int d, int b) {
  return (((b + src_off(d)) % NG) * 32 + src_sbase(d) + (b - src_lo(d)) / NG) * 4;
}
__device__ __forceinline__ bool slot_block(int slot, int g, bool helper, int q, int& d, int& b) {
  d = slot < 5 ? 0 : slot < 17 ? (slot - 1) >> 2 : ((slot - 9) >> 1) & 7;
  const int lo = src_lo(d);
  const int hi = d == 0 ? (helper ? ND : 32) : d < 4 ? ND : ((helper && d == 4 + q) ? 0 : 32);
  b = lo + (g + 70 - src_off(d) - lo) % NG + NG * (slot - src_sbase(d));
  return slot < NSLOT && b < hi;
}
__device__ __forceinline__ void stage_local(int gi, unsigned call, rsrc_t rws, rsrc_t rx, rsrc_t rw, rsrc_t rcnt, int vx, int vw, int lane) {
  f4 sx[8], sw[8];
  load_xpiece(sx, rx, vx, gi);
  load_wgran(sw, rw, vw, gi >> 6, gi & 63);
  const int xbase = OFF_X + (call & 1u) * (32u << 20);
  const int wbase = OFF_W + (call & 1u) * (8u << 20);
#pragma unroll
  for (int t = 0; t < 4; ++t) st_sys(cvtcol(sw, t), rws, lane * 16 + t * 1024, wbase + gi * 4096);
#pragma unroll
  for (int t = 0; t < 4; ++t) st_sys(cvt8(sx[2 * t], sx[2 * t + 1]), rws, lane * 16 + t * 1024, xbase + gi * 4096);
  WAITV0();
  const int tv = __builtin_amdgcn_raw_ptr_buffer_atomic_add_i32(1, rcnt, (lane == 0 ? CI_LX + (gi >> 5) : lane == 1 ? CI_LW + (gi >> 4) : CI_END) * 64, 0, 0);
  if ((__builtin_amdgcn_readlane(tv, 0) & 31) == 31)
    st_sys32(call, rws, lane < 8 ? lane * 1024 : OOB, OFF_SL + flag_off(0, gi >> 5));
  if ((__builtin_amdgcn_readlane(tv, 1) & 15) == 15)
    st_sys32(call, rws, lane < NG ? lane * 4096 : OOB, OFF_PL + (gi >> 6) * 128 + (F_W + ((gi >> 4) & 3)) * 4);
}

constexpr bool k_in_lds(int t) { return ((t & 15) & 1) && (t & 15) != 15; }
constexpr int lds_index(int t) { return (t >> 4) * 7 + ((t & 15) >> 1); }
constexpr int reg_index(int t) { return (t >> 4) * 9 + ((t & 15) == 15 ? 8 : ((t & 15) >> 1)); }

__global__ void __launch_bounds__(512, 2) ag_gemm_relu(Params p) {
  __shared__ __attribute__((aligned(16))) char smem[LDS_BYTES];
  unsigned* ctl = (unsigned*)(smem + L_CTL);
  const int lane = threadIdx.x & 63;
  const int wid = __builtin_amdgcn_readfirstlane(threadIdx.x >> 6);
  const bool comm = blockIdx.x < 32;
  unsigned xcc;
  asm volatile("s_getreg_b32 %0, hwreg(HW_REG_XCC_ID)" : "=s"(xcc));
  xcc &= 7u;
  const rsrc_t rws = mk_rsrc(p.ws[0], WS_USED);
  const rsrc_t rx = mk_rsrc(p.x, 512u * 4096u * 4u);
  const rsrc_t rw = mk_rsrc(p.w, 4096u * 512u * 4u);
  const rsrc_t rcnt = mk_rsrc(p.ws[0] + OFF_CNT, CI_END * 64);
  const int vx = (lane & 15) * 16384 + (lane >> 4) * 16;
  const int vw = (32 * (lane >> 5) + 4 * ((lane >> 3) & 3)) * 2048 + (lane & 7) * 16;
  const int vl = lane * 16;
  const int v0 = lane == 0 ? 0 : OOB;

  const unsigned cprev = ld_sys32(rws, 0, OFF_CTR + blockIdx.x * 64);

  const int j = blockIdx.x;
  bool sent = false;
  if (__builtin_expect(comm ? wid == 0 : wid == 6 && j < S0, 1)) {
    f4 rd[8], ra[8], rb[24];
    load_xpiece(rd, rx, vx, j);
    load_wgran(ra, rw, vw, j & 3, j >> 2);
#pragma unroll
    for (int t = 0; t < 3; ++t) load_wgran(rb + 8 * t, rw, vw, 4 * (t + 1) + (j & 3), j >> 2);
    asm volatile("" :: "s"(p.ws[1]), "s"(p.ws[2]), "s"(p.ws[3]), "s"(p.ws[4]), "s"(p.ws[5]), "s"(p.ws[6]), "s"(p.ws[7]));
    const unsigned call = __builtin_amdgcn_readfirstlane(cprev) + 1u;
    asm volatile("" :: "s"(call) : "memory");
    __builtin_amdgcn_s_barrier();
    CBAR();
    __builtin_amdgcn_s_setprio(2);
    const unsigned par = call & 1u;
    rsrc_t rp[NR];
#pragma unroll
    for (int d = 1; d < NR; ++d) rp[d] = mk_rsrc(p.ws[d], WS_USED);
    const int xbase = OFF_X + par * (32u << 20);
    const int wbase = OFF_W + par * (8u << 20);
    const int vfl = lane < 8 ? lane * 1024 : OOB;
    const int vgl = lane < NG ? lane * 4096 : OOB;
    const int qstep = !comm ? 0 : xcc >= 4u && j != 0 ? QUIT_STEP : NSTEP;
    int i = j, prev = -1;
    WAITV0();
    if (j >= S1 + 8) pause_ticks(((j - S1) >> 3) * T_TIER);
    for (;;) {
      const bool quit = i >= qstep;
      int cv = CI_END;
      if (lane == 0 && !quit) cv = CI_TICK + 2 * par;
      if (lane == 1 && prev >= 0) cv = CI_SX + (prev >> 5);
      if (lane == 2 && prev >= 0 && prev < E2) cv = prev < NWST ? CI_SW + (prev >> 6) * 4 + (prev & 3) : CI_SX + ND + ((prev - NWST) >> 5);
      const int tv = __builtin_amdgcn_raw_ptr_buffer_atomic_add_i32(1, rcnt, cv * 64, 0, 0);
      int t0v, t1v, t2v;
#define READ_BACK() do { t0v = __builtin_amdgcn_readlane(tv, 0); t1v = __builtin_amdgcn_readlane(tv, 1); t2v = __builtin_amdgcn_readlane(tv, 2); } while (0)
      if (i == GATE_STEP) {
        st_sys32(call, rws, (lane >> 4) * 4096 + (lane & 15) * 128, OFF_PL + 16 * 128 + F_GATE * 4);
        st_sys32(call, rws, lane < 48 ? (lane >> 4) * 4096 + (lane & 15) * 128 : OOB, OFF_PL + 4 * 4096 + 16 * 128 + F_GATE * 4);
      }
      if (i >= NSTEP) {
        READ_BACK();
      } else if (i < NWST) {
        const int wo = wbase + (16 + (i & 3)) * 262144 + (i >> 2) * 4096;
#pragma unroll
        for (int t = 0; t < 4; ++t) {
          const u4 h = cvt8(rd[2 * t], rd[2 * t + 1]);
#pragma unroll
          for (int d = 1; d < 4; ++d) st_sys(h, rp[d], vl + t * 1024, xbase + d * (4 << 20) + i * 4096);
          st_sys(cvtcol(ra, t), rp[4], vl + t * 1024, wo);
#pragma unroll
          for (int d = 5; d < 8; ++d) st_sys(cvtcol(rb + 8 * (d - 5), t), rp[d], vl + t * 1024, wo + (d - 4) * (4 * 262144));
        }
        READ_BACK();
      } else if (i < E2) {
        const int xi = NSTEP + i - NWST;
#pragma unroll
        for (int t = 0; t < 4; ++t) {
          const u4 h = cvt8(rd[2 * t], rd[2 * t + 1]);
          const u4 gg = cvt8(ra[2 * t], ra[2 * t + 1]);
#pragma unroll
          for (int d = 1; d < 4; ++d) st_sys(h, rp[d], vl + t * 1024, xbase + d * (4 << 20) + i * 4096);
#pragma unroll
          for (int d = 4; d < 8; ++d) st_sys(gg, rp[d], vl + t * 1024, xbase + d * (4 << 20) + xi * 4096);
        }
        READ_BACK();
      } else {
#pragma unroll
        for (int t = 0; t < 4; ++t) {
          const u4 h = cvt8(rd[2 * t], rd[2 * t + 1]);
#pragma unroll
          for (int d = 1; d < 4; ++d) st_sys(h, rp[d], vl + t * 1024, xbase + d * (4 << 20) + i * 4096);
        }
        READ_BACK();
      }
      if (prev >= 0) {
        if ((t1v & 31) == 31) {
#pragma unroll
          for (int d = 1; d < 4; ++d) st_sys32(call, rp[d], vfl, OFF_SL + flag_off(d, prev >> 5));
        }
        if (prev < NWST) {
          if ((t2v & 15) == 15) {
#pragma unroll
            for (int d = 4; d < 8; ++d)
              st_sys32(call, rp[d], vgl, OFF_PL + (16 + 4 * (d - 4) + (prev & 3)) * 128 + (F_W + (prev >> 6)) * 4);
          }
        } else if (prev < E2) {
          if ((t2v & 31) == 31) {
#pragma unroll
            for (int d = 4; d < 8; ++d) st_sys32(call, rp[d], vfl, OFF_SL + flag_off(d, ND + ((prev - NWST) >> 5)));
          }
        }
      }
      if (i >= NSTEP) break;
      int nxt = t0v + S0;
      if (quit || (unsigned)nxt >= (unsigned)NSTEP) nxt = NSTEP;
      if (nxt < NSTEP) {
        load_xpiece(rd, rx, vx, nxt);
        if (nxt < NWST) {
          load_wgran(ra, rw, vw, nxt & 3, nxt >> 2);
#pragma unroll
          for (int t = 0; t < 3; ++t) load_wgran(rb + 8 * t, rw, vw, 4 * (t + 1) + (nxt & 3), nxt >> 2);
        } else if (nxt < E2) {
          load_xpiece(ra, rx, vx, NSTEP + nxt - NWST);
        }
      }
      WAITV0();
      prev = i;
      i = nxt;
    }
    if (comm) return;
    sent = true;
  }

  const unsigned call = __builtin_amdgcn_readfirstlane(cprev) + 1u;
  if (!comm && wid == 4) {
    int role = 0;
    for (unsigned x = xcc; x < xcc + 8u; ++x) {
      const int t = __builtin_amdgcn_readfirstlane(__builtin_amdgcn_raw_ptr_buffer_atomic_add_i32(
          1, rcnt, lane == 0 ? 0 : CI_END * 64, (CI_ROLE + (call & 1u) * 8 + (x & 7u)) * 64, 0));
      if ((unsigned)t < 28u) {
        const int h = (x & 3u) * 28 + t;
        role = (h >> 4) * 32 + (h & 15) + (x & 4u) * 4;
        break;
      }
    }
    lds_st(ctl + lane, lane == C_ROLE ? (unsigned)role : 0u);
  }
  asm volatile("s_waitcnt lgkmcnt(0)" :: "s"(call) : "memory");
  if (!sent) __builtin_amdgcn_s_barrier();
  CBAR();
  if (comm) {
    if (wid == 7) {
      st_sys32(call, rws, v0, OFF_CTR + blockIdx.x * 64);
      if (blockIdx.x == 31)
        st_sys32(0u, rcnt, lane == 8 ? (CI_TICK + 2 * ((call & 1u) ^ 1u)) * 64 : lane < 8 ? (CI_ROLE + ((call & 1u) ^ 1u) * 8 + lane) * 64 : OOB, 0);
    }
    return;
  }
  const int cidx = __builtin_amdgcn_readfirstlane(lds_ld(ctl + C_ROLE));
  const int g = cidx >> 5;
  const int cu = cidx & 31;
  const bool helper = cu >= 16;
  const int q = (cu >> 2) & 3;

  if (wid == 6) {
    if (!sent) {
      pause_ticks(T_STAGE + ((blockIdx.x >> 3) - 4) * T_STIER);
      stage_local(896 + j - S0, call, rws, rx, rw, rcnt, vx, vw, lane);
    }
    if (helper) return;
    __builtin_amdgcn_s_setprio(2);
    const int z = q, sub = cu & 3, eg = g, e = z * 28 + sub * NG + g;
    int pd, pb;
    unsigned ppend = (unsigned)__ballot(slot_block(lane, eg, true, z, pd, pb));
    const int prow = (phys((p.rank ^ (4 + z) ^ pd) & 7) * 512 + pb * 16) * 2048;
    const rsrc_t ro = mk_rsrc(p.out[0], 4096u * 512u * 4u);
    const int vo = (lane >> 3) * 2048 + (lane & 7) * 16 + (4 * z + sub) * 128;
    const int tb = OFF_T + e * TBYTES;
    char* sc = smem + L_EXP;
    const int jr = (lane >> 3) & 3;
    const int sl0 = (lane >> 5) * 16 + (lane & 7);
    const int pv = lane == 0 ? (int)(OFF_RP + e * 128) : lane == 2 ? (int)OFF_TMO : OOB;
    int rl = 0, tried = 0, oc = 0;
    bool abort = false;
    const u64 t0 = __builtin_amdgcn_s_memrealtime();
    for (;;) {
      const unsigned fv = ld_sys32(rws, pv, 0);
      const unsigned ow = __builtin_amdgcn_readlane(fv, 0) - (call << 14);
      if (ow < 1024u && (int)ow > oc) oc = (int)ow;
      const int avail = (oc & 511) - rl;
      if (avail > tried) {
        const int got = avail < 64 ? avail : 64;
        u4 t[8];
#pragma unroll
        for (int y = 0; y < 8; ++y) t[y] = ld_sys(rws, (lane >> 3) + 8 * y < got ? vl : OOB, tb + rl * 128 + y * 1024);
#pragma unroll
        for (int y = 0; y < 8; ++y) *(u4*)(sc + y * 1024 + vl) = t[y];
        CBAR();
        int at = 0;
        while (at < got) {
          const char* h = sc + at * 128;
          const unsigned hd = *(const unsigned*)(h + 80 + (lane & 1) * 4);
          const u64 mk0 = *(const u64*)(h + jr * 8), mk1 = *(const u64*)(h + 32 + jr * 8);
          const int b0 = *(const unsigned short*)(h + 64 + jr * 2), b1 = *(const unsigned short*)(h + 72 + jr * 2);
          const int nl = (96 + 127 + 4 * (__builtin_amdgcn_readlane(hd, 1) & 1023)) >> 7;
          if (at + nl > got) break;
          const int sl = __builtin_amdgcn_readlane(hd, 0) & 31;
          const int ro_s = __builtin_amdgcn_readlane(prow, sl);
          ppend &= ~(1u << sl);
          unsigned o[2][4];
#pragma unroll
          for (int m = 0; m < 2; ++m) {
            const u64 mk = m ? mk1 : mk0;
            const int base = m ? b1 : b0;
#pragma unroll
            for (int hh = 0; hh < 2; ++hh) {
#pragma unroll
              for (int i1 = 0; i1 < 2; ++i1) {
                const int s = sl0 + hh * 32 + i1 * 8;
                const int idx = (base + __builtin_popcountll(mk & ((1ull << s) - 1ull))) & 511;
                const unsigned v = *(const unsigned*)(h + 96 + idx * 4);
                o[hh][m * 2 + i1] = ((mk >> s) & 1ull) ? v : 0u;
              }
            }
          }
          st_sys(u4{o[0][0], o[0][1], o[0][2], o[0][3]}, ro, vo, ro_s);
          st_sys(u4{o[1][0], o[1][1], o[1][2], o[1][3]}, ro, vo, ro_s + 16384);
          at += nl;
        }
        CBAR();
        tried = got == avail ? got - at : 0;
        if (tried == 0 && at == 0) { abort = true; break; }
        rl += at;
        continue;
      }
      if (avail == 0 && (oc & 512) != 0) break;
      if (__builtin_amdgcn_readlane(fv, 2) == call || __builtin_amdgcn_s_memrealtime() - t0 > DEADLINE) { abort = true; break; }
      if (oc == 0) __builtin_amdgcn_s_sleep(16); else __builtin_amdgcn_s_sleep(2);
    }
    if (abort) {
      st_sys32(call, rws, v0, OFF_TMO);
      ppend |= 1u;
      WAITV0();
      while (ppend != 0u) {
        const int ro_s = __builtin_amdgcn_readlane(prow, __builtin_ctz(ppend));
        ppend &= ppend - 1u;
        const u4 nan4 = u4{0x7fc00000u, 0x7fc00000u, 0x7fc00000u, 0x7fc00000u};
        st_sys(nan4, ro, vo, ro_s);
        st_sys(nan4, ro, vo, ro_s + 16384);
      }
    }
    return;
  }

  int sd, sb;
  const bool sused = slot_block(lane, g, helper, q, sd, sb);
  const unsigned needmask = (unsigned)__ballot(sused);
  const int nneed = __builtin_popcount(needmask);

  if (wid >= 4) {
    __builtin_amdgcn_s_setprio(2);
    const int npack = !helper ? 0 : nneed > RAWTAIL ? nneed - RAWTAIL : 0;

    if (wid == 4) {
      const unsigned want = needmask | (15u << F_W) | (helper ? 1u << F_GATE : 0u);
      const int pv = lane < NSLOT ? (int)(OFF_SL + ((cu >> 2) * 8 + g) * 128 + lane * 4)
                   : lane < 32 ? (int)(OFF_PL + cidx * 128 + lane * 4) : lane == 32 ? (int)OFF_TMO : OOB;
      const unsigned boff = OFF_X + (call & 1u) * (32u << 20) + sd * (4 << 20) + sb * 131072;
      unsigned seen = 0;
      const u64 t0 = __builtin_amdgcn_s_memrealtime();
      for (;;) {
        const unsigned fv = ld_sys32(rws, pv, 0);
        const unsigned m = (unsigned)__ballot((int)(fv - call) >= 0) & want;
        if (m != seen) {
          const unsigned fresh = m & ~seen & needmask;
          if (lane < 32 && ((fresh >> lane) & 1u)) {
            const int pos = __builtin_popcount(seen & needmask) + __builtin_popcount(fresh >> lane) - 1;
            lds_st(ctl + C_QSLOT + pos, (unsigned)lane);
            lds_st(ctl + C_QOFF + pos, boff);
          }
          if (lane == 0) lds_st(ctl + C_MASK, m);
          seen = m;
          if (m == want) break;
        }
        if (__builtin_amdgcn_readlane(fv, 32) == call || __builtin_amdgcn_s_memrealtime() - t0 > DEADLINE) {
          st_sys32(call, rws, v0, OFF_TMO);
          if (lane == 0) lds_st(ctl + C_MASK, seen | (1u << F_ABORT));
          break;
        }
        __builtin_amdgcn_s_sleep(1);
      }
    } else if (wid == 5) {
      unsigned pending = needmask;
      const int rowoff = (phys((p.rank ^ sd) & 7) * 512 + sb * 16) * 2048;
      const int vo = (lane >> 3) * 2048 + (lane & 7) * 16 + (cu & 15) * 128;
      const rsrc_t ro = mk_rsrc(!helper ? p.out[0] : q == 0 ? p.out[4] : q == 1 ? p.out[5] : q == 2 ? p.out[6] : p.out[7], 4096u * 512u * 4u);
      const int e = q * 28 + (cu & 3) * NG + g;
      const rsrc_t rq = mk_rsrc(q == 0 ? p.ws[4] : q == 1 ? p.ws[5] : q == 2 ? p.ws[6] : p.ws[7], WS_USED);
      const int fo = OFF_RP + e * 128;
      const int tb = OFF_T + e * TBYTES;
      const int nb = helper ? NBACK : NBOWN;
      int bi = 0, wl = 0, told = 0;
      bool abort = false, word_last = false;
      for (int n = 0; n < nneed;) {
        if (word_last) asm volatile("s_waitcnt vmcnt(1)" ::: "memory");
        else if (helper) asm volatile("s_waitcnt vmcnt(0)" ::: "memory");
        const int arrived = wl;
        unsigned c;
        int red;
        for (;;) {
          c = lds_ld(ctl + lane);
          const unsigned mk = __builtin_amdgcn_readlane(c, C_MASK);
          red = __builtin_amdgcn_readlane(c, C_RED);
          abort = (mk >> F_ABORT) & 1u;
          const bool hold = n >= npack && red < nneed && (mk & needmask) == needmask;
          if (abort || (red > n && (!helper || (((mk >> F_GATE) & 1u) && !hold)))) break;
          if (told != arrived) { st_sys32((call << 14) | (unsigned)arrived, rq, v0, fo); told = arrived; }
          __builtin_amdgcn_s_sleep(1);
        }
        if (abort) break;
        CBAR();
        const int nend = red < n + FWIN || n >= npack ? red : n + FWIN;
        for (; n < nend; ++n) {
          const int sl = __builtin_amdgcn_readlane(c, C_QSLOT + n);
          pending &= ~(1u << sl);
          const char* eb = smem + L_BACK + bi * BPITCH;
          const u4 o0 = *(const u4*)(eb + vl);
          const u4 o1 = *(const u4*)(eb + 1024 + vl);
          if (n >= npack) {
            const int ro_s = __builtin_amdgcn_readlane(rowoff, sl);
            st_sys(o0, ro, vo, ro_s);
            st_sys(o1, ro, vo, ro_s + 16384);
          } else {
            const u4 o2 = *(const u4*)(eb + 2048 + (lane & 7) * 16);
            const int nl = (96 + 127 + 4 * __builtin_amdgcn_readlane(o0.y, 5)) >> 7;
            st_sys(o0, rq, lane < 8 * nl ? vl : OOB, tb + wl * 128);
            if (nl > 8) st_sys(o1, rq, lane + 64 < 8 * nl ? vl : OOB, tb + wl * 128 + 1024);
            if (nl > 16) st_sys(o2, rq, lane < 8 ? vl : OOB, tb + wl * 128 + 2048);
            wl += nl;
          }
          bi = bi + 1 == nb ? 0 : bi + 1;
        }
        CBAR();
        word_last = told != arrived;
        if (word_last) { st_sys32((call << 14) | (unsigned)arrived, rq, v0, fo); told = arrived; }
        if (lane == 0) lds_st(ctl + C_FL, (unsigned)n);
      }
      if (helper || abort) WAITV0();
      if (abort) {
        pending |= 1u;
        while (pending != 0u) {
          const int ro_s = __builtin_amdgcn_readlane(rowoff, __builtin_ctz(pending));
          pending &= pending - 1u;
          const u4 nan4 = u4{0x7fc00000u, 0x7fc00000u, 0x7fc00000u, 0x7fc00000u};
          st_sys(nan4, ro, vo, ro_s);
          st_sys(nan4, ro, vo, ro_s + 16384);
        }
      } else if (helper) {
        st_sys32((call << 14) | 512u | (unsigned)wl, rq, v0, fo);
      }
    } else {
      st_sys32(call, rws, v0, OFF_CTR + blockIdx.x * 64);
      const int wo = (lane >> 4) * 512 + (lane & 7) * 16 + ((lane >> 3) & 1) * 4;
      const int nb = helper ? NBACK : NBOWN;
      int bi = 0;
      for (int n = 0; n < nneed; ++n) {
        bool abort;
        unsigned c;
        for (;;) {
          c = lds_ld(ctl + lane);
          abort = (__builtin_amdgcn_readlane(c, C_MASK) >> F_ABORT) & 1u;
          const int pc = (n & 1) ? __builtin_amdgcn_readlane(c, C_PCNT + 1) : __builtin_amdgcn_readlane(c, C_PCNT);
          if (abort || (pc >= 4 * ((n >> 1) + 1) && n - (int)__builtin_amdgcn_readlane(c, C_FL) < nb)) break;
          __builtin_amdgcn_s_sleep(1);
        }
        if (abort) break;
        CBAR();
        const char* pp = smem + L_PART + (n & 1) * 8192 + vl;
        char* e = smem + L_BACK + bi * BPITCH;
        float v[8];
#pragma unroll
        for (int m = 0; m < 2; ++m) {
          const f4 s = (*(const f4*)(pp + m * 1024) + *(const f4*)(pp + 2048 + m * 1024)) +
                       (*(const f4*)(pp + 4096 + m * 1024) + *(const f4*)(pp + 6144 + m * 1024));
#pragma unroll
          for (int jj = 0; jj < 4; ++jj) v[m * 4 + jj] = s[jj] < 0.f ? 0.f : s[jj];
        }
        if (n >= npack) {
#pragma unroll
          for (int r = 0; r < 8; ++r) *(float*)(e + wo + (r & 3) * 128 + (r >> 2) * 8) = v[r];
        } else {
          int base = 0, hb = 0;
          u64 hm = 0;
#pragma unroll
          for (int r = 0; r < 8; ++r) {
            const bool nz = __builtin_bit_cast(unsigned, v[r]) != 0u;
            const u64 mk = __ballot(nz);
            const int pos = base + __builtin_amdgcn_mbcnt_hi((unsigned)(mk >> 32), __builtin_amdgcn_mbcnt_lo((unsigned)mk, 0u));
            if (nz) *(float*)(e + 96 + pos * 4) = v[r];
            if (lane == r) { hm = mk; hb = base; }
            base += __builtin_popcountll(mk);
          }
          if (lane < 8) { *(u64*)(e + lane * 8) = hm; *(unsigned short*)(e + 64 + lane * 2) = (unsigned short)hb; }
          if (lane < 2) *(unsigned*)(e + 80 + lane * 4) = lane == 0 ? (unsigned)__builtin_amdgcn_readlane(c, C_QSLOT + n) : (unsigned)base;
        }
        CBAR();
        if (lane == 0) lds_st(ctl + C_RED, (unsigned)(n + 1));
        bi = bi + 1 == nb ? 0 : bi + 1;
      }
    }
    return;
  }

  {
    pause_ticks(T_STAGE + ((blockIdx.x >> 3) - 4) * T_STIER);
    stage_local(((blockIdx.x & 7) * 28 + (blockIdx.x >> 3) - 4) * 4 + wid, call, rws, rx, rw, rcnt, vx, vw, lane);
    const int wbase = OFF_W + (call & 1u) * (8u << 20);

    for (;;) {
      const unsigned mk = __builtin_amdgcn_readfirstlane(lds_ld(ctl + C_MASK));
      if ((mk >> F_ABORT) & 1u) return;
      if ((mk >> (F_W + wid)) & 1u) break;
      __builtin_amdgcn_s_sleep(4);
    }
    CBAR();
    int qo[8];
#pragma unroll
    for (int qd = 0; qd < 8; ++qd) qo[qd] = ((qd + cu) & 7) * 4096;
    const int vb = ((lane >> 3) & 1) * 1024 + (lane >> 4) * 128 + (lane & 7) * 16;
    u4 breg[18][2];
    char* ldsb = smem + L_B + wid * (NLDSK * 2048) + vl;
#pragma unroll
    for (int hf = 0; hf < 2; ++hf) {
      const int so = wbase + cu * 262144 + wid * 65536;
      u4 tmp[7][2];
#pragma unroll
      for (int t = 0; t < 16; ++t) {
#pragma unroll
        for (int m = 0; m < 2; ++m) {
          const u4 v = ld_sys(rws, vb + m * 2048 + (t & 1) * 512, so + 2 * qo[hf * 4 + (t >> 2)] + ((t >> 1) & 1) * 4096);
          if (k_in_lds(t)) tmp[t >> 1][m] = v; else breg[reg_index(hf * 16 + t)][m] = v;
        }
      }
#pragma unroll
      for (int t = 0; t < 7; ++t) {
#pragma unroll
        for (int m = 0; m < 2; ++m) *(u4*)(ldsb + (hf * 7 + t) * 2048 + m * 1024) = tmp[t][m];
      }
    }

    u4 ring[16];
    u4 bt0 = *(const u4*)ldsb, bt1 = *(const u4*)(ldsb + 1024);
    int va = vl + wid * 32768;
    int cur = (int)WS_USED;
    for (int n = 0; n < nneed; ++n) {
      if (cur == (int)WS_USED) {
        for (;;) {
          cur = __builtin_amdgcn_readfirstlane(lds_ld(ctl + C_QOFF + n));
          if (cur != 0) break;
          if ((__builtin_amdgcn_readfirstlane(lds_ld(ctl + C_MASK)) >> F_ABORT) & 1u) return;
          __builtin_amdgcn_s_sleep(1);
        }
        asm volatile("" : "+v"(va) :: "memory");
#pragma unroll
        for (int t = 0; t < 16; ++t) {
          ring[t] = ld_sys(rws, va + (t & 3) * 1024, cur + qo[t >> 2]);
          __builtin_amdgcn_sched_barrier(0);
        }
      }
      f4 acc0 = f4{0.f, 0.f, 0.f, 0.f}, acc1 = f4{0.f, 0.f, 0.f, 0.f};
      int nb = 0;
      unsigned r0 = 0;
      asm volatile("" : "+v"(va));
#pragma unroll
      for (int t = 0; t < 32; ++t) {
        if (t == 16) { nb = __builtin_amdgcn_readfirstlane(r0); nb = nb != 0 ? nb : (int)WS_USED; CBAR(); }
        const u4 b0 = k_in_lds(t) ? bt0 : breg[reg_index(t)][0];
        const u4 b1 = k_in_lds(t) ? bt1 : breg[reg_index(t)][1];
        const h8 a = __builtin_bit_cast(h8, ring[t & 15]);
        acc0 = __builtin_amdgcn_mfma_f32_16x16x32_f16(a, __builtin_bit_cast(h8, b0), acc0, 0, 0, 0);
        acc1 = __builtin_amdgcn_mfma_f32_16x16x32_f16(a, __builtin_bit_cast(h8, b1), acc1, 0, 0, 0);
        __builtin_amdgcn_sched_barrier(0);
        ring[t & 15] = ld_sys(rws, va + (t & 3) * 1024, (t < 16 ? cur : nb) + qo[((t + 16) & 31) >> 2]);
        if (k_in_lds(t)) {
          const int nn = (lds_index(t) + 1) % NLDSK;
          bt0 = *(const u4*)(ldsb + nn * 2048);
          bt1 = *(const u4*)(ldsb + nn * 2048 + 1024);
        }
        if (t == 14) r0 = lds_ld(ctl + C_QOFF + n + 1);
        if (t == 30) r0 = lds_ld(ctl + C_RED);
        __builtin_amdgcn_sched_barrier(0);
      }
      int red = __builtin_amdgcn_readfirstlane(r0);
      while (red < n - 1) {
        __builtin_amdgcn_s_sleep(1);
        if ((__builtin_amdgcn_readfirstlane(lds_ld(ctl + C_MASK)) >> F_ABORT) & 1u) return;
        red = __builtin_amdgcn_readfirstlane(lds_ld(ctl + C_RED));
      }
      CBAR();
      char* pp = smem + L_PART + (n & 1) * 8192 + wid * 2048 + vl;
      *(f4*)pp = acc0;
      *(f4*)(pp + 1024) = acc1;
      CBAR();
      if (lane == 0) __hip_atomic_fetch_add(ctl + C_PCNT + (n & 1), 1u, __ATOMIC_RELAXED, __HIP_MEMORY_SCOPE_WORKGROUP);
      CBAR();
      cur = nb;
    }
  }
}

}

extern "C" void kernel_launch(int rank, int nranks, ncclComm_t comm,
                              hipStream_t stream,
                              void* const* d_in, const int* in_sizes, int n_in,
                              void* d_out, int out_size,
                              const int* in_dtypes, int out_dtype,
                              void* const* d_out_all,
                              void* const* d_ws_all, size_t ws_bytes) {
  if (nranks != NR || ws_bytes < (size_t)WS_USED + (1u << 20) || n_in < 2 || (unsigned)rank >= (unsigned)NR) {
    fprintf(stderr, "unsupported configuration\n");
    return;
  }
  Params p;
  p.x = (const float*)d_in[0];
  p.w = (const float*)d_in[1];
  int pos = 0;
  while (phys(pos) != rank) ++pos;
  for (int d = 0; d < NR; ++d) {
    p.ws[d] = (char*)d_ws_all[phys(pos ^ d)];
    p.out[d] = (float*)d_out_all[phys(pos ^ d)];
  }
  p.rank = pos;
  hipLaunchKernelGGL(ag_gemm_relu, dim3(256), dim3(512), 0, stream, p);
  HIP_CHECK(hipGetLastError());
}
```
